```python
import numpy as np

kernel_name = "gpu_a2a_gemm_m4096_k4096_n32768_f32_gelu"


ABSMAX_RTOL = 0.001

COMPUTE_DTYPE = "float32"
EPILOGUE = "gelu"
IN_QUANT = None
OUT_QUANT = None
IN_DTYPES = {"x": "float32", "w_mat": "float32"}
WORLD_SIZE = 8

def setup_inputs():

    x = np.random.randn(4096, 4096).astype(np.float32)
    w_mat = (np.random.randn(4096, 32768) / np.sqrt(4096)).astype(np.float32)
    return {"x": x, "w_mat": w_mat}

def _epilogue(y):
    c = 0.7978845608028654
    return (0.5 * y * (1.0 + np.tanh(c * (y + 0.044715 * y ** 3)))).astype(np.float32)

def reference(x, w_mat):
    y = (x @ w_mat).astype(np.float32)
    return _epilogue(y)
```

```cpp
#include <hip/hip_runtime.h>
#include <rccl/rccl.h>
#include <cstdio>
#include <cstdint>

typedef _Float16 f16x8 __attribute__((ext_vector_type(8)));
typedef _Float16 f16x2 __attribute__((ext_vector_type(2)));
typedef float f32x4 __attribute__((ext_vector_type(4)));
typedef float f32x2 __attribute__((ext_vector_type(2)));
typedef unsigned u32x4 __attribute__((ext_vector_type(4)));

constexpr int NR = 8;
constexpr int M_LOC = 512, K_TOT = 4096, N_TOT = 32768, K_LOC = 512;
constexpr size_t SLOT_BYTES = 4u << 20;
constexpr size_t FLAG_OFF = 8u << 20;
constexpr size_t CNT_OFF = FLAG_OFF + 4096;
constexpr size_t TMO_OFF = FLAG_OFF + 16384;
constexpr size_t WS_NEED = TMO_OFF + 256;
constexpr unsigned long long DEADLINE = 50000000ull;

struct Params { const float* x; const float* w; float* out; unsigned char* ws[NR]; int rank; };

__global__ __launch_bounds__(512, 2) void fused_a2a_gemm(Params p) {
    const int tid = threadIdx.x, lane = tid & 63, wave = __builtin_amdgcn_readfirstlane(tid >> 6);
    const int b = blockIdx.x;
    unsigned char* wsl = p.ws[p.rank];
    unsigned* cntp = (unsigned*)(wsl + CNT_OFF) + b * 8 + wave;
    const unsigned call = __builtin_amdgcn_readfirstlane(__hip_atomic_load(cntp, __ATOMIC_RELAXED, __HIP_MEMORY_SCOPE_AGENT)) + 1u;
    const unsigned slot = call & 1u;
    const int c = lane & 15, q = lane >> 4;
    {
        const unsigned d = b & 7, j = b >> 3;
        const int pmj = j & 1, kk = j >> 1, ksA = kk >> 1, mt = (kk & 1) * 8 + wave;
        unsigned char* wsd = p.ws[d];
        for (int half = 0; half < 2; ++half) {
            const int ks = ksA + 8 * half;
            const float* src = p.x + (size_t)(d * 512 + pmj * 256 + mt * 16 + c) * K_LOC + ks * 32 + q * 8;
            f32x4 v0 = *(const f32x4*)src, v1 = *(const f32x4*)(src + 4);
            f16x8 h;
            h[0] = (_Float16)v0[0]; h[1] = (_Float16)v0[1]; h[2] = (_Float16)v0[2]; h[3] = (_Float16)v0[3];
            h[4] = (_Float16)v1[0]; h[5] = (_Float16)v1[1]; h[6] = (_Float16)v1[2]; h[7] = (_Float16)v1[3];
            size_t off = slot * SLOT_BYTES + (size_t)(p.rank * 8 + (ks >> 1)) * 65536 + pmj * 32768 + (ks & 1) * 16384 + mt * 1024 + lane * 16;
            *(f16x8*)(wsd + off) = h;
            asm volatile("s_waitcnt vmcnt(0)" ::: "memory");
            __syncthreads();
            if (tid == 0) {
                __builtin_amdgcn_fence(__ATOMIC_RELEASE, "");
                asm volatile("s_waitcnt vmcnt(0)" ::: "memory");
                unsigned* f = (unsigned*)(wsd + FLAG_OFF) + (p.rank * 2 + half) * 32 + j;
                __hip_atomic_store(f, call, __ATOMIC_RELAXED, __HIP_MEMORY_SCOPE_SYSTEM);
            }
        }
    }
    const int xcd = b & 7, idx = b >> 3, pm = idx & 1, pn = xcd * 16 + (idx >> 1);
    const int n0 = pn * 256 + wave * 32;
    f32x4 acc[16][2];
#pragma unroll
    for (int m = 0; m < 16; ++m) { acc[m][0] = (f32x4){0, 0, 0, 0}; acc[m][1] = (f32x4){0, 0, 0, 0}; }
    unsigned* tmo = (unsigned*)(wsl + TMO_OFF);
    for (int hc = 0; hc < 16; ++hc) {
        const int g0 = (p.rank * 8 + hc * 4) & 63;
        if (tid < 32) {
            unsigned* f = (unsigned*)(wsl + FLAG_OFF) + (g0 >> 2) * 32 + tid;
            unsigned long long t0 = __builtin_amdgcn_s_memrealtime();
            while (true) {
                unsigned v = __hip_atomic_load(f, __ATOMIC_RELAXED, __HIP_MEMORY_SCOPE_SYSTEM);
                if (__all(v >= call)) break;
                if (__hip_atomic_load(tmo, __ATOMIC_RELAXED, __HIP_MEMORY_SCOPE_AGENT) == call ||
                    __builtin_amdgcn_s_memrealtime() - t0 > DEADLINE) {
                    __hip_atomic_store(tmo, call, __ATOMIC_RELAXED, __HIP_MEMORY_SCOPE_AGENT);
                    break;
                }
                __builtin_amdgcn_s_sleep(2);
            }
            __builtin_amdgcn_fence(__ATOMIC_ACQUIRE, "");
        }
        __syncthreads();
        for (int s8 = 0; s8 < 8; ++s8) {
            const int g = g0 + (s8 >> 1), ks = s8 & 1;
            const int k0 = g * 64 + ks * 32;
            const float* wp = p.w + (size_t)(k0 + q * 8) * N_TOT + n0 + 2 * c;
            f32x2 wv[8];
#pragma unroll
            for (int jj = 0; jj < 8; ++jj) wv[jj] = *(const f32x2*)(wp + (size_t)jj * N_TOT);
            f16x8 bf[2];
#pragma unroll
            for (int jj = 0; jj < 8; ++jj) { bf[0][jj] = (_Float16)wv[jj][0]; bf[1][jj] = (_Float16)wv[jj][1]; }
            const unsigned char* ap = wsl + slot * SLOT_BYTES + (size_t)g * 65536 + pm * 32768 + ks * 16384 + lane * 16;
#pragma unroll
            for (int m = 0; m < 16; ++m) {
                f16x8 af = *(const f16x8*)(ap + m * 1024);
                acc[m][0] = __builtin_amdgcn_mfma_f32_16x16x32_f16(af, bf[0], acc[m][0], 0, 0, 0);
                acc[m][1] = __builtin_amdgcn_mfma_f32_16x16x32_f16(af, bf[1], acc[m][1], 0, 0, 0);
            }
        }
    }
    const bool bad = __hip_atomic_load(tmo, __ATOMIC_RELAXED, __HIP_MEMORY_SCOPE_AGENT) == call;
    const float c0 = -2.0f * 0.7978845608028654f * 1.4426950408889634f, c1 = c0 * 0.044715f;
#pragma unroll
    for (int m = 0; m < 16; ++m)
#pragma unroll
        for (int r = 0; r < 4; ++r) {
            f32x2 o;
#pragma unroll
            for (int i = 0; i < 2; ++i) {
                float y = acc[m][i][r];
                float e = __builtin_amdgcn_exp2f(y * (y * y * c1 + c0));
                o[i] = y * __builtin_amdgcn_rcpf(1.0f + e);
                if (bad) o[i] = __builtin_nanf("");
            }
            *(f32x2*)(p.out + (size_t)(pm * 256 + m * 16 + q * 4 + r) * N_TOT + n0 + 2 * c) = o;
        }
    if (lane == 0) __hip_atomic_store(cntp, call, __ATOMIC_RELAXED, __HIP_MEMORY_SCOPE_AGENT);
}

extern "C" void kernel_launch(int rank, int nranks, ncclComm_t comm, hipStream_t stream,
                              void* const* d_in, const int* in_sizes, int n_in, void* d_out, int out_size,
                              const int* in_dtypes, int out_dtype, void* const* d_out_all,
                              void* const* d_ws_all, size_t ws_bytes) {
    if (nranks != NR || ws_bytes < WS_NEED || n_in != 2 || (unsigned)rank >= (unsigned)NR) { fprintf(stderr, "bad config\n"); return; }
    Params p;
    p.x = (const float*)d_in[0]; p.w = (const float*)d_in[1]; p.out = (float*)d_out; p.rank = rank;
    for (int i = 0; i < NR; ++i) p.ws[i] = (unsigned char*)d_ws_all[i];
    fused_a2a_gemm<<<256, 512, 0, stream>>>(p);
}
```

```cpp
#include <hip/hip_runtime.h>
#include <rccl/rccl.h>
#include <cstdio>
#include <cstdint>

constexpr int NR = 8;
constexpr size_t WS_NEED = (8u << 20) + 65536;
struct Params { const float* x; const float* w; float* out; unsigned char* ws[NR]; int rank; };

__global__ __launch_bounds__(256, 1) void fused_a2a_gemm(Params p) {
    __shared__ __attribute__((aligned(1024))) unsigned char smem[131328];
    const unsigned wave = __builtin_amdgcn_readfirstlane(threadIdx.x >> 6);
    const unsigned bid = blockIdx.x;
    unsigned char* wsl = p.ws[p.rank & 7];
    unsigned char* wsd = p.ws[bid & 7];
    const unsigned lds = (unsigned)(uintptr_t)(__attribute__((address_space(3))) unsigned char*)smem;
    asm volatile(
        "s_mov_b32 s83, %[wave]\n"
        "s_mov_b32 s85, %[bid]\n"
        "s_mov_b32 s84, %[rank]\n"
        "s_mov_b32 s87, %[lds]\n"
        "s_mov_b64 s[56:57], %[wsl]\n"
        "s_add_u32 s56, s56, 0x800000\n"
        "s_addc_u32 s57, s57, 0\n"
        "s_and_b32 s57, s57, 0xffff\n"
        "s_mov_b32 s58, 0x10000\n"
        "s_mov_b32 s59, 0x00020000\n"
        "s_mov_b64 s[48:49], %[x]\n"
        "s_and_b32 s49, s49, 0xffff\n"
        "s_mov_b32 s50, 0x800000\n"
        "s_mov_b32 s51, 0x00020000\n"
        "v_mbcnt_lo_u32_b32 v15, -1, 0\n"
        "v_mbcnt_hi_u32_b32 v15, -1, v15\n"
        "v_lshlrev_b32 v1, 4, v15\n"
        "v_and_b32 v13, 15, v15\n"
        "v_lshrrev_b32 v14, 4, v15\n"
        "s_and_b32 s81, s85, 7\n"
        "s_lshr_b32 s82, s85, 3\n"
        "s_lshl_b32 s80, s81, 5\n"
        "s_xor_b32 s70, s82, 0\n"
        "s_add_u32 s80, s80, s70\n"
        "s_lshl_b32 s86, s85, 3\n"
        "s_add_u32 s86, s86, s83\n"
        "s_lshl_b32 s86, s86, 2\n"
        "s_add_u32 s86, s86, 4096\n"
        "v_mov_b32 v3, 0\n"
        "buffer_load_dword v4, v3, s[56:59], s86 offen sc1\n"
        "v_lshlrev_b32 v2, 11, v13\n"
        "v_lshl_add_u32 v2, v14, 5, v2\n"
        "s_lshr_b32 s89, s82, 2\n"
        "s_and_b32 s88, s82, 3\n"
        "s_lshl_b32 s88, s88, 3\n"
        "s_lshl_b32 s70, s83, 1\n"
        "s_add_u32 s88, s88, s70\n"
        "s_lshl_b32 s70, s81, 9\n"
        "s_lshl_b32 s72, s88, 4\n"
        "s_add_u32 s70, s70, s72\n"
        "s_lshl_b32 s70, s70, 11\n"
        "s_lshl_b32 s72, s89, 7\n"
        "s_add_u32 s90, s70, s72\n"
        "s_add_u32 s70, s90, 0\n"
        "buffer_load_dwordx4 v[16:19], v2, s[48:51], s70 offen\n"
        "buffer_load_dwordx4 v[20:23], v2, s[48:51], s70 offen offset:16\n"
        "s_add_u32 s70, s90, 32768\n"
        "buffer_load_dwordx4 v[24:27], v2, s[48:51], s70 offen\n"
        "buffer_load_dwordx4 v[28:31], v2, s[48:51], s70 offen offset:16\n"
        "s_add_u32 s70, s90, 1024\n"
        "buffer_load_dwordx4 v[32:35], v2, s[48:51], s70 offen\n"
        "buffer_load_dwordx4 v[36:39], v2, s[48:51], s70 offen offset:16\n"
        "s_add_u32 s70, s90, 33792\n"
        "buffer_load_dwordx4 v[40:43], v2, s[48:51], s70 offen\n"
        "buffer_load_dwordx4 v[44:47], v2, s[48:51], s70 offen offset:16\n"
        "s_mov_b64 s[36:37], %[w]\n"
        "s_and_b32 s37, s37, 0xffff\n"
        "s_mov_b32 s38, 0x20000000\n"
        "s_mov_b32 s39, 0x00020000\n"
        "s_mov_b64 s[40:41], %[wsl]\n"
        "s_and_b32 s41, s41, 0xffff\n"
        "s_mov_b32 s42, 0x800000\n"
        "s_mov_b32 s43, 0x00020000\n"
        "s_mov_b64 s[96:97], %[wsl]\n"
        "s_and_b32 s97, s97, 0xffff\n"
        "s_mov_b32 s98, 0x800000\n"
        "s_mov_b32 s99, 0x00020000\n"
        "s_mov_b64 s[44:45], %[out]\n"
        "s_and_b32 s45, s45, 0xffff\n"
        "s_mov_b32 s46, 0x4000000\n"
        "s_mov_b32 s47, 0x00020000\n"
        "s_mov_b64 s[52:53], %[wsd]\n"
        "s_and_b32 s53, s53, 0xffff\n"
        "s_mov_b32 s54, 0x810000\n"
        "s_mov_b32 s55, 0x00020000\n"
        "s_lshl_b32 s70, s80, 9\n"
        "s_lshl_b32 s71, s83, 7\n"
        "s_add_u32 s70, s70, s71\n"
        "v_lshlrev_b32 v14, 20, v14\n"
        "v_lshl_add_u32 v5, v13, 3, v14\n"
        "v_add_u32 v5, s70, v5\n"
        "v_add_u32 v6, 131072, v5\n"
        "v_add_u32 v7, 262144, v5\n"
        "v_add_u32 v8, 393216, v5\n"
        "v_add_u32 v9, 524288, v5\n"
        "v_add_u32 v10, 655360, v5\n"
        "v_add_u32 v11, 786432, v5\n"
        "v_add_u32 v12, 917504, v5\n"
        "s_mov_b32 s79, 0\n"
        "s_add_u32 s70, s84, 0\n"
        "s_and_b32 s70, s70, 7\n"
        "s_lshl_b32 s70, s70, 0\n"
        "s_or_b32 s79, s79, s70\n"
        "s_add_u32 s70, s84, 1\n"
        "s_and_b32 s70, s70, 7\n"
        "s_lshl_b32 s70, s70, 4\n"
        "s_or_b32 s79, s79, s70\n"
        "s_add_u32 s70, s84, 2\n"
        "s_and_b32 s70, s70, 7\n"
        "s_lshl_b32 s70, s70, 8\n"
        "s_or_b32 s79, s79, s70\n"
        "s_add_u32 s70, s84, 3\n"
        "s_and_b32 s70, s70, 7\n"
        "s_lshl_b32 s70, s70, 12\n"
        "s_or_b32 s79, s79, s70\n"
        "s_add_u32 s70, s84, 4\n"
        "s_and_b32 s70, s70, 7\n"
        "s_lshl_b32 s70, s70, 16\n"
        "s_or_b32 s79, s79, s70\n"
        "s_add_u32 s70, s84, 5\n"
        "s_and_b32 s70, s70, 7\n"
        "s_lshl_b32 s70, s70, 20\n"
        "s_or_b32 s79, s79, s70\n"
        "s_add_u32 s70, s84, 6\n"
        "s_and_b32 s70, s70, 7\n"
        "s_lshl_b32 s70, s70, 24\n"
        "s_or_b32 s79, s79, s70\n"
        "s_add_u32 s70, s84, 7\n"
        "s_and_b32 s70, s70, 7\n"
        "s_lshl_b32 s70, s70, 28\n"
        "s_or_b32 s79, s79, s70\n"
        "s_lshl_b32 s29, 1, s84\n"
        "s_mov_b32 s67, 0\n"
        "s_lshl_b32 s70, s84, 8\n"
        "s_lshl_b32 s71, s82, 2\n"
        "s_add_u32 s70, s70, s71\n"
        "s_add_u32 s28, s70, 0x800000\n"
        "s_lshl_b32 s70, s84, 4\n"
        "s_add_u32 s70, s70, s89\n"
        "s_lshl_b32 s70, s70, 15\n"
        "s_lshl_b32 s71, s88, 10\n"
        "s_add_u32 s31, s70, s71\n"
        "s_mov_b32 s24, 255\n"
        "s_mov_b32 s25, 255\n"
        "s_mov_b32 s26, 255\n"
        "s_mov_b32 s27, 255\n"
        "s_cmp_lg_u32 s81, s84\n"
        "s_cbranch_scc1 L_defer_%=\n"
        "s_waitcnt vmcnt(0)\n"
        "v_readfirstlane_b32 s60, v4\n"
        "v_cvt_pk_f16_f32 v128, v16, v17\n"
        "v_cvt_pk_f16_f32 v129, v18, v19\n"
        "v_cvt_pk_f16_f32 v130, v20, v21\n"
        "v_cvt_pk_f16_f32 v131, v22, v23\n"
        "v_cvt_pk_f16_f32 v132, v24, v25\n"
        "v_cvt_pk_f16_f32 v133, v26, v27\n"
        "v_cvt_pk_f16_f32 v134, v28, v29\n"
        "v_cvt_pk_f16_f32 v135, v30, v31\n"
        "v_cvt_pk_f16_f32 v136, v32, v33\n"
        "v_cvt_pk_f16_f32 v137, v34, v35\n"
        "v_cvt_pk_f16_f32 v138, v36, v37\n"
        "v_cvt_pk_f16_f32 v139, v38, v39\n"
        "v_cvt_pk_f16_f32 v140, v40, v41\n"
        "v_cvt_pk_f16_f32 v141, v42, v43\n"
        "v_cvt_pk_f16_f32 v142, v44, v45\n"
        "v_cvt_pk_f16_f32 v143, v46, v47\n"
        "s_add_u32 s60, s60, 1\n"
        "s_and_b32 s70, s60, 1\n"
        "s_lshl_b32 s78, s70, 22\n"
        "s_add_u32 s31, s31, s78\n"
        "s_add_u32 s70, s31, 0\n"
        "buffer_store_dwordx4 v[128:131], v1, s[52:55], s70 offen sc0 sc1\n"
        "s_add_u32 s70, s31, 1024\n"
        "buffer_store_dwordx4 v[132:135], v1, s[52:55], s70 offen sc0 sc1\n"
        "s_add_u32 s70, s31, 262144\n"
        "buffer_store_dwordx4 v[136:139], v1, s[52:55], s70 offen sc0 sc1\n"
        "s_add_u32 s70, s31, 263168\n"
        "buffer_store_dwordx4 v[140:143], v1, s[52:55], s70 offen sc0 sc1\n"
        "s_waitcnt vmcnt(0)\n"
        "s_barrier\n"
        "s_cmp_gt_u32 s83, 1\n"
        "s_cbranch_scc1 L_nfu1_%=\n"
        "s_lshl_b32 s72, s83, 7\n"
        "s_add_u32 s72, s72, s28\n"
        "v_mov_b32 v14, s60\n"
        "v_mov_b32 v13, 0\n"
        "s_mov_b64 exec, 1\n"
        "buffer_store_dword v14, v13, s[52:55], s72 offen sc0 sc1\n"
        "s_mov_b64 exec, -1\n"
        "L_nfu1_%=:\n"
        "s_mov_b32 s67, 0\n"
        "s_lshr_b32 s62, s67, 2\n"
        "s_and_b32 s62, s62, 28\n"
        "s_lshr_b32 s62, s79, s62\n"
        "s_and_b32 s62, s62, 7\n"
        "s_lshl_b32 s62, s62, 4\n"
        "s_and_b32 s23, s67, 15\n"
        "s_or_b32 s62, s62, s23\n"
        "s_lshl_b32 s62, s62, 22\n"
        "s_add_u32 s67, s67, 1\n"
        "buffer_load_dwordx2 v[64:65], v5, s[36:39], s62 offen nt\n"
        "buffer_load_dwordx2 v[66:67], v6, s[36:39], s62 offen nt\n"
        "buffer_load_dwordx2 v[68:69], v7, s[36:39], s62 offen nt\n"
        "buffer_load_dwordx2 v[70:71], v8, s[36:39], s62 offen nt\n"
        "buffer_load_dwordx2 v[72:73], v9, s[36:39], s62 offen nt\n"
        "buffer_load_dwordx2 v[74:75], v10, s[36:39], s62 offen nt\n"
        "buffer_load_dwordx2 v[76:77], v11, s[36:39], s62 offen nt\n"
        "buffer_load_dwordx2 v[78:79], v12, s[36:39], s62 offen nt\n"
        "s_lshr_b32 s62, s67, 2\n"
        "s_and_b32 s62, s62, 28\n"
        "s_lshr_b32 s62, s79, s62\n"
        "s_and_b32 s62, s62, 7\n"
        "s_lshl_b32 s62, s62, 4\n"
        "s_and_b32 s23, s67, 15\n"
        "s_or_b32 s62, s62, s23\n"
        "s_lshl_b32 s62, s62, 22\n"
        "s_add_u32 s67, s67, 1\n"
        "buffer_load_dwordx2 v[80:81], v5, s[36:39], s62 offen nt\n"
        "buffer_load_dwordx2 v[82:83], v6, s[36:39], s62 offen nt\n"
        "buffer_load_dwordx2 v[84:85], v7, s[36:39], s62 offen nt\n"
        "buffer_load_dwordx2 v[86:87], v8, s[36:39], s62 offen nt\n"
        "buffer_load_dwordx2 v[88:89], v9, s[36:39], s62 offen nt\n"
        "buffer_load_dwordx2 v[90:91], v10, s[36:39], s62 offen nt\n"
        "buffer_load_dwordx2 v[92:93], v11, s[36:39], s62 offen nt\n"
        "buffer_load_dwordx2 v[94:95], v12, s[36:39], s62 offen nt\n"
        "s_lshr_b32 s62, s67, 2\n"
        "s_and_b32 s62, s62, 28\n"
        "s_lshr_b32 s62, s79, s62\n"
        "s_and_b32 s62, s62, 7\n"
        "s_lshl_b32 s62, s62, 4\n"
        "s_and_b32 s23, s67, 15\n"
        "s_or_b32 s62, s62, s23\n"
        "s_lshl_b32 s62, s62, 22\n"
        "s_add_u32 s67, s67, 1\n"
        "buffer_load_dwordx2 v[96:97], v5, s[36:39], s62 offen nt\n"
        "buffer_load_dwordx2 v[98:99], v6, s[36:39], s62 offen nt\n"
        "buffer_load_dwordx2 v[100:101], v7, s[36:39], s62 offen nt\n"
        "buffer_load_dwordx2 v[102:103], v8, s[36:39], s62 offen nt\n"
        "buffer_load_dwordx2 v[104:105], v9, s[36:39], s62 offen nt\n"
        "buffer_load_dwordx2 v[106:107], v10, s[36:39], s62 offen nt\n"
        "buffer_load_dwordx2 v[108:109], v11, s[36:39], s62 offen nt\n"
        "buffer_load_dwordx2 v[110:111], v12, s[36:39], s62 offen nt\n"
        "s_lshr_b32 s62, s67, 2\n"
        "s_and_b32 s62, s62, 28\n"
        "s_lshr_b32 s62, s79, s62\n"
        "s_and_b32 s62, s62, 7\n"
        "s_lshl_b32 s62, s62, 4\n"
        "s_and_b32 s23, s67, 15\n"
        "s_or_b32 s62, s62, s23\n"
        "s_lshl_b32 s62, s62, 22\n"
        "s_add_u32 s67, s67, 1\n"
        "buffer_load_dwordx2 v[112:113], v5, s[36:39], s62 offen nt\n"
        "buffer_load_dwordx2 v[114:115], v6, s[36:39], s62 offen nt\n"
        "buffer_load_dwordx2 v[116:117], v7, s[36:39], s62 offen nt\n"
        "buffer_load_dwordx2 v[118:119], v8, s[36:39], s62 offen nt\n"
        "buffer_load_dwordx2 v[120:121], v9, s[36:39], s62 offen nt\n"
        "buffer_load_dwordx2 v[122:123], v10, s[36:39], s62 offen nt\n"
        "buffer_load_dwordx2 v[124:125], v11, s[36:39], s62 offen nt\n"
        "buffer_load_dwordx2 v[126:127], v12, s[36:39], s62 offen nt\n"
        "v_accvgpr_write_b32 a0, 0\n"
        "v_accvgpr_write_b32 a1, 0\n"
        "v_accvgpr_write_b32 a2, 0\n"
        "v_accvgpr_write_b32 a3, 0\n"
        "v_accvgpr_write_b32 a4, 0\n"
        "v_accvgpr_write_b32 a5, 0\n"
        "v_accvgpr_write_b32 a6, 0\n"
        "v_accvgpr_write_b32 a7, 0\n"
        "v_accvgpr_write_b32 a8, 0\n"
        "v_accvgpr_write_b32 a9, 0\n"
        "v_accvgpr_write_b32 a10, 0\n"
        "v_accvgpr_write_b32 a11, 0\n"
        "v_accvgpr_write_b32 a12, 0\n"
        "v_accvgpr_write_b32 a13, 0\n"
        "v_accvgpr_write_b32 a14, 0\n"
        "v_accvgpr_write_b32 a15, 0\n"
        "v_accvgpr_write_b32 a16, 0\n"
        "v_accvgpr_write_b32 a17, 0\n"
        "v_accvgpr_write_b32 a18, 0\n"
        "v_accvgpr_write_b32 a19, 0\n"
        "v_accvgpr_write_b32 a20, 0\n"
        "v_accvgpr_write_b32 a21, 0\n"
        "v_accvgpr_write_b32 a22, 0\n"
        "v_accvgpr_write_b32 a23, 0\n"
        "v_accvgpr_write_b32 a24, 0\n"
        "v_accvgpr_write_b32 a25, 0\n"
        "v_accvgpr_write_b32 a26, 0\n"
        "v_accvgpr_write_b32 a27, 0\n"
        "v_accvgpr_write_b32 a28, 0\n"
        "v_accvgpr_write_b32 a29, 0\n"
        "v_accvgpr_write_b32 a30, 0\n"
        "v_accvgpr_write_b32 a31, 0\n"
        "v_accvgpr_write_b32 a32, 0\n"
        "v_accvgpr_write_b32 a33, 0\n"
        "v_accvgpr_write_b32 a34, 0\n"
        "v_accvgpr_write_b32 a35, 0\n"
        "v_accvgpr_write_b32 a36, 0\n"
        "v_accvgpr_write_b32 a37, 0\n"
        "v_accvgpr_write_b32 a38, 0\n"
        "v_accvgpr_write_b32 a39, 0\n"
        "v_accvgpr_write_b32 a40, 0\n"
        "v_accvgpr_write_b32 a41, 0\n"
        "v_accvgpr_write_b32 a42, 0\n"
        "v_accvgpr_write_b32 a43, 0\n"
        "v_accvgpr_write_b32 a44, 0\n"
        "v_accvgpr_write_b32 a45, 0\n"
        "v_accvgpr_write_b32 a46, 0\n"
        "v_accvgpr_write_b32 a47, 0\n"
        "v_accvgpr_write_b32 a48, 0\n"
        "v_accvgpr_write_b32 a49, 0\n"
        "v_accvgpr_write_b32 a50, 0\n"
        "v_accvgpr_write_b32 a51, 0\n"
        "v_accvgpr_write_b32 a52, 0\n"
        "v_accvgpr_write_b32 a53, 0\n"
        "v_accvgpr_write_b32 a54, 0\n"
        "v_accvgpr_write_b32 a55, 0\n"
        "v_accvgpr_write_b32 a56, 0\n"
        "v_accvgpr_write_b32 a57, 0\n"
        "v_accvgpr_write_b32 a58, 0\n"
        "v_accvgpr_write_b32 a59, 0\n"
        "v_accvgpr_write_b32 a60, 0\n"
        "v_accvgpr_write_b32 a61, 0\n"
        "v_accvgpr_write_b32 a62, 0\n"
        "v_accvgpr_write_b32 a63, 0\n"
        "v_accvgpr_write_b32 a64, 0\n"
        "v_accvgpr_write_b32 a65, 0\n"
        "v_accvgpr_write_b32 a66, 0\n"
        "v_accvgpr_write_b32 a67, 0\n"
        "v_accvgpr_write_b32 a68, 0\n"
        "v_accvgpr_write_b32 a69, 0\n"
        "v_accvgpr_write_b32 a70, 0\n"
        "v_accvgpr_write_b32 a71, 0\n"
        "v_accvgpr_write_b32 a72, 0\n"
        "v_accvgpr_write_b32 a73, 0\n"
        "v_accvgpr_write_b32 a74, 0\n"
        "v_accvgpr_write_b32 a75, 0\n"
        "v_accvgpr_write_b32 a76, 0\n"
        "v_accvgpr_write_b32 a77, 0\n"
        "v_accvgpr_write_b32 a78, 0\n"
        "v_accvgpr_write_b32 a79, 0\n"
        "v_accvgpr_write_b32 a80, 0\n"
        "v_accvgpr_write_b32 a81, 0\n"
        "v_accvgpr_write_b32 a82, 0\n"
        "v_accvgpr_write_b32 a83, 0\n"
        "v_accvgpr_write_b32 a84, 0\n"
        "v_accvgpr_write_b32 a85, 0\n"
        "v_accvgpr_write_b32 a86, 0\n"
        "v_accvgpr_write_b32 a87, 0\n"
        "v_accvgpr_write_b32 a88, 0\n"
        "v_accvgpr_write_b32 a89, 0\n"
        "v_accvgpr_write_b32 a90, 0\n"
        "v_accvgpr_write_b32 a91, 0\n"
        "v_accvgpr_write_b32 a92, 0\n"
        "v_accvgpr_write_b32 a93, 0\n"
        "v_accvgpr_write_b32 a94, 0\n"
        "v_accvgpr_write_b32 a95, 0\n"
        "v_accvgpr_write_b32 a96, 0\n"
        "v_accvgpr_write_b32 a97, 0\n"
        "v_accvgpr_write_b32 a98, 0\n"
        "v_accvgpr_write_b32 a99, 0\n"
        "v_accvgpr_write_b32 a100, 0\n"
        "v_accvgpr_write_b32 a101, 0\n"
        "v_accvgpr_write_b32 a102, 0\n"
        "v_accvgpr_write_b32 a103, 0\n"
        "v_accvgpr_write_b32 a104, 0\n"
        "v_accvgpr_write_b32 a105, 0\n"
        "v_accvgpr_write_b32 a106, 0\n"
        "v_accvgpr_write_b32 a107, 0\n"
        "v_accvgpr_write_b32 a108, 0\n"
        "v_accvgpr_write_b32 a109, 0\n"
        "v_accvgpr_write_b32 a110, 0\n"
        "v_accvgpr_write_b32 a111, 0\n"
        "v_accvgpr_write_b32 a112, 0\n"
        "v_accvgpr_write_b32 a113, 0\n"
        "v_accvgpr_write_b32 a114, 0\n"
        "v_accvgpr_write_b32 a115, 0\n"
        "v_accvgpr_write_b32 a116, 0\n"
        "v_accvgpr_write_b32 a117, 0\n"
        "v_accvgpr_write_b32 a118, 0\n"
        "v_accvgpr_write_b32 a119, 0\n"
        "v_accvgpr_write_b32 a120, 0\n"
        "v_accvgpr_write_b32 a121, 0\n"
        "v_accvgpr_write_b32 a122, 0\n"
        "v_accvgpr_write_b32 a123, 0\n"
        "v_accvgpr_write_b32 a124, 0\n"
        "v_accvgpr_write_b32 a125, 0\n"
        "v_accvgpr_write_b32 a126, 0\n"
        "v_accvgpr_write_b32 a127, 0\n"
        "s_branch L_pushed_%=\n"
        "L_defer_%=:\n"
        "s_mov_b32 s67, 0\n"
        "s_lshr_b32 s62, s67, 2\n"
        "s_and_b32 s62, s62, 28\n"
        "s_lshr_b32 s62, s79, s62\n"
        "s_and_b32 s62, s62, 7\n"
        "s_lshl_b32 s62, s62, 4\n"
        "s_and_b32 s23, s67, 15\n"
        "s_or_b32 s62, s62, s23\n"
        "s_lshl_b32 s62, s62, 22\n"
        "s_add_u32 s67, s67, 1\n"
        "buffer_load_dwordx2 v[64:65], v5, s[36:39], s62 offen nt\n"
        "buffer_load_dwordx2 v[66:67], v6, s[36:39], s62 offen nt\n"
        "buffer_load_dwordx2 v[68:69], v7, s[36:39], s62 offen nt\n"
        "buffer_load_dwordx2 v[70:71], v8, s[36:39], s62 offen nt\n"
        "buffer_load_dwordx2 v[72:73], v9, s[36:39], s62 offen nt\n"
        "buffer_load_dwordx2 v[74:75], v10, s[36:39], s62 offen nt\n"
        "buffer_load_dwordx2 v[76:77], v11, s[36:39], s62 offen nt\n"
        "buffer_load_dwordx2 v[78:79], v12, s[36:39], s62 offen nt\n"
        "s_lshr_b32 s62, s67, 2\n"
        "s_and_b32 s62, s62, 28\n"
        "s_lshr_b32 s62, s79, s62\n"
        "s_and_b32 s62, s62, 7\n"
        "s_lshl_b32 s62, s62, 4\n"
        "s_and_b32 s23, s67, 15\n"
        "s_or_b32 s62, s62, s23\n"
        "s_lshl_b32 s62, s62, 22\n"
        "s_add_u32 s67, s67, 1\n"
        "buffer_load_dwordx2 v[80:81], v5, s[36:39], s62 offen nt\n"
        "buffer_load_dwordx2 v[82:83], v6, s[36:39], s62 offen nt\n"
        "buffer_load_dwordx2 v[84:85], v7, s[36:39], s62 offen nt\n"
        "buffer_load_dwordx2 v[86:87], v8, s[36:39], s62 offen nt\n"
        "buffer_load_dwordx2 v[88:89], v9, s[36:39], s62 offen nt\n"
        "buffer_load_dwordx2 v[90:91], v10, s[36:39], s62 offen nt\n"
        "buffer_load_dwordx2 v[92:93], v11, s[36:39], s62 offen nt\n"
        "buffer_load_dwordx2 v[94:95], v12, s[36:39], s62 offen nt\n"
        "s_lshr_b32 s62, s67, 2\n"
        "s_and_b32 s62, s62, 28\n"
        "s_lshr_b32 s62, s79, s62\n"
        "s_and_b32 s62, s62, 7\n"
        "s_lshl_b32 s62, s62, 4\n"
        "s_and_b32 s23, s67, 15\n"
        "s_or_b32 s62, s62, s23\n"
        "s_lshl_b32 s62, s62, 22\n"
        "s_add_u32 s67, s67, 1\n"
        "buffer_load_dwordx2 v[96:97], v5, s[36:39], s62 offen nt\n"
        "buffer_load_dwordx2 v[98:99], v6, s[36:39], s62 offen nt\n"
        "buffer_load_dwordx2 v[100:101], v7, s[36:39], s62 offen nt\n"
        "buffer_load_dwordx2 v[102:103], v8, s[36:39], s62 offen nt\n"
        "buffer_load_dwordx2 v[104:105], v9, s[36:39], s62 offen nt\n"
        "buffer_load_dwordx2 v[106:107], v10, s[36:39], s62 offen nt\n"
        "buffer_load_dwordx2 v[108:109], v11, s[36:39], s62 offen nt\n"
        "buffer_load_dwordx2 v[110:111], v12, s[36:39], s62 offen nt\n"
        "s_lshr_b32 s62, s67, 2\n"
        "s_and_b32 s62, s62, 28\n"
        "s_lshr_b32 s62, s79, s62\n"
        "s_and_b32 s62, s62, 7\n"
        "s_lshl_b32 s62, s62, 4\n"
        "s_and_b32 s23, s67, 15\n"
        "s_or_b32 s62, s62, s23\n"
        "s_lshl_b32 s62, s62, 22\n"
        "s_add_u32 s67, s67, 1\n"
        "buffer_load_dwordx2 v[112:113], v5, s[36:39], s62 offen nt\n"
        "buffer_load_dwordx2 v[114:115], v6, s[36:39], s62 offen nt\n"
        "buffer_load_dwordx2 v[116:117], v7, s[36:39], s62 offen nt\n"
        "buffer_load_dwordx2 v[118:119], v8, s[36:39], s62 offen nt\n"
        "buffer_load_dwordx2 v[120:121], v9, s[36:39], s62 offen nt\n"
        "buffer_load_dwordx2 v[122:123], v10, s[36:39], s62 offen nt\n"
        "buffer_load_dwordx2 v[124:125], v11, s[36:39], s62 offen nt\n"
        "buffer_load_dwordx2 v[126:127], v12, s[36:39], s62 offen nt\n"
        "v_accvgpr_write_b32 a0, 0\n"
        "v_accvgpr_write_b32 a1, 0\n"
        "v_accvgpr_write_b32 a2, 0\n"
        "v_accvgpr_write_b32 a3, 0\n"
        "v_accvgpr_write_b32 a4, 0\n"
        "v_accvgpr_write_b32 a5, 0\n"
        "v_accvgpr_write_b32 a6, 0\n"
        "v_accvgpr_write_b32 a7, 0\n"
        "v_accvgpr_write_b32 a8, 0\n"
        "v_accvgpr_write_b32 a9, 0\n"
        "v_accvgpr_write_b32 a10, 0\n"
        "v_accvgpr_write_b32 a11, 0\n"
        "v_accvgpr_write_b32 a12, 0\n"
        "v_accvgpr_write_b32 a13, 0\n"
        "v_accvgpr_write_b32 a14, 0\n"
        "v_accvgpr_write_b32 a15, 0\n"
        "v_accvgpr_write_b32 a16, 0\n"
        "v_accvgpr_write_b32 a17, 0\n"
        "v_accvgpr_write_b32 a18, 0\n"
        "v_accvgpr_write_b32 a19, 0\n"
        "v_accvgpr_write_b32 a20, 0\n"
        "v_accvgpr_write_b32 a21, 0\n"
        "v_accvgpr_write_b32 a22, 0\n"
        "v_accvgpr_write_b32 a23, 0\n"
        "v_accvgpr_write_b32 a24, 0\n"
        "v_accvgpr_write_b32 a25, 0\n"
        "v_accvgpr_write_b32 a26, 0\n"
        "v_accvgpr_write_b32 a27, 0\n"
        "v_accvgpr_write_b32 a28, 0\n"
        "v_accvgpr_write_b32 a29, 0\n"
        "v_accvgpr_write_b32 a30, 0\n"
        "v_accvgpr_write_b32 a31, 0\n"
        "v_accvgpr_write_b32 a32, 0\n"
        "v_accvgpr_write_b32 a33, 0\n"
        "v_accvgpr_write_b32 a34, 0\n"
        "v_accvgpr_write_b32 a35, 0\n"
        "v_accvgpr_write_b32 a36, 0\n"
        "v_accvgpr_write_b32 a37, 0\n"
        "v_accvgpr_write_b32 a38, 0\n"
        "v_accvgpr_write_b32 a39, 0\n"
        "v_accvgpr_write_b32 a40, 0\n"
        "v_accvgpr_write_b32 a41, 0\n"
        "v_accvgpr_write_b32 a42, 0\n"
        "v_accvgpr_write_b32 a43, 0\n"
        "v_accvgpr_write_b32 a44, 0\n"
        "v_accvgpr_write_b32 a45, 0\n"
        "v_accvgpr_write_b32 a46, 0\n"
        "v_accvgpr_write_b32 a47, 0\n"
        "v_accvgpr_write_b32 a48, 0\n"
        "v_accvgpr_write_b32 a49, 0\n"
        "v_accvgpr_write_b32 a50, 0\n"
        "v_accvgpr_write_b32 a51, 0\n"
        "v_accvgpr_write_b32 a52, 0\n"
        "v_accvgpr_write_b32 a53, 0\n"
        "v_accvgpr_write_b32 a54, 0\n"
        "v_accvgpr_write_b32 a55, 0\n"
        "v_accvgpr_write_b32 a56, 0\n"
        "v_accvgpr_write_b32 a57, 0\n"
        "v_accvgpr_write_b32 a58, 0\n"
        "v_accvgpr_write_b32 a59, 0\n"
        "v_accvgpr_write_b32 a60, 0\n"
        "v_accvgpr_write_b32 a61, 0\n"
        "v_accvgpr_write_b32 a62, 0\n"
        "v_accvgpr_write_b32 a63, 0\n"
        "v_accvgpr_write_b32 a64, 0\n"
        "v_accvgpr_write_b32 a65, 0\n"
        "v_accvgpr_write_b32 a66, 0\n"
        "v_accvgpr_write_b32 a67, 0\n"
        "v_accvgpr_write_b32 a68, 0\n"
        "v_accvgpr_write_b32 a69, 0\n"
        "v_accvgpr_write_b32 a70, 0\n"
        "v_accvgpr_write_b32 a71, 0\n"
        "v_accvgpr_write_b32 a72, 0\n"
        "v_accvgpr_write_b32 a73, 0\n"
        "v_accvgpr_write_b32 a74, 0\n"
        "v_accvgpr_write_b32 a75, 0\n"
        "v_accvgpr_write_b32 a76, 0\n"
        "v_accvgpr_write_b32 a77, 0\n"
        "v_accvgpr_write_b32 a78, 0\n"
        "v_accvgpr_write_b32 a79, 0\n"
        "v_accvgpr_write_b32 a80, 0\n"
        "v_accvgpr_write_b32 a81, 0\n"
        "v_accvgpr_write_b32 a82, 0\n"
        "v_accvgpr_write_b32 a83, 0\n"
        "v_accvgpr_write_b32 a84, 0\n"
        "v_accvgpr_write_b32 a85, 0\n"
        "v_accvgpr_write_b32 a86, 0\n"
        "v_accvgpr_write_b32 a87, 0\n"
        "v_accvgpr_write_b32 a88, 0\n"
        "v_accvgpr_write_b32 a89, 0\n"
        "v_accvgpr_write_b32 a90, 0\n"
        "v_accvgpr_write_b32 a91, 0\n"
        "v_accvgpr_write_b32 a92, 0\n"
        "v_accvgpr_write_b32 a93, 0\n"
        "v_accvgpr_write_b32 a94, 0\n"
        "v_accvgpr_write_b32 a95, 0\n"
        "v_accvgpr_write_b32 a96, 0\n"
        "v_accvgpr_write_b32 a97, 0\n"
        "v_accvgpr_write_b32 a98, 0\n"
        "v_accvgpr_write_b32 a99, 0\n"
        "v_accvgpr_write_b32 a100, 0\n"
        "v_accvgpr_write_b32 a101, 0\n"
        "v_accvgpr_write_b32 a102, 0\n"
        "v_accvgpr_write_b32 a103, 0\n"
        "v_accvgpr_write_b32 a104, 0\n"
        "v_accvgpr_write_b32 a105, 0\n"
        "v_accvgpr_write_b32 a106, 0\n"
        "v_accvgpr_write_b32 a107, 0\n"
        "v_accvgpr_write_b32 a108, 0\n"
        "v_accvgpr_write_b32 a109, 0\n"
        "v_accvgpr_write_b32 a110, 0\n"
        "v_accvgpr_write_b32 a111, 0\n"
        "v_accvgpr_write_b32 a112, 0\n"
        "v_accvgpr_write_b32 a113, 0\n"
        "v_accvgpr_write_b32 a114, 0\n"
        "v_accvgpr_write_b32 a115, 0\n"
        "v_accvgpr_write_b32 a116, 0\n"
        "v_accvgpr_write_b32 a117, 0\n"
        "v_accvgpr_write_b32 a118, 0\n"
        "v_accvgpr_write_b32 a119, 0\n"
        "v_accvgpr_write_b32 a120, 0\n"
        "v_accvgpr_write_b32 a121, 0\n"
        "v_accvgpr_write_b32 a122, 0\n"
        "v_accvgpr_write_b32 a123, 0\n"
        "v_accvgpr_write_b32 a124, 0\n"
        "v_accvgpr_write_b32 a125, 0\n"
        "v_accvgpr_write_b32 a126, 0\n"
        "v_accvgpr_write_b32 a127, 0\n"
        "s_waitcnt vmcnt(32)\n"
        "v_readfirstlane_b32 s60, v4\n"
        "v_cvt_pk_f16_f32 v128, v16, v17\n"
        "v_cvt_pk_f16_f32 v129, v18, v19\n"
        "v_cvt_pk_f16_f32 v130, v20, v21\n"
        "v_cvt_pk_f16_f32 v131, v22, v23\n"
        "v_cvt_pk_f16_f32 v132, v24, v25\n"
        "v_cvt_pk_f16_f32 v133, v26, v27\n"
        "v_cvt_pk_f16_f32 v134, v28, v29\n"
        "v_cvt_pk_f16_f32 v135, v30, v31\n"
        "v_cvt_pk_f16_f32 v136, v32, v33\n"
        "v_cvt_pk_f16_f32 v137, v34, v35\n"
        "v_cvt_pk_f16_f32 v138, v36, v37\n"
        "v_cvt_pk_f16_f32 v139, v38, v39\n"
        "v_cvt_pk_f16_f32 v140, v40, v41\n"
        "v_cvt_pk_f16_f32 v141, v42, v43\n"
        "v_cvt_pk_f16_f32 v142, v44, v45\n"
        "v_cvt_pk_f16_f32 v143, v46, v47\n"
        "s_add_u32 s60, s60, 1\n"
        "s_and_b32 s70, s60, 1\n"
        "s_lshl_b32 s78, s70, 22\n"
        "s_add_u32 s31, s31, s78\n"
        "s_mul_hi_u32 s70, s82, 1073741825\n"
        "s_mul_i32 s70, s70, 4\n"
        "s_sub_u32 s24, s82, s70\n"
        "s_mul_hi_u32 s70, s82, 477218589\n"
        "s_mul_i32 s70, s70, 9\n"
        "s_sub_u32 s25, s82, s70\n"
        "s_add_u32 s25, s25, 4\n"
        "s_cmp_lg_u32 s83, 0\n"
        "s_cbranch_scc1 L_pushed_%=\n"
        "s_add_u32 s26, s24, 1\n"
        "s_add_u32 s27, s25, 1\n"
        "L_pushed_%=:\n"
        "v_accvgpr_write_b32 a128, 0\n"
        "v_accvgpr_write_b32 a129, 0\n"
        "v_accvgpr_write_b32 a130, 0\n"
        "v_accvgpr_write_b32 a131, 0\n"
        "v_accvgpr_write_b32 a132, 0\n"
        "v_accvgpr_write_b32 a133, 0\n"
        "v_accvgpr_write_b32 a134, 0\n"
        "v_accvgpr_write_b32 a135, 0\n"
        "v_accvgpr_write_b32 a136, 0\n"
        "v_accvgpr_write_b32 a137, 0\n"
        "v_accvgpr_write_b32 a138, 0\n"
        "v_accvgpr_write_b32 a139, 0\n"
        "v_accvgpr_write_b32 a140, 0\n"
        "v_accvgpr_write_b32 a141, 0\n"
        "v_accvgpr_write_b32 a142, 0\n"
        "v_accvgpr_write_b32 a143, 0\n"
        "v_accvgpr_write_b32 a144, 0\n"
        "v_accvgpr_write_b32 a145, 0\n"
        "v_accvgpr_write_b32 a146, 0\n"
        "v_accvgpr_write_b32 a147, 0\n"
        "v_accvgpr_write_b32 a148, 0\n"
        "v_accvgpr_write_b32 a149, 0\n"
        "v_accvgpr_write_b32 a150, 0\n"
        "v_accvgpr_write_b32 a151, 0\n"
        "v_accvgpr_write_b32 a152, 0\n"
        "v_accvgpr_write_b32 a153, 0\n"
        "v_accvgpr_write_b32 a154, 0\n"
        "v_accvgpr_write_b32 a155, 0\n"
        "v_accvgpr_write_b32 a156, 0\n"
        "v_accvgpr_write_b32 a157, 0\n"
        "v_accvgpr_write_b32 a158, 0\n"
        "v_accvgpr_write_b32 a159, 0\n"
        "v_accvgpr_write_b32 a160, 0\n"
        "v_accvgpr_write_b32 a161, 0\n"
        "v_accvgpr_write_b32 a162, 0\n"
        "v_accvgpr_write_b32 a163, 0\n"
        "v_accvgpr_write_b32 a164, 0\n"
        "v_accvgpr_write_b32 a165, 0\n"
        "v_accvgpr_write_b32 a166, 0\n"
        "v_accvgpr_write_b32 a167, 0\n"
        "v_accvgpr_write_b32 a168, 0\n"
        "v_accvgpr_write_b32 a169, 0\n"
        "v_accvgpr_write_b32 a170, 0\n"
        "v_accvgpr_write_b32 a171, 0\n"
        "v_accvgpr_write_b32 a172, 0\n"
        "v_accvgpr_write_b32 a173, 0\n"
        "v_accvgpr_write_b32 a174, 0\n"
        "v_accvgpr_write_b32 a175, 0\n"
        "v_accvgpr_write_b32 a176, 0\n"
        "v_accvgpr_write_b32 a177, 0\n"
        "v_accvgpr_write_b32 a178, 0\n"
        "v_accvgpr_write_b32 a179, 0\n"
        "v_accvgpr_write_b32 a180, 0\n"
        "v_accvgpr_write_b32 a181, 0\n"
        "v_accvgpr_write_b32 a182, 0\n"
        "v_accvgpr_write_b32 a183, 0\n"
        "v_accvgpr_write_b32 a184, 0\n"
        "v_accvgpr_write_b32 a185, 0\n"
        "v_accvgpr_write_b32 a186, 0\n"
        "v_accvgpr_write_b32 a187, 0\n"
        "v_accvgpr_write_b32 a188, 0\n"
        "v_accvgpr_write_b32 a189, 0\n"
        "v_accvgpr_write_b32 a190, 0\n"
        "v_accvgpr_write_b32 a191, 0\n"
        "v_accvgpr_write_b32 a192, 0\n"
        "v_accvgpr_write_b32 a193, 0\n"
        "v_accvgpr_write_b32 a194, 0\n"
        "v_accvgpr_write_b32 a195, 0\n"
        "v_accvgpr_write_b32 a196, 0\n"
        "v_accvgpr_write_b32 a197, 0\n"
        "v_accvgpr_write_b32 a198, 0\n"
        "v_accvgpr_write_b32 a199, 0\n"
        "v_accvgpr_write_b32 a200, 0\n"
        "v_accvgpr_write_b32 a201, 0\n"
        "v_accvgpr_write_b32 a202, 0\n"
        "v_accvgpr_write_b32 a203, 0\n"
        "v_accvgpr_write_b32 a204, 0\n"
        "v_accvgpr_write_b32 a205, 0\n"
        "v_accvgpr_write_b32 a206, 0\n"
        "v_accvgpr_write_b32 a207, 0\n"
        "v_accvgpr_write_b32 a208, 0\n"
        "v_accvgpr_write_b32 a209, 0\n"
        "v_accvgpr_write_b32 a210, 0\n"
        "v_accvgpr_write_b32 a211, 0\n"
        "v_accvgpr_write_b32 a212, 0\n"
        "v_accvgpr_write_b32 a213, 0\n"
        "v_accvgpr_write_b32 a214, 0\n"
        "v_accvgpr_write_b32 a215, 0\n"
        "v_accvgpr_write_b32 a216, 0\n"
        "v_accvgpr_write_b32 a217, 0\n"
        "v_accvgpr_write_b32 a218, 0\n"
        "v_accvgpr_write_b32 a219, 0\n"
        "v_accvgpr_write_b32 a220, 0\n"
        "v_accvgpr_write_b32 a221, 0\n"
        "v_accvgpr_write_b32 a222, 0\n"
        "v_accvgpr_write_b32 a223, 0\n"
        "v_accvgpr_write_b32 a224, 0\n"
        "v_accvgpr_write_b32 a225, 0\n"
        "v_accvgpr_write_b32 a226, 0\n"
        "v_accvgpr_write_b32 a227, 0\n"
        "v_accvgpr_write_b32 a228, 0\n"
        "v_accvgpr_write_b32 a229, 0\n"
        "v_accvgpr_write_b32 a230, 0\n"
        "v_accvgpr_write_b32 a231, 0\n"
        "v_accvgpr_write_b32 a232, 0\n"
        "v_accvgpr_write_b32 a233, 0\n"
        "v_accvgpr_write_b32 a234, 0\n"
        "v_accvgpr_write_b32 a235, 0\n"
        "v_accvgpr_write_b32 a236, 0\n"
        "v_accvgpr_write_b32 a237, 0\n"
        "v_accvgpr_write_b32 a238, 0\n"
        "v_accvgpr_write_b32 a239, 0\n"
        "v_accvgpr_write_b32 a240, 0\n"
        "v_accvgpr_write_b32 a241, 0\n"
        "v_accvgpr_write_b32 a242, 0\n"
        "v_accvgpr_write_b32 a243, 0\n"
        "v_accvgpr_write_b32 a244, 0\n"
        "v_accvgpr_write_b32 a245, 0\n"
        "v_accvgpr_write_b32 a246, 0\n"
        "v_accvgpr_write_b32 a247, 0\n"
        "v_accvgpr_write_b32 a248, 0\n"
        "v_accvgpr_write_b32 a249, 0\n"
        "v_accvgpr_write_b32 a250, 0\n"
        "v_accvgpr_write_b32 a251, 0\n"
        "v_accvgpr_write_b32 a252, 0\n"
        "v_accvgpr_write_b32 a253, 0\n"
        "v_accvgpr_write_b32 a254, 0\n"
        "v_accvgpr_write_b32 a255, 0\n"
        "s_mov_b32 s69, 0\n"
        "s_mov_b32 s68, 0\n"
        "s_lshl_b32 s70, s83, 13\n"
        "v_add_u32 v2, s70, v1\n"
        "s_add_u32 s64, s87, s70\n"
        "s_add_u32 s92, s64, 131072\n"
        "s_mov_b32 s93, s64\n"
        "s_mov_b32 s65, s87\n"
        "s_add_u32 s94, s87, 131072\n"
        "v_and_b32 v15, 3, v15\n"
        "s_lshl_b32 s70, s82, 4\n"
        "s_lshl_b32 s71, s83, 2\n"
        "s_add_u32 s70, s70, s71\n"
        "v_add_u32 v15, s70, v15\n"
        "v_lshlrev_b32 v15, 7, v15\n"
        "s_mov_b32 s88, 4\n"
        "v_and_b32 v153, 0x1f0, v1\n"
        "v_lshrrev_b32 v153, 2, v153\n"
        "s_mov_b32 s35, 0x800000\n"
        "s_lshr_b32 s66, s68, 2\n"
        "s_and_b32 s66, s66, 28\n"
        "s_lshr_b32 s66, s79, s66\n"
        "s_and_b32 s66, s66, 7\n"
        "s_lshl_b32 s66, s66, 4\n"
        "s_and_b32 s30, s68, 15\n"
        "s_or_b32 s66, s66, s30\n"
        "s_and_b32 s66, s66, 120\n"
        "s_lshl_b32 s66, s66, 4\n"
        "v_add_u32 v3, s66, v153\n"
        "s_memrealtime s[74:75]\n"
        "L_spinu2_%=:\n"
        "buffer_load_dword v4, v3, s[56:59], 0 offen sc0 sc1\n"
        "v_mov_b32 v14, 0\n"
        "s_mov_b32 s70, 16384\n"
        "buffer_load_dword v13, v14, s[56:59], s70 offen sc1\n"
        "s_waitcnt vmcnt(0) lgkmcnt(0)\n"
        "v_cmp_le_u32 vcc, s60, v4\n"
        "s_nop 1\n"
        "s_cmp_eq_u64 vcc, -1\n"
        "s_cbranch_scc1 L_oku2_%=\n"
        "v_readfirstlane_b32 s70, v13\n"
        "s_nop 3\n"
        "s_cmp_eq_u32 s70, s60\n"
        "s_cbranch_scc1 L_badu2_%=\n"
        "s_memrealtime s[76:77]\n"
        "s_waitcnt lgkmcnt(0)\n"
        "s_sub_u32 s70, s76, s74\n"
        "s_cmp_gt_u32 s70, 50000000\n"
        "s_cbranch_scc1 L_badu2_%=\n"
        "s_sleep 2\n"
        "s_branch L_spinu2_%=\n"
        "L_badu2_%=:\n"
        "v_mov_b32 v13, s60\n"
        "v_mov_b32 v14, 0\n"
        "s_mov_b32 s70, 16384\n"
        "buffer_store_dword v13, v14, s[56:59], s70 offen sc1\n"
        "s_waitcnt vmcnt(0)\n"
        "s_mov_b32 s69, 1\n"
        "s_mov_b64 vcc, -1\n"
        "L_oku2_%=:\n"
        "s_lshr_b32 s63, s68, 2\n"
        "s_and_b32 s63, s63, 28\n"
        "s_lshr_b32 s63, s79, s63\n"
        "s_and_b32 s63, s63, 7\n"
        "s_lshl_b32 s63, s63, 4\n"
        "s_and_b32 s22, s68, 15\n"
        "s_or_b32 s63, s63, s22\n"
        "s_lshl_b32 s63, s63, 15\n"
        "s_add_u32 s63, s63, s78\n"
        "s_add_u32 s68, s68, 1\n"
        "s_mov_b32 m0, s64\n"
        "s_nop 0\n"
        "buffer_load_dwordx4 v2, s[40:43], s63 offen offset:0 sc1 lds\n"
        "buffer_load_dwordx4 v2, s[40:43], s63 offen offset:1024 sc1 lds\n"
        "buffer_load_dwordx4 v2, s[40:43], s63 offen offset:2048 sc1 lds\n"
        "buffer_load_dwordx4 v2, s[40:43], s63 offen offset:3072 sc1 lds\n"
        "s_add_u32 m0, s64, 4096\n"
        "s_add_u32 s63, s63, 4096\n"
        "s_nop 0\n"
        "buffer_load_dwordx4 v2, s[40:43], s63 offen offset:0 sc1 lds\n"
        "buffer_load_dwordx4 v2, s[40:43], s63 offen offset:1024 sc1 lds\n"
        "buffer_load_dwordx4 v2, s[40:43], s63 offen offset:2048 sc1 lds\n"
        "buffer_load_dwordx4 v2, s[40:43], s63 offen offset:3072 sc1 lds\n"
        "s_add_u32 s64, s64, 32768\n"
        "s_cmp_ge_u32 s64, s92\n"
        "s_cselect_b32 s64, s93, s64\n"
        "s_lshr_b32 s63, s68, 2\n"
        "s_and_b32 s63, s63, 28\n"
        "s_lshr_b32 s63, s79, s63\n"
        "s_and_b32 s63, s63, 7\n"
        "s_lshl_b32 s63, s63, 4\n"
        "s_and_b32 s22, s68, 15\n"
        "s_or_b32 s63, s63, s22\n"
        "s_lshl_b32 s63, s63, 15\n"
        "s_add_u32 s63, s63, s78\n"
        "s_add_u32 s68, s68, 1\n"
        "s_mov_b32 m0, s64\n"
        "s_nop 0\n"
        "buffer_load_dwordx4 v2, s[40:43], s63 offen offset:0 sc1 lds\n"
        "buffer_load_dwordx4 v2, s[40:43], s63 offen offset:1024 sc1 lds\n"
        "buffer_load_dwordx4 v2, s[40:43], s63 offen offset:2048 sc1 lds\n"
        "buffer_load_dwordx4 v2, s[40:43], s63 offen offset:3072 sc1 lds\n"
        "s_add_u32 m0, s64, 4096\n"
        "s_add_u32 s63, s63, 4096\n"
        "s_nop 0\n"
        "buffer_load_dwordx4 v2, s[40:43], s63 offen offset:0 sc1 lds\n"
        "buffer_load_dwordx4 v2, s[40:43], s63 offen offset:1024 sc1 lds\n"
        "buffer_load_dwordx4 v2, s[40:43], s63 offen offset:2048 sc1 lds\n"
        "buffer_load_dwordx4 v2, s[40:43], s63 offen offset:3072 sc1 lds\n"
        "s_add_u32 s64, s64, 32768\n"
        "s_cmp_ge_u32 s64, s92\n"
        "s_cselect_b32 s64, s93, s64\n"
        "s_add_u32 s30, s68, 1\n"
        "s_lshr_b32 s66, s30, 2\n"
        "s_and_b32 s66, s66, 28\n"
        "s_lshr_b32 s66, s79, s66\n"
        "s_and_b32 s66, s66, 7\n"
        "s_lshl_b32 s66, s66, 4\n"
        "s_and_b32 s30, s30, 15\n"
        "s_or_b32 s66, s66, s30\n"
        "s_and_b32 s66, s66, 120\n"
        "s_lshl_b32 s66, s66, 4\n"
        "v_add_u32 v3, s66, v153\n"
        "buffer_load_dword v4, v3, s[56:59], 0 offen sc0 sc1\n"
        "s_lshr_b32 s63, s68, 2\n"
        "s_and_b32 s63, s63, 28\n"
        "s_lshr_b32 s63, s79, s63\n"
        "s_and_b32 s63, s63, 7\n"
        "s_lshl_b32 s63, s63, 4\n"
        "s_and_b32 s22, s68, 15\n"
        "s_or_b32 s63, s63, s22\n"
        "s_lshl_b32 s63, s63, 15\n"
        "s_add_u32 s63, s63, s78\n"
        "s_add_u32 s68, s68, 1\n"
        "s_mov_b32 m0, s64\n"
        "v_cvt_pk_f16_f32 v48, v64, v66\n"
        "v_cvt_pk_f16_f32 v52, v65, v67\n"
        "v_cvt_pk_f16_f32 v49, v68, v70\n"
        "v_cvt_pk_f16_f32 v53, v69, v71\n"
        "v_cvt_pk_f16_f32 v50, v72, v74\n"
        "v_cvt_pk_f16_f32 v54, v73, v75\n"
        "v_cvt_pk_f16_f32 v51, v76, v78\n"
        "v_cvt_pk_f16_f32 v55, v77, v79\n"
        "s_waitcnt vmcnt(0)\n"
        "s_barrier\n"
        "v_add_u32 v0, s65, v1\n"
        "s_add_u32 s65, s65, 32768\n"
        "s_cmp_ge_u32 s65, s94\n"
        "s_cselect_b32 s65, s87, s65\n"
        "ds_read_b128 v[16:19], v0 offset:0\n"
        "ds_read_b128 v[20:23], v0 offset:1024\n"
        "ds_read_b128 v[24:27], v0 offset:2048\n"
        "ds_read_b128 v[28:31], v0 offset:3072\n"
        "ds_read_b128 v[32:35], v0 offset:4096\n"
        "ds_read_b128 v[36:39], v0 offset:5120\n"
        "ds_read_b128 v[40:43], v0 offset:6144\n"
        "s_mov_b32 s61, 0\n"
        "s_mov_b32 s95, 0\n"
        ".p2align 6\n"
        "L_loop1_%=:\n"
        "s_and_b32 s70, s95, 15\n"
        "s_cmp_lg_u32 s70, 12\n"
        "s_cbranch_scc1 L_d3u8_%=\n"
        "s_cmp_ge_u32 s95, 112\n"
        "s_cbranch_scc1 L_d3u8_%=\n"
        "v_mov_b32 v13, s87\n"
        "v_add_u32 v13, 131072, v13\n"
        "ds_read_b32 v14, v13\n"
        "s_waitcnt lgkmcnt(0)\n"
        "v_readfirstlane_b32 s71, v14\n"
        "s_nop 3\n"
        "s_lshr_b32 s72, s95, 4\n"
        "s_add_u32 s72, s72, 1\n"
        "s_lshl_b32 s72, s72, 2\n"
        "s_lshl_b32 s73, 15, s72\n"
        "s_andn2_b32 s79, s79, s73\n"
        "s_lshl_b32 s73, s71, s72\n"
        "s_or_b32 s79, s79, s73\n"
        "s_lshl_b32 s73, 1, s71\n"
        "s_or_b32 s29, s29, s73\n"
        "L_d3u8_%=:\n"
        "s_cmp_lg_u32 s95, s24\n"
        "s_cbranch_scc1 L_npu4_%=\n"
        "s_add_u32 s70, s31, 0\n"
        "buffer_store_dwordx4 v[128:131], v1, s[52:55], s70 offen sc0 sc1\n"
        "s_add_u32 s70, s31, 1024\n"
        "buffer_store_dwordx4 v[132:135], v1, s[52:55], s70 offen sc0 sc1\n"
        "L_npu4_%=:\n"
        "s_cmp_lg_u32 s95, s25\n"
        "s_cbranch_scc1 L_npu5_%=\n"
        "s_add_u32 s70, s31, 262144\n"
        "buffer_store_dwordx4 v[136:139], v1, s[52:55], s70 offen sc0 sc1\n"
        "s_add_u32 s70, s31, 263168\n"
        "buffer_store_dwordx4 v[140:143], v1, s[52:55], s70 offen sc0 sc1\n"
        "L_npu5_%=:\n"
        "s_waitcnt lgkmcnt(5)\n"
        "v_mfma_f32_16x16x32_f16 a[0:3], v[16:19], v[48:51], a[0:3]\n"
        "s_waitcnt vmcnt(32)\n"
        "v_cmp_le_u32 vcc, s60, v4\n"
        "s_nop 1\n"
        "s_cmp_eq_u64 vcc, -1\n"
        "s_cbranch_scc1 L_oku3_%=\n"
        "s_memrealtime s[74:75]\n"
        "L_spinu3_%=:\n"
        "buffer_load_dword v4, v3, s[56:59], 0 offen sc0 sc1\n"
        "v_mov_b32 v14, 0\n"
        "s_mov_b32 s70, 16384\n"
        "buffer_load_dword v13, v14, s[56:59], s70 offen sc1\n"
        "s_waitcnt vmcnt(0) lgkmcnt(0)\n"
        "v_cmp_le_u32 vcc, s60, v4\n"
        "s_nop 1\n"
        "s_cmp_eq_u64 vcc, -1\n"
        "s_cbranch_scc1 L_oku3_%=\n"
        "v_readfirstlane_b32 s70, v13\n"
        "s_nop 3\n"
        "s_cmp_eq_u32 s70, s60\n"
        "s_cbranch_scc1 L_badu3_%=\n"
        "s_memrealtime s[76:77]\n"
        "s_waitcnt lgkmcnt(0)\n"
        "s_sub_u32 s70, s76, s74\n"
        "s_cmp_gt_u32 s70, 50000000\n"
        "s_cbranch_scc1 L_badu3_%=\n"
        "s_sleep 2\n"
        "s_branch L_spinu3_%=\n"
        "L_badu3_%=:\n"
        "v_mov_b32 v13, s60\n"
        "v_mov_b32 v14, 0\n"
        "s_mov_b32 s70, 16384\n"
        "buffer_store_dword v13, v14, s[56:59], s70 offen sc1\n"
        "s_waitcnt vmcnt(0)\n"
        "s_mov_b32 s69, 1\n"
        "s_mov_b64 vcc, -1\n"
        "L_oku3_%=:\n"
        "ds_read_b128 v[44:47], v0 offset:7168\n"
        "s_add_u32 s30, s68, 2\n"
        "s_lshr_b32 s66, s30, 2\n"
        "v_mfma_f32_16x16x32_f16 a[4:7], v[16:19], v[52:55], a[4:7]\n"
        "s_waitcnt vmcnt(48)\n"
        "v_cvt_pk_f16_f32 v56, v80, v82\n"
        "s_and_b32 s66, s66, 28\n"
        "s_lshr_b32 s66, s79, s66\n"
        "v_mfma_f32_16x16x32_f16 a[8:11], v[20:23], v[48:51], a[8:11]\n"
        "ds_read_b128 v[16:19], v0 offset:8192\n"
        "buffer_load_dwordx4 v2, s[40:43], s63 offen offset:0 sc1 lds\n"
        "s_and_b32 s66, s66, 7\n"
        "s_lshl_b32 s66, s66, 4\n"
        "v_mfma_f32_16x16x32_f16 a[12:15], v[20:23], v[52:55], a[12:15]\n"
        "v_cvt_pk_f16_f32 v60, v81, v83\n"
        "s_and_b32 s30, s30, 15\n"
        "s_or_b32 s66, s66, s30\n"
        "s_waitcnt lgkmcnt(5)\n"
        "v_mfma_f32_16x16x32_f16 a[16:19], v[24:27], v[48:51], a[16:19]\n"
        "ds_read_b128 v[20:23], v0 offset:9216\n"
        "s_and_b32 s66, s66, 120\n"
        "s_lshl_b32 s66, s66, 4\n"
        "v_mfma_f32_16x16x32_f16 a[20:23], v[24:27], v[52:55], a[20:23]\n"
        "s_waitcnt vmcnt(47)\n"
        "v_cvt_pk_f16_f32 v57, v84, v86\n"
        "v_add_u32 v3, s66, v153\n"
        "buffer_load_dword v4, v3, s[56:59], 0 offen sc0 sc1\n"
        "v_mfma_f32_16x16x32_f16 a[24:27], v[28:31], v[48:51], a[24:27]\n"
        "ds_read_b128 v[24:27], v0 offset:10240\n"
        "buffer_load_dwordx4 v2, s[40:43], s63 offen offset:1024 sc1 lds\n"
        "v_mfma_f32_16x16x32_f16 a[28:31], v[28:31], v[52:55], a[28:31]\n"
        "v_cvt_pk_f16_f32 v61, v85, v87\n"
        "s_waitcnt lgkmcnt(5)\n"
        "v_mfma_f32_16x16x32_f16 a[32:35], v[32:35], v[48:51], a[32:35]\n"
        "ds_read_b128 v[28:31], v0 offset:11264\n"
        "v_mfma_f32_16x16x32_f16 a[36:39], v[32:35], v[52:55], a[36:39]\n"
        "s_waitcnt vmcnt(47)\n"
        "v_cvt_pk_f16_f32 v58, v88, v90\n"
        "v_mfma_f32_16x16x32_f16 a[40:43], v[36:39], v[48:51], a[40:43]\n"
        "ds_read_b128 v[32:35], v0 offset:12288\n"
        "buffer_load_dwordx4 v2, s[40:43], s63 offen offset:2048 sc1 lds\n"
        "v_mfma_f32_16x16x32_f16 a[44:47], v[36:39], v[52:55], a[44:47]\n"
        "v_cvt_pk_f16_f32 v62, v89, v91\n"
        "s_waitcnt lgkmcnt(5)\n"
        "v_mfma_f32_16x16x32_f16 a[48:51], v[40:43], v[48:51], a[48:51]\n"
        "ds_read_b128 v[36:39], v0 offset:13312\n"
        "v_mfma_f32_16x16x32_f16 a[52:55], v[40:43], v[52:55], a[52:55]\n"
        "s_waitcnt vmcnt(46)\n"
        "v_cvt_pk_f16_f32 v59, v92, v94\n"
        "v_mfma_f32_16x16x32_f16 a[56:59], v[44:47], v[48:51], a[56:59]\n"
        "ds_read_b128 v[40:43], v0 offset:14336\n"
        "buffer_load_dwordx4 v2, s[40:43], s63 offen offset:3072 sc1 lds\n"
        "v_mfma_f32_16x16x32_f16 a[60:63], v[44:47], v[52:55], a[60:63]\n"
        "v_cvt_pk_f16_f32 v63, v93, v95\n"
        "s_waitcnt lgkmcnt(5)\n"
        "v_mfma_f32_16x16x32_f16 a[64:67], v[16:19], v[48:51], a[64:67]\n"
        "ds_read_b128 v[44:47], v0 offset:15360\n"
        "s_and_b32 s22, s88, 7\n"
        "v_mfma_f32_16x16x32_f16 a[68:71], v[16:19], v[52:55], a[68:71]\n"
        "s_cmp_eq_u32 s22, 0\n"
        "s_cselect_b32 s98, 0, s35\n"
        "v_mfma_f32_16x16x32_f16 a[72:75], v[20:23], v[48:51], a[72:75]\n"
        "ds_read_b128 v[16:19], v0 offset:16384\n"
        "s_add_u32 m0, s64, 4096\n"
        "s_add_u32 s63, s63, 4096\n"
        "s_nop 0\n"
        "buffer_load_dwordx4 v2, s[40:43], s63 offen offset:0 sc1 lds\n"
        "s_lshr_b32 s89, s88, 2\n"
        "v_mfma_f32_16x16x32_f16 a[76:79], v[20:23], v[52:55], a[76:79]\n"
        "s_and_b32 s89, s89, 28\n"
        "s_waitcnt lgkmcnt(5)\n"
        "v_mfma_f32_16x16x32_f16 a[80:83], v[24:27], v[48:51], a[80:83]\n"
        "ds_read_b128 v[20:23], v0 offset:17408\n"
        "s_lshr_b32 s89, s79, s89\n"
        "v_mfma_f32_16x16x32_f16 a[84:87], v[24:27], v[52:55], a[84:87]\n"
        "s_and_b32 s89, s89, 7\n"
        "v_mfma_f32_16x16x32_f16 a[88:91], v[28:31], v[48:51], a[88:91]\n"
        "ds_read_b128 v[24:27], v0 offset:18432\n"
        "buffer_load_dwordx4 v2, s[40:43], s63 offen offset:1024 sc1 lds\n"
        "s_lshl_b32 s89, s89, 4\n"
        "v_mfma_f32_16x16x32_f16 a[92:95], v[28:31], v[52:55], a[92:95]\n"
        "s_lshr_b32 s62, s67, 2\n"
        "s_and_b32 s22, s88, 15\n"
        "s_waitcnt lgkmcnt(5)\n"
        "v_mfma_f32_16x16x32_f16 a[96:99], v[32:35], v[48:51], a[96:99]\n"
        "ds_read_b128 v[28:31], v0 offset:19456\n"
        "s_and_b32 s62, s62, 28\n"
        "s_or_b32 s89, s89, s22\n"
        "v_mfma_f32_16x16x32_f16 a[100:103], v[32:35], v[52:55], a[100:103]\n"
        "s_lshr_b32 s62, s79, s62\n"
        "s_lshl_b32 s89, s89, 15\n"
        "v_mfma_f32_16x16x32_f16 a[104:107], v[36:39], v[48:51], a[104:107]\n"
        "ds_read_b128 v[32:35], v0 offset:20480\n"
        "buffer_load_dwordx4 v2, s[40:43], s63 offen offset:2048 sc1 lds\n"
        "s_and_b32 s62, s62, 7\n"
        "s_add_u32 s89, s89, s78\n"
        "v_mfma_f32_16x16x32_f16 a[108:111], v[36:39], v[52:55], a[108:111]\n"
        "s_lshl_b32 s62, s62, 4\n"
        "s_waitcnt lgkmcnt(5)\n"
        "v_mfma_f32_16x16x32_f16 a[112:115], v[40:43], v[48:51], a[112:115]\n"
        "ds_read_b128 v[36:39], v0 offset:21504\n"
        "s_and_b32 s23, s67, 15\n"
        "buffer_load_dword v152, v15, s[96:99], s89 offen\n"
        "s_add_u32 s88, s88, 2\n"
        "v_mfma_f32_16x16x32_f16 a[116:119], v[40:43], v[52:55], a[116:119]\n"
        "s_or_b32 s62, s62, s23\n"
        "v_mfma_f32_16x16x32_f16 a[120:123], v[44:47], v[48:51], a[120:123]\n"
        "ds_read_b128 v[40:43], v0 offset:22528\n"
        "buffer_load_dwordx4 v2, s[40:43], s63 offen offset:3072 sc1 lds\n"
        "s_add_u32 s64, s64, 32768\n"
        "s_cmp_ge_u32 s64, s92\n"
        "s_cselect_b32 s64, s93, s64\n"
        "s_lshl_b32 s62, s62, 22\n"
        "v_mfma_f32_16x16x32_f16 a[124:127], v[44:47], v[52:55], a[124:127]\n"
        "s_add_u32 s67, s67, 1\n"
        "s_waitcnt lgkmcnt(5)\n"
        "v_mfma_f32_16x16x32_f16 a[128:131], v[16:19], v[48:51], a[128:131]\n"
        "ds_read_b128 v[44:47], v0 offset:23552\n"
        "buffer_load_dwordx2 v[64:65], v5, s[36:39], s62 offen nt\n"
        "v_mfma_f32_16x16x32_f16 a[132:135], v[16:19], v[52:55], a[132:135]\n"
        "v_mfma_f32_16x16x32_f16 a[136:139], v[20:23], v[48:51], a[136:139]\n"
        "ds_read_b128 v[16:19], v0 offset:24576\n"
        "buffer_load_dwordx2 v[66:67], v6, s[36:39], s62 offen nt\n"
        "v_mfma_f32_16x16x32_f16 a[140:143], v[20:23], v[52:55], a[140:143]\n"
        "s_waitcnt lgkmcnt(5)\n"
        "v_mfma_f32_16x16x32_f16 a[144:147], v[24:27], v[48:51], a[144:147]\n"
        "ds_read_b128 v[20:23], v0 offset:25600\n"
        "buffer_load_dwordx2 v[68:69], v7, s[36:39], s62 offen nt\n"
        "v_mfma_f32_16x16x32_f16 a[148:151], v[24:27], v[52:55], a[148:151]\n"
        "v_mfma_f32_16x16x32_f16 a[152:155], v[28:31], v[48:51], a[152:155]\n"
        "ds_read_b128 v[24:27], v0 offset:26624\n"
        "buffer_load_dwordx2 v[70:71], v8, s[36:39], s62 offen nt\n"
        "v_mfma_f32_16x16x32_f16 a[156:159], v[28:31], v[52:55], a[156:159]\n"
        "s_waitcnt lgkmcnt(5)\n"
        "v_mfma_f32_16x16x32_f16 a[160:163], v[32:35], v[48:51], a[160:163]\n"
        "ds_read_b128 v[28:31], v0 offset:27648\n"
        "buffer_load_dwordx2 v[72:73], v9, s[36:39], s62 offen nt\n"
        "v_mfma_f32_16x16x32_f16 a[164:167], v[32:35], v[52:55], a[164:167]\n"
        "v_mfma_f32_16x16x32_f16 a[168:171], v[36:39], v[48:51], a[168:171]\n"
        "ds_read_b128 v[32:35], v0 offset:28672\n"
        "buffer_load_dwordx2 v[74:75], v10, s[36:39], s62 offen nt\n"
        "v_mfma_f32_16x16x32_f16 a[172:175], v[36:39], v[52:55], a[172:175]\n"
        "s_waitcnt lgkmcnt(5)\n"
        "v_mfma_f32_16x16x32_f16 a[176:179], v[40:43], v[48:51], a[176:179]\n"
        "ds_read_b128 v[36:39], v0 offset:29696\n"
        "buffer_load_dwordx2 v[76:77], v11, s[36:39], s62 offen nt\n"
        "v_mfma_f32_16x16x32_f16 a[180:183], v[40:43], v[52:55], a[180:183]\n"
        "v_mfma_f32_16x16x32_f16 a[184:187], v[44:47], v[48:51], a[184:187]\n"
        "ds_read_b128 v[40:43], v0 offset:30720\n"
        "buffer_load_dwordx2 v[78:79], v12, s[36:39], s62 offen nt\n"
        "v_mfma_f32_16x16x32_f16 a[188:191], v[44:47], v[52:55], a[188:191]\n"
        "s_waitcnt lgkmcnt(5)\n"
        "v_mfma_f32_16x16x32_f16 a[192:195], v[16:19], v[48:51], a[192:195]\n"
        "ds_read_b128 v[44:47], v0 offset:31744\n"
        "s_lshr_b32 s62, s67, 2\n"
        "s_and_b32 s62, s62, 28\n"
        "s_lshr_b32 s62, s79, s62\n"
        "s_and_b32 s62, s62, 7\n"
        "s_lshl_b32 s62, s62, 4\n"
        "s_and_b32 s23, s67, 15\n"
        "s_or_b32 s62, s62, s23\n"
        "s_lshl_b32 s62, s62, 22\n"
        "s_add_u32 s67, s67, 1\n"
        "buffer_load_dwordx2 v[80:81], v5, s[36:39], s62 offen nt\n"
        "v_mfma_f32_16x16x32_f16 a[196:199], v[16:19], v[52:55], a[196:199]\n"
        "v_mfma_f32_16x16x32_f16 a[200:203], v[20:23], v[48:51], a[200:203]\n"
        "s_waitcnt vmcnt(19)\n"
        "s_barrier\n"
        "v_add_u32 v0, s65, v1\n"
        "s_add_u32 s65, s65, 32768\n"
        "s_cmp_ge_u32 s65, s94\n"
        "s_cselect_b32 s65, s87, s65\n"
        "ds_read_b128 v[16:19], v0 offset:0\n"
        "buffer_load_dwordx2 v[82:83], v6, s[36:39], s62 offen nt\n"
        "v_mfma_f32_16x16x32_f16 a[204:207], v[20:23], v[52:55], a[204:207]\n"
        "s_cmp_lg_u32 s95, s26\n"
        "s_cbranch_scc1 L_nfu6_%=\n"
        "v_mov_b32 v14, s60\n"
        "v_mov_b32 v13, 0\n"
        "s_mov_b64 exec, 1\n"
        "buffer_store_dword v14, v13, s[52:55], s28 offen offset:0 sc0 sc1\n"
        "s_mov_b64 exec, -1\n"
        "L_nfu6_%=:\n"
        "s_cmp_lg_u32 s95, s27\n"
        "s_cbranch_scc1 L_nfu7_%=\n"
        "v_mov_b32 v14, s60\n"
        "v_mov_b32 v13, 0\n"
        "s_mov_b64 exec, 1\n"
        "buffer_store_dword v14, v13, s[52:55], s28 offen offset:128 sc0 sc1\n"
        "s_mov_b64 exec, -1\n"
        "L_nfu7_%=:\n"
        "s_waitcnt lgkmcnt(5)\n"
        "v_mfma_f32_16x16x32_f16 a[208:211], v[24:27], v[48:51], a[208:211]\n"
        "ds_read_b128 v[20:23], v0 offset:1024\n"
        "buffer_load_dwordx2 v[84:85], v7, s[36:39], s62 offen nt\n"
        "v_mfma_f32_16x16x32_f16 a[212:215], v[24:27], v[52:55], a[212:215]\n"
        "v_mfma_f32_16x16x32_f16 a[216:219], v[28:31], v[48:51], a[216:219]\n"
        "ds_read_b128 v[24:27], v0 offset:2048\n"
        "buffer_load_dwordx2 v[86:87], v8, s[36:39], s62 offen nt\n"
        "v_mfma_f32_16x16x32_f16 a[220:223], v[28:31], v[52:55], a[220:223]\n"
        "s_waitcnt lgkmcnt(5)\n"
        "v_mfma_f32_16x16x32_f16 a[224:227], v[32:35], v[48:51], a[224:227]\n"
        "ds_read_b128 v[28:31], v0 offset:3072\n"
        "buffer_load_dwordx2 v[88:89], v9, s[36:39], s62 offen nt\n"
        "v_mfma_f32_16x16x32_f16 a[228:231], v[32:35], v[52:55], a[228:231]\n"
        "v_mfma_f32_16x16x32_f16 a[232:235], v[36:39], v[48:51], a[232:235]\n"
        "ds_read_b128 v[32:35], v0 offset:4096\n"
        "buffer_load_dwordx2 v[90:91], v10, s[36:39], s62 offen nt\n"
        "v_mfma_f32_16x16x32_f16 a[236:239], v[36:39], v[52:55], a[236:239]\n"
        "s_waitcnt lgkmcnt(5)\n"
        "v_mfma_f32_16x16x32_f16 a[240:243], v[40:43], v[48:51], a[240:243]\n"
        "ds_read_b128 v[36:39], v0 offset:5120\n"
        "buffer_load_dwordx2 v[92:93], v11, s[36:39], s62 offen nt\n"
        "v_mfma_f32_16x16x32_f16 a[244:247], v[40:43], v[52:55], a[244:247]\n"
        "v_mfma_f32_16x16x32_f16 a[248:251], v[44:47], v[48:51], a[248:251]\n"
        "ds_read_b128 v[40:43], v0 offset:6144\n"
        "buffer_load_dwordx2 v[94:95], v12, s[36:39], s62 offen nt\n"
        "v_mfma_f32_16x16x32_f16 a[252:255], v[44:47], v[52:55], a[252:255]\n"
        "s_add_u32 s95, s95, 1\n"
        "s_cmp_lg_u32 s95, s24\n"
        "s_cbranch_scc1 L_npu9_%=\n"
        "s_add_u32 s70, s31, 0\n"
        "buffer_store_dwordx4 v[128:131], v1, s[52:55], s70 offen sc0 sc1\n"
        "s_add_u32 s70, s31, 1024\n"
        "buffer_store_dwordx4 v[132:135], v1, s[52:55], s70 offen sc0 sc1\n"
        "L_npu9_%=:\n"
        "s_cmp_lg_u32 s95, s25\n"
        "s_cbranch_scc1 L_npu10_%=\n"
        "s_add_u32 s70, s31, 262144\n"
        "buffer_store_dwordx4 v[136:139], v1, s[52:55], s70 offen sc0 sc1\n"
        "s_add_u32 s70, s31, 263168\n"
        "buffer_store_dwordx4 v[140:143], v1, s[52:55], s70 offen sc0 sc1\n"
        "L_npu10_%=:\n"
        "s_waitcnt lgkmcnt(5)\n"
        "v_mfma_f32_16x16x32_f16 a[0:3], v[16:19], v[56:59], a[0:3]\n"
        "ds_read_b128 v[44:47], v0 offset:7168\n"
        "v_mfma_f32_16x16x32_f16 a[4:7], v[16:19], v[60:63], a[4:7]\n"
        "s_waitcnt vmcnt(48)\n"
        "v_cvt_pk_f16_f32 v48, v96, v98\n"
        "v_mfma_f32_16x16x32_f16 a[8:11], v[20:23], v[56:59], a[8:11]\n"
        "ds_read_b128 v[16:19], v0 offset:8192\n"
        "v_mfma_f32_16x16x32_f16 a[12:15], v[20:23], v[60:63], a[12:15]\n"
        "v_cvt_pk_f16_f32 v52, v97, v99\n"
        "s_waitcnt lgkmcnt(5)\n"
        "v_mfma_f32_16x16x32_f16 a[16:19], v[24:27], v[56:59], a[16:19]\n"
        "ds_read_b128 v[20:23], v0 offset:9216\n"
        "v_mfma_f32_16x16x32_f16 a[20:23], v[24:27], v[60:63], a[20:23]\n"
        "s_waitcnt vmcnt(46)\n"
        "v_cvt_pk_f16_f32 v49, v100, v102\n"
        "v_mfma_f32_16x16x32_f16 a[24:27], v[28:31], v[56:59], a[24:27]\n"
        "ds_read_b128 v[24:27], v0 offset:10240\n"
        "v_mfma_f32_16x16x32_f16 a[28:31], v[28:31], v[60:63], a[28:31]\n"
        "v_cvt_pk_f16_f32 v53, v101, v103\n"
        "s_waitcnt lgkmcnt(5)\n"
        "v_mfma_f32_16x16x32_f16 a[32:35], v[32:35], v[56:59], a[32:35]\n"
        "ds_read_b128 v[28:31], v0 offset:11264\n"
        "v_mfma_f32_16x16x32_f16 a[36:39], v[32:35], v[60:63], a[36:39]\n"
        "s_waitcnt vmcnt(44)\n"
        "v_cvt_pk_f16_f32 v50, v104, v106\n"
        "v_mfma_f32_16x16x32_f16 a[40:43], v[36:39], v[56:59], a[40:43]\n"
        "ds_read_b128 v[32:35], v0 offset:12288\n"
        "v_mfma_f32_16x16x32_f16 a[44:47], v[36:39], v[60:63], a[44:47]\n"
        "v_cvt_pk_f16_f32 v54, v105, v107\n"
        "s_waitcnt lgkmcnt(5)\n"
        "v_mfma_f32_16x16x32_f16 a[48:51], v[40:43], v[56:59], a[48:51]\n"
        "ds_read_b128 v[36:39], v0 offset:13312\n"
        "s_lshr_b32 s63, s68, 2\n"
        "v_mfma_f32_16x16x32_f16 a[52:55], v[40:43], v[60:63], a[52:55]\n"
        "s_waitcnt vmcnt(42)\n"
        "v_cvt_pk_f16_f32 v51, v108, v110\n"
        "s_and_b32 s63, s63, 28\n"
        "v_mfma_f32_16x16x32_f16 a[56:59], v[44:47], v[56:59], a[56:59]\n"
        "ds_read_b128 v[40:43], v0 offset:14336\n"
        "s_lshr_b32 s63, s79, s63\n"
        "v_mfma_f32_16x16x32_f16 a[60:63], v[44:47], v[60:63], a[60:63]\n"
        "v_cvt_pk_f16_f32 v55, v109, v111\n"
        "s_and_b32 s63, s63, 7\n"
        "s_waitcnt lgkmcnt(5)\n"
        "v_mfma_f32_16x16x32_f16 a[64:67], v[16:19], v[56:59], a[64:67]\n"
        "ds_read_b128 v[44:47], v0 offset:15360\n"
        "s_lshl_b32 s63, s63, 4\n"
        "v_mfma_f32_16x16x32_f16 a[68:71], v[16:19], v[60:63], a[68:71]\n"
        "s_and_b32 s22, s68, 15\n"
        "v_mfma_f32_16x16x32_f16 a[72:75], v[20:23], v[56:59], a[72:75]\n"
        "ds_read_b128 v[16:19], v0 offset:16384\n"
        "s_or_b32 s63, s63, s22\n"
        "v_mfma_f32_16x16x32_f16 a[76:79], v[20:23], v[60:63], a[76:79]\n"
        "s_lshl_b32 s63, s63, 15\n"
        "s_waitcnt lgkmcnt(5)\n"
        "v_mfma_f32_16x16x32_f16 a[80:83], v[24:27], v[56:59], a[80:83]\n"
        "ds_read_b128 v[20:23], v0 offset:17408\n"
        "s_add_u32 s63, s63, s78\n"
        "v_mfma_f32_16x16x32_f16 a[84:87], v[24:27], v[60:63], a[84:87]\n"
        "s_add_u32 s68, s68, 1\n"
        "v_mfma_f32_16x16x32_f16 a[88:91], v[28:31], v[56:59], a[88:91]\n"
        "ds_read_b128 v[24:27], v0 offset:18432\n"
        "s_mov_b32 m0, s64\n"
        "v_mfma_f32_16x16x32_f16 a[92:95], v[28:31], v[60:63], a[92:95]\n"
        "s_waitcnt lgkmcnt(5)\n"
        "v_mfma_f32_16x16x32_f16 a[96:99], v[32:35], v[56:59], a[96:99]\n"
        "ds_read_b128 v[28:31], v0 offset:19456\n"
        "buffer_load_dwordx4 v2, s[40:43], s63 offen offset:0 sc1 lds\n"
        "v_mfma_f32_16x16x32_f16 a[100:103], v[32:35], v[60:63], a[100:103]\n"
        "v_mfma_f32_16x16x32_f16 a[104:107], v[36:39], v[56:59], a[104:107]\n"
        "ds_read_b128 v[32:35], v0 offset:20480\n"
        "v_mfma_f32_16x16x32_f16 a[108:111], v[36:39], v[60:63], a[108:111]\n"
        "s_waitcnt lgkmcnt(5)\n"
        "v_mfma_f32_16x16x32_f16 a[112:115], v[40:43], v[56:59], a[112:115]\n"
        "ds_read_b128 v[36:39], v0 offset:21504\n"
        "buffer_load_dwordx4 v2, s[40:43], s63 offen offset:1024 sc1 lds\n"
        "v_mfma_f32_16x16x32_f16 a[116:119], v[40:43], v[60:63], a[116:119]\n"
        "v_mfma_f32_16x16x32_f16 a[120:123], v[44:47], v[56:59], a[120:123]\n"
        "ds_read_b128 v[40:43], v0 offset:22528\n"
        "v_mfma_f32_16x16x32_f16 a[124:127], v[44:47], v[60:63], a[124:127]\n"
        "s_waitcnt lgkmcnt(5)\n"
        "v_mfma_f32_16x16x32_f16 a[128:131], v[16:19], v[56:59], a[128:131]\n"
        "ds_read_b128 v[44:47], v0 offset:23552\n"
        "buffer_load_dwordx4 v2, s[40:43], s63 offen offset:2048 sc1 lds\n"
        "v_mfma_f32_16x16x32_f16 a[132:135], v[16:19], v[60:63], a[132:135]\n"
        "v_mfma_f32_16x16x32_f16 a[136:139], v[20:23], v[56:59], a[136:139]\n"
        "ds_read_b128 v[16:19], v0 offset:24576\n"
        "v_mfma_f32_16x16x32_f16 a[140:143], v[20:23], v[60:63], a[140:143]\n"
        "s_waitcnt lgkmcnt(5)\n"
        "v_mfma_f32_16x16x32_f16 a[144:147], v[24:27], v[56:59], a[144:147]\n"
        "ds_read_b128 v[20:23], v0 offset:25600\n"
        "buffer_load_dwordx4 v2, s[40:43], s63 offen offset:3072 sc1 lds\n"
        "v_mfma_f32_16x16x32_f16 a[148:151], v[24:27], v[60:63], a[148:151]\n"
        "v_mfma_f32_16x16x32_f16 a[152:155], v[28:31], v[56:59], a[152:155]\n"
        "ds_read_b128 v[24:27], v0 offset:26624\n"
        "v_mfma_f32_16x16x32_f16 a[156:159], v[28:31], v[60:63], a[156:159]\n"
        "s_waitcnt lgkmcnt(5)\n"
        "v_mfma_f32_16x16x32_f16 a[160:163], v[32:35], v[56:59], a[160:163]\n"
        "ds_read_b128 v[28:31], v0 offset:27648\n"
        "s_add_u32 m0, s64, 4096\n"
        "s_add_u32 s63, s63, 4096\n"
        "s_nop 0\n"
        "buffer_load_dwordx4 v2, s[40:43], s63 offen offset:0 sc1 lds\n"
        "v_mfma_f32_16x16x32_f16 a[164:167], v[32:35], v[60:63], a[164:167]\n"
        "v_mfma_f32_16x16x32_f16 a[168:171], v[36:39], v[56:59], a[168:171]\n"
        "ds_read_b128 v[32:35], v0 offset:28672\n"
        "v_mfma_f32_16x16x32_f16 a[172:175], v[36:39], v[60:63], a[172:175]\n"
        "s_waitcnt lgkmcnt(5)\n"
        "v_mfma_f32_16x16x32_f16 a[176:179], v[40:43], v[56:59], a[176:179]\n"
        "ds_read_b128 v[36:39], v0 offset:29696\n"
        "buffer_load_dwordx4 v2, s[40:43], s63 offen offset:1024 sc1 lds\n"
        "v_mfma_f32_16x16x32_f16 a[180:183], v[40:43], v[60:63], a[180:183]\n"
        "v_mfma_f32_16x16x32_f16 a[184:187], v[44:47], v[56:59], a[184:187]\n"
        "ds_read_b128 v[40:43], v0 offset:30720\n"
        "v_mfma_f32_16x16x32_f16 a[188:191], v[44:47], v[60:63], a[188:191]\n"
        "s_waitcnt lgkmcnt(5)\n"
        "v_mfma_f32_16x16x32_f16 a[192:195], v[16:19], v[56:59], a[192:195]\n"
        "ds_read_b128 v[44:47], v0 offset:31744\n"
        "buffer_load_dwordx4 v2, s[40:43], s63 offen offset:2048 sc1 lds\n"
        "v_mfma_f32_16x16x32_f16 a[196:199], v[16:19], v[60:63], a[196:199]\n"
        "v_mfma_f32_16x16x32_f16 a[200:203], v[20:23], v[56:59], a[200:203]\n"
        "s_waitcnt vmcnt(23)\n"
        "s_barrier\n"
        "v_add_u32 v0, s65, v1\n"
        "s_add_u32 s65, s65, 32768\n"
        "s_cmp_ge_u32 s65, s94\n"
        "s_cselect_b32 s65, s87, s65\n"
        "ds_read_b128 v[16:19], v0 offset:0\n"
        "v_mfma_f32_16x16x32_f16 a[204:207], v[20:23], v[60:63], a[204:207]\n"
        "s_cmp_lg_u32 s95, s26\n"
        "s_cbranch_scc1 L_nfu11_%=\n"
        "v_mov_b32 v14, s60\n"
        "v_mov_b32 v13, 0\n"
        "s_mov_b64 exec, 1\n"
        "buffer_store_dword v14, v13, s[52:55], s28 offen offset:0 sc0 sc1\n"
        "s_mov_b64 exec, -1\n"
        "L_nfu11_%=:\n"
        "s_cmp_lg_u32 s95, s27\n"
        "s_cbranch_scc1 L_nfu12_%=\n"
        "v_mov_b32 v14, s60\n"
        "v_mov_b32 v13, 0\n"
        "s_mov_b64 exec, 1\n"
        "buffer_store_dword v14, v13, s[52:55], s28 offen offset:128 sc0 sc1\n"
        "s_mov_b64 exec, -1\n"
        "L_nfu12_%=:\n"
        "s_waitcnt lgkmcnt(5)\n"
        "v_mfma_f32_16x16x32_f16 a[208:211], v[24:27], v[56:59], a[208:211]\n"
        "ds_read_b128 v[20:23], v0 offset:1024\n"
        "buffer_load_dwordx4 v2, s[40:43], s63 offen offset:3072 sc1 lds\n"
        "s_add_u32 s64, s64, 32768\n"
        "s_cmp_ge_u32 s64, s92\n"
        "s_cselect_b32 s64, s93, s64\n"
        "v_mfma_f32_16x16x32_f16 a[212:215], v[24:27], v[60:63], a[212:215]\n"
        "s_lshr_b32 s63, s68, 2\n"
        "s_and_b32 s63, s63, 28\n"
        "v_mfma_f32_16x16x32_f16 a[216:219], v[28:31], v[56:59], a[216:219]\n"
        "ds_read_b128 v[24:27], v0 offset:2048\n"
        "s_lshr_b32 s63, s79, s63\n"
        "s_and_b32 s63, s63, 7\n"
        "v_mfma_f32_16x16x32_f16 a[220:223], v[28:31], v[60:63], a[220:223]\n"
        "s_lshl_b32 s63, s63, 4\n"
        "s_and_b32 s22, s68, 15\n"
        "s_waitcnt lgkmcnt(5)\n"
        "v_mfma_f32_16x16x32_f16 a[224:227], v[32:35], v[56:59], a[224:227]\n"
        "ds_read_b128 v[28:31], v0 offset:3072\n"
        "s_or_b32 s63, s63, s22\n"
        "s_lshl_b32 s63, s63, 15\n"
        "v_mfma_f32_16x16x32_f16 a[228:231], v[32:35], v[60:63], a[228:231]\n"
        "s_add_u32 s63, s63, s78\n"
        "s_add_u32 s68, s68, 1\n"
        "v_mfma_f32_16x16x32_f16 a[232:235], v[36:39], v[56:59], a[232:235]\n"
        "ds_read_b128 v[32:35], v0 offset:4096\n"
        "s_mov_b32 m0, s64\n"
        "v_mfma_f32_16x16x32_f16 a[236:239], v[36:39], v[60:63], a[236:239]\n"
        "s_waitcnt lgkmcnt(5)\n"
        "v_mfma_f32_16x16x32_f16 a[240:243], v[40:43], v[56:59], a[240:243]\n"
        "ds_read_b128 v[36:39], v0 offset:5120\n"
        "v_mfma_f32_16x16x32_f16 a[244:247], v[40:43], v[60:63], a[244:247]\n"
        "v_mfma_f32_16x16x32_f16 a[248:251], v[44:47], v[56:59], a[248:251]\n"
        "ds_read_b128 v[40:43], v0 offset:6144\n"
        "s_and_b32 s70, s95, 15\n"
        "s_cmp_lg_u32 s70, 9\n"
        "s_cbranch_scc1 L_d1u13_%=\n"
        "buffer_load_dwordx4 v[144:147], v1, s[56:59], 0 offen sc0 sc1\n"
        "buffer_load_dwordx4 v[148:151], v1, s[56:59], 0 offen offset:1024 sc0 sc1\n"
        "L_d1u13_%=:\n"
        "v_mfma_f32_16x16x32_f16 a[252:255], v[44:47], v[60:63], a[252:255]\n"
        "s_add_u32 s95, s95, 1\n"
        "s_cmp_lg_u32 s95, s24\n"
        "s_cbranch_scc1 L_npu15_%=\n"
        "s_add_u32 s70, s31, 0\n"
        "buffer_store_dwordx4 v[128:131], v1, s[52:55], s70 offen sc0 sc1\n"
        "s_add_u32 s70, s31, 1024\n"
        "buffer_store_dwordx4 v[132:135], v1, s[52:55], s70 offen sc0 sc1\n"
        "L_npu15_%=:\n"
        "s_cmp_lg_u32 s95, s25\n"
        "s_cbranch_scc1 L_npu16_%=\n"
        "s_add_u32 s70, s31, 262144\n"
        "buffer_store_dwordx4 v[136:139], v1, s[52:55], s70 offen sc0 sc1\n"
        "s_add_u32 s70, s31, 263168\n"
        "buffer_store_dwordx4 v[140:143], v1, s[52:55], s70 offen sc0 sc1\n"
        "L_npu16_%=:\n"
        "s_waitcnt lgkmcnt(5)\n"
        "v_mfma_f32_16x16x32_f16 a[0:3], v[16:19], v[48:51], a[0:3]\n"
        "s_waitcnt vmcnt(32)\n"
        "v_cmp_le_u32 vcc, s60, v4\n"
        "s_nop 1\n"
        "s_cmp_eq_u64 vcc, -1\n"
        "s_cbranch_scc1 L_oku14_%=\n"
        "s_memrealtime s[74:75]\n"
        "L_spinu14_%=:\n"
        "buffer_load_dword v4, v3, s[56:59], 0 offen sc0 sc1\n"
        "v_mov_b32 v14, 0\n"
        "s_mov_b32 s70, 16384\n"
        "buffer_load_dword v13, v14, s[56:59], s70 offen sc1\n"
        "s_waitcnt vmcnt(0) lgkmcnt(0)\n"
        "v_cmp_le_u32 vcc, s60, v4\n"
        "s_nop 1\n"
        "s_cmp_eq_u64 vcc, -1\n"
        "s_cbranch_scc1 L_oku14_%=\n"
        "v_readfirstlane_b32 s70, v13\n"
        "s_nop 3\n"
        "s_cmp_eq_u32 s70, s60\n"
        "s_cbranch_scc1 L_badu14_%=\n"
        "s_memrealtime s[76:77]\n"
        "s_waitcnt lgkmcnt(0)\n"
        "s_sub_u32 s70, s76, s74\n"
        "s_cmp_gt_u32 s70, 50000000\n"
        "s_cbranch_scc1 L_badu14_%=\n"
        "s_sleep 2\n"
        "s_branch L_spinu14_%=\n"
        "L_badu14_%=:\n"
        "v_mov_b32 v13, s60\n"
        "v_mov_b32 v14, 0\n"
        "s_mov_b32 s70, 16384\n"
        "buffer_store_dword v13, v14, s[56:59], s70 offen sc1\n"
        "s_waitcnt vmcnt(0)\n"
        "s_mov_b32 s69, 1\n"
        "s_mov_b64 vcc, -1\n"
        "L_oku14_%=:\n"
        "ds_read_b128 v[44:47], v0 offset:7168\n"
        "s_add_u32 s30, s68, 2\n"
        "s_lshr_b32 s66, s30, 2\n"
        "v_mfma_f32_16x16x32_f16 a[4:7], v[16:19], v[52:55], a[4:7]\n"
        "s_waitcnt vmcnt(48)\n"
        "v_cvt_pk_f16_f32 v56, v112, v114\n"
        "s_and_b32 s66, s66, 28\n"
        "s_lshr_b32 s66, s79, s66\n"
        "v_mfma_f32_16x16x32_f16 a[8:11], v[20:23], v[48:51], a[8:11]\n"
        "ds_read_b128 v[16:19], v0 offset:8192\n"
        "buffer_load_dwordx4 v2, s[40:43], s63 offen offset:0 sc1 lds\n"
        "s_and_b32 s66, s66, 7\n"
        "s_lshl_b32 s66, s66, 4\n"
        "v_mfma_f32_16x16x32_f16 a[12:15], v[20:23], v[52:55], a[12:15]\n"
        "v_cvt_pk_f16_f32 v60, v113, v115\n"
        "s_and_b32 s30, s30, 15\n"
        "s_or_b32 s66, s66, s30\n"
        "s_waitcnt lgkmcnt(5)\n"
        "v_mfma_f32_16x16x32_f16 a[16:19], v[24:27], v[48:51], a[16:19]\n"
        "ds_read_b128 v[20:23], v0 offset:9216\n"
        "s_and_b32 s66, s66, 120\n"
        "s_lshl_b32 s66, s66, 4\n"
        "v_mfma_f32_16x16x32_f16 a[20:23], v[24:27], v[52:55], a[20:23]\n"
        "s_waitcnt vmcnt(47)\n"
        "v_cvt_pk_f16_f32 v57, v116, v118\n"
        "v_add_u32 v3, s66, v153\n"
        "buffer_load_dword v4, v3, s[56:59], 0 offen sc0 sc1\n"
        "v_mfma_f32_16x16x32_f16 a[24:27], v[28:31], v[48:51], a[24:27]\n"
        "ds_read_b128 v[24:27], v0 offset:10240\n"
        "buffer_load_dwordx4 v2, s[40:43], s63 offen offset:1024 sc1 lds\n"
        "v_mfma_f32_16x16x32_f16 a[28:31], v[28:31], v[52:55], a[28:31]\n"
        "v_cvt_pk_f16_f32 v61, v117, v119\n"
        "s_waitcnt lgkmcnt(5)\n"
        "v_mfma_f32_16x16x32_f16 a[32:35], v[32:35], v[48:51], a[32:35]\n"
        "ds_read_b128 v[28:31], v0 offset:11264\n"
        "v_mfma_f32_16x16x32_f16 a[36:39], v[32:35], v[52:55], a[36:39]\n"
        "s_waitcnt vmcnt(47)\n"
        "v_cvt_pk_f16_f32 v58, v120, v122\n"
        "v_mfma_f32_16x16x32_f16 a[40:43], v[36:39], v[48:51], a[40:43]\n"
        "ds_read_b128 v[32:35], v0 offset:12288\n"
        "buffer_load_dwordx4 v2, s[40:43], s63 offen offset:2048 sc1 lds\n"
        "v_mfma_f32_16x16x32_f16 a[44:47], v[36:39], v[52:55], a[44:47]\n"
        "v_cvt_pk_f16_f32 v62, v121, v123\n"
        "s_waitcnt lgkmcnt(5)\n"
        "v_mfma_f32_16x16x32_f16 a[48:51], v[40:43], v[48:51], a[48:51]\n"
        "ds_read_b128 v[36:39], v0 offset:13312\n"
        "v_mfma_f32_16x16x32_f16 a[52:55], v[40:43], v[52:55], a[52:55]\n"
        "s_waitcnt vmcnt(46)\n"
        "v_cvt_pk_f16_f32 v59, v124, v126\n"
        "v_mfma_f32_16x16x32_f16 a[56:59], v[44:47], v[48:51], a[56:59]\n"
        "ds_read_b128 v[40:43], v0 offset:14336\n"
        "buffer_load_dwordx4 v2, s[40:43], s63 offen offset:3072 sc1 lds\n"
        "v_mfma_f32_16x16x32_f16 a[60:63], v[44:47], v[52:55], a[60:63]\n"
        "v_cvt_pk_f16_f32 v63, v125, v127\n"
        "s_waitcnt lgkmcnt(5)\n"
        "v_mfma_f32_16x16x32_f16 a[64:67], v[16:19], v[48:51], a[64:67]\n"
        "ds_read_b128 v[44:47], v0 offset:15360\n"
        "s_and_b32 s22, s88, 7\n"
        "v_mfma_f32_16x16x32_f16 a[68:71], v[16:19], v[52:55], a[68:71]\n"
        "s_cmp_eq_u32 s22, 0\n"
        "s_cselect_b32 s98, 0, s35\n"
        "v_mfma_f32_16x16x32_f16 a[72:75], v[20:23], v[48:51], a[72:75]\n"
        "ds_read_b128 v[16:19], v0 offset:16384\n"
        "s_add_u32 m0, s64, 4096\n"
        "s_add_u32 s63, s63, 4096\n"
        "s_nop 0\n"
        "buffer_load_dwordx4 v2, s[40:43], s63 offen offset:0 sc1 lds\n"
        "s_lshr_b32 s89, s88, 2\n"
        "v_mfma_f32_16x16x32_f16 a[76:79], v[20:23], v[52:55], a[76:79]\n"
        "s_and_b32 s89, s89, 28\n"
        "s_waitcnt lgkmcnt(5)\n"
        "v_mfma_f32_16x16x32_f16 a[80:83], v[24:27], v[48:51], a[80:83]\n"
        "ds_read_b128 v[20:23], v0 offset:17408\n"
        "s_lshr_b32 s89, s79, s89\n"
        "v_mfma_f32_16x16x32_f16 a[84:87], v[24:27], v[52:55], a[84:87]\n"
        "s_and_b32 s89, s89, 7\n"
        "v_mfma_f32_16x16x32_f16 a[88:91], v[28:31], v[48:51], a[88:91]\n"
        "ds_read_b128 v[24:27], v0 offset:18432\n"
        "buffer_load_dwordx4 v2, s[40:43], s63 offen offset:1024 sc1 lds\n"
        "s_lshl_b32 s89, s89, 4\n"
        "v_mfma_f32_16x16x32_f16 a[92:95], v[28:31], v[52:55], a[92:95]\n"
        "s_lshr_b32 s62, s67, 2\n"
        "s_and_b32 s22, s88, 15\n"
        "s_waitcnt lgkmcnt(5)\n"
        "v_mfma_f32_16x16x32_f16 a[96:99], v[32:35], v[48:51], a[96:99]\n"
        "ds_read_b128 v[28:31], v0 offset:19456\n"
        "s_and_b32 s62, s62, 28\n"
        "s_or_b32 s89, s89, s22\n"
        "v_mfma_f32_16x16x32_f16 a[100:103], v[32:35], v[52:55], a[100:103]\n"
        "s_lshr_b32 s62, s79, s62\n"
        "s_lshl_b32 s89, s89, 15\n"
        "v_mfma_f32_16x16x32_f16 a[104:107], v[36:39], v[48:51], a[104:107]\n"
        "ds_read_b128 v[32:35], v0 offset:20480\n"
        "buffer_load_dwordx4 v2, s[40:43], s63 offen offset:2048 sc1 lds\n"
        "s_and_b32 s62, s62, 7\n"
        "s_add_u32 s89, s89, s78\n"
        "v_mfma_f32_16x16x32_f16 a[108:111], v[36:39], v[52:55], a[108:111]\n"
        "s_lshl_b32 s62, s62, 4\n"
        "s_waitcnt lgkmcnt(5)\n"
        "v_mfma_f32_16x16x32_f16 a[112:115], v[40:43], v[48:51], a[112:115]\n"
        "ds_read_b128 v[36:39], v0 offset:21504\n"
        "s_and_b32 s23, s67, 15\n"
        "buffer_load_dword v152, v15, s[96:99], s89 offen\n"
        "s_add_u32 s88, s88, 2\n"
        "v_mfma_f32_16x16x32_f16 a[116:119], v[40:43], v[52:55], a[116:119]\n"
        "s_or_b32 s62, s62, s23\n"
        "v_mfma_f32_16x16x32_f16 a[120:123], v[44:47], v[48:51], a[120:123]\n"
        "ds_read_b128 v[40:43], v0 offset:22528\n"
        "buffer_load_dwordx4 v2, s[40:43], s63 offen offset:3072 sc1 lds\n"
        "s_add_u32 s64, s64, 32768\n"
        "s_cmp_ge_u32 s64, s92\n"
        "s_cselect_b32 s64, s93, s64\n"
        "s_lshl_b32 s62, s62, 22\n"
        "v_mfma_f32_16x16x32_f16 a[124:127], v[44:47], v[52:55], a[124:127]\n"
        "s_add_u32 s67, s67, 1\n"
        "s_waitcnt lgkmcnt(5)\n"
        "v_mfma_f32_16x16x32_f16 a[128:131], v[16:19], v[48:51], a[128:131]\n"
        "ds_read_b128 v[44:47], v0 offset:23552\n"
        "buffer_load_dwordx2 v[96:97], v5, s[36:39], s62 offen nt\n"
        "v_mfma_f32_16x16x32_f16 a[132:135], v[16:19], v[52:55], a[132:135]\n"
        "v_mfma_f32_16x16x32_f16 a[136:139], v[20:23], v[48:51], a[136:139]\n"
        "ds_read_b128 v[16:19], v0 offset:24576\n"
        "buffer_load_dwordx2 v[98:99], v6, s[36:39], s62 offen nt\n"
        "v_mfma_f32_16x16x32_f16 a[140:143], v[20:23], v[52:55], a[140:143]\n"
        "s_waitcnt lgkmcnt(5)\n"
        "v_mfma_f32_16x16x32_f16 a[144:147], v[24:27], v[48:51], a[144:147]\n"
        "ds_read_b128 v[20:23], v0 offset:25600\n"
        "buffer_load_dwordx2 v[100:101], v7, s[36:39], s62 offen nt\n"
        "v_mfma_f32_16x16x32_f16 a[148:151], v[24:27], v[52:55], a[148:151]\n"
        "v_mfma_f32_16x16x32_f16 a[152:155], v[28:31], v[48:51], a[152:155]\n"
        "ds_read_b128 v[24:27], v0 offset:26624\n"
        "buffer_load_dwordx2 v[102:103], v8, s[36:39], s62 offen nt\n"
        "v_mfma_f32_16x16x32_f16 a[156:159], v[28:31], v[52:55], a[156:159]\n"
        "s_waitcnt lgkmcnt(5)\n"
        "v_mfma_f32_16x16x32_f16 a[160:163], v[32:35], v[48:51], a[160:163]\n"
        "ds_read_b128 v[28:31], v0 offset:27648\n"
        "buffer_load_dwordx2 v[104:105], v9, s[36:39], s62 offen nt\n"
        "v_mfma_f32_16x16x32_f16 a[164:167], v[32:35], v[52:55], a[164:167]\n"
        "v_mfma_f32_16x16x32_f16 a[168:171], v[36:39], v[48:51], a[168:171]\n"
        "ds_read_b128 v[32:35], v0 offset:28672\n"
        "buffer_load_dwordx2 v[106:107], v10, s[36:39], s62 offen nt\n"
        "v_mfma_f32_16x16x32_f16 a[172:175], v[36:39], v[52:55], a[172:175]\n"
        "s_waitcnt lgkmcnt(5)\n"
        "v_mfma_f32_16x16x32_f16 a[176:179], v[40:43], v[48:51], a[176:179]\n"
        "ds_read_b128 v[36:39], v0 offset:29696\n"
        "buffer_load_dwordx2 v[108:109], v11, s[36:39], s62 offen nt\n"
        "v_mfma_f32_16x16x32_f16 a[180:183], v[40:43], v[52:55], a[180:183]\n"
        "v_mfma_f32_16x16x32_f16 a[184:187], v[44:47], v[48:51], a[184:187]\n"
        "ds_read_b128 v[40:43], v0 offset:30720\n"
        "buffer_load_dwordx2 v[110:111], v12, s[36:39], s62 offen nt\n"
        "v_mfma_f32_16x16x32_f16 a[188:191], v[44:47], v[52:55], a[188:191]\n"
        "s_waitcnt lgkmcnt(5)\n"
        "v_mfma_f32_16x16x32_f16 a[192:195], v[16:19], v[48:51], a[192:195]\n"
        "ds_read_b128 v[44:47], v0 offset:31744\n"
        "s_lshr_b32 s62, s67, 2\n"
        "s_and_b32 s62, s62, 28\n"
        "s_lshr_b32 s62, s79, s62\n"
        "s_and_b32 s62, s62, 7\n"
        "s_lshl_b32 s62, s62, 4\n"
        "s_and_b32 s23, s67, 15\n"
        "s_or_b32 s62, s62, s23\n"
        "s_lshl_b32 s62, s62, 22\n"
        "s_add_u32 s67, s67, 1\n"
        "buffer_load_dwordx2 v[112:113], v5, s[36:39], s62 offen nt\n"
        "v_mfma_f32_16x16x32_f16 a[196:199], v[16:19], v[52:55], a[196:199]\n"
        "v_mfma_f32_16x16x32_f16 a[200:203], v[20:23], v[48:51], a[200:203]\n"
        "s_waitcnt vmcnt(19)\n"
        "s_barrier\n"
        "v_add_u32 v0, s65, v1\n"
        "s_add_u32 s65, s65, 32768\n"
        "s_cmp_ge_u32 s65, s94\n"
        "s_cselect_b32 s65, s87, s65\n"
        "ds_read_b128 v[16:19], v0 offset:0\n"
        "buffer_load_dwordx2 v[114:115], v6, s[36:39], s62 offen nt\n"
        "v_mfma_f32_16x16x32_f16 a[204:207], v[20:23], v[52:55], a[204:207]\n"
        "s_cmp_lg_u32 s95, s26\n"
        "s_cbranch_scc1 L_nfu17_%=\n"
        "v_mov_b32 v14, s60\n"
        "v_mov_b32 v13, 0\n"
        "s_mov_b64 exec, 1\n"
        "buffer_store_dword v14, v13, s[52:55], s28 offen offset:0 sc0 sc1\n"
        "s_mov_b64 exec, -1\n"
        "L_nfu17_%=:\n"
        "s_cmp_lg_u32 s95, s27\n"
        "s_cbranch_scc1 L_nfu18_%=\n"
        "v_mov_b32 v14, s60\n"
        "v_mov_b32 v13, 0\n"
        "s_mov_b64 exec, 1\n"
        "buffer_store_dword v14, v13, s[52:55], s28 offen offset:128 sc0 sc1\n"
        "s_mov_b64 exec, -1\n"
        "L_nfu18_%=:\n"
        "s_waitcnt lgkmcnt(5)\n"
        "v_mfma_f32_16x16x32_f16 a[208:211], v[24:27], v[48:51], a[208:211]\n"
        "ds_read_b128 v[20:23], v0 offset:1024\n"
        "buffer_load_dwordx2 v[116:117], v7, s[36:39], s62 offen nt\n"
        "v_mfma_f32_16x16x32_f16 a[212:215], v[24:27], v[52:55], a[212:215]\n"
        "v_mfma_f32_16x16x32_f16 a[216:219], v[28:31], v[48:51], a[216:219]\n"
        "ds_read_b128 v[24:27], v0 offset:2048\n"
        "buffer_load_dwordx2 v[118:119], v8, s[36:39], s62 offen nt\n"
        "v_mfma_f32_16x16x32_f16 a[220:223], v[28:31], v[52:55], a[220:223]\n"
        "s_waitcnt lgkmcnt(5)\n"
        "v_mfma_f32_16x16x32_f16 a[224:227], v[32:35], v[48:51], a[224:227]\n"
        "ds_read_b128 v[28:31], v0 offset:3072\n"
        "buffer_load_dwordx2 v[120:121], v9, s[36:39], s62 offen nt\n"
        "v_mfma_f32_16x16x32_f16 a[228:231], v[32:35], v[52:55], a[228:231]\n"
        "v_mfma_f32_16x16x32_f16 a[232:235], v[36:39], v[48:51], a[232:235]\n"
        "ds_read_b128 v[32:35], v0 offset:4096\n"
        "buffer_load_dwordx2 v[122:123], v10, s[36:39], s62 offen nt\n"
        "v_mfma_f32_16x16x32_f16 a[236:239], v[36:39], v[52:55], a[236:239]\n"
        "s_waitcnt lgkmcnt(5)\n"
        "v_mfma_f32_16x16x32_f16 a[240:243], v[40:43], v[48:51], a[240:243]\n"
        "ds_read_b128 v[36:39], v0 offset:5120\n"
        "buffer_load_dwordx2 v[124:125], v11, s[36:39], s62 offen nt\n"
        "v_mfma_f32_16x16x32_f16 a[244:247], v[40:43], v[52:55], a[244:247]\n"
        "v_mfma_f32_16x16x32_f16 a[248:251], v[44:47], v[48:51], a[248:251]\n"
        "ds_read_b128 v[40:43], v0 offset:6144\n"
        "buffer_load_dwordx2 v[126:127], v12, s[36:39], s62 offen nt\n"
        "v_mfma_f32_16x16x32_f16 a[252:255], v[44:47], v[52:55], a[252:255]\n"
        "s_add_u32 s95, s95, 1\n"
        "s_cmp_lg_u32 s95, s24\n"
        "s_cbranch_scc1 L_npu19_%=\n"
        "s_add_u32 s70, s31, 0\n"
        "buffer_store_dwordx4 v[128:131], v1, s[52:55], s70 offen sc0 sc1\n"
        "s_add_u32 s70, s31, 1024\n"
        "buffer_store_dwordx4 v[132:135], v1, s[52:55], s70 offen sc0 sc1\n"
        "L_npu19_%=:\n"
        "s_cmp_lg_u32 s95, s25\n"
        "s_cbranch_scc1 L_npu20_%=\n"
        "s_add_u32 s70, s31, 262144\n"
        "buffer_store_dwordx4 v[136:139], v1, s[52:55], s70 offen sc0 sc1\n"
        "s_add_u32 s70, s31, 263168\n"
        "buffer_store_dwordx4 v[140:143], v1, s[52:55], s70 offen sc0 sc1\n"
        "L_npu20_%=:\n"
        "s_waitcnt lgkmcnt(5)\n"
        "v_mfma_f32_16x16x32_f16 a[0:3], v[16:19], v[56:59], a[0:3]\n"
        "ds_read_b128 v[44:47], v0 offset:7168\n"
        "v_mfma_f32_16x16x32_f16 a[4:7], v[16:19], v[60:63], a[4:7]\n"
        "s_waitcnt vmcnt(48)\n"
        "v_cvt_pk_f16_f32 v48, v64, v66\n"
        "v_mfma_f32_16x16x32_f16 a[8:11], v[20:23], v[56:59], a[8:11]\n"
        "ds_read_b128 v[16:19], v0 offset:8192\n"
        "v_mfma_f32_16x16x32_f16 a[12:15], v[20:23], v[60:63], a[12:15]\n"
        "v_cvt_pk_f16_f32 v52, v65, v67\n"
        "s_waitcnt lgkmcnt(5)\n"
        "v_mfma_f32_16x16x32_f16 a[16:19], v[24:27], v[56:59], a[16:19]\n"
        "ds_read_b128 v[20:23], v0 offset:9216\n"
        "v_mfma_f32_16x16x32_f16 a[20:23], v[24:27], v[60:63], a[20:23]\n"
        "s_waitcnt vmcnt(46)\n"
        "v_cvt_pk_f16_f32 v49, v68, v70\n"
        "v_mfma_f32_16x16x32_f16 a[24:27], v[28:31], v[56:59], a[24:27]\n"
        "ds_read_b128 v[24:27], v0 offset:10240\n"
        "v_mfma_f32_16x16x32_f16 a[28:31], v[28:31], v[60:63], a[28:31]\n"
        "v_cvt_pk_f16_f32 v53, v69, v71\n"
        "s_waitcnt lgkmcnt(5)\n"
        "v_mfma_f32_16x16x32_f16 a[32:35], v[32:35], v[56:59], a[32:35]\n"
        "ds_read_b128 v[28:31], v0 offset:11264\n"
        "v_mfma_f32_16x16x32_f16 a[36:39], v[32:35], v[60:63], a[36:39]\n"
        "s_waitcnt vmcnt(44)\n"
        "v_cvt_pk_f16_f32 v50, v72, v74\n"
        "v_mfma_f32_16x16x32_f16 a[40:43], v[36:39], v[56:59], a[40:43]\n"
        "ds_read_b128 v[32:35], v0 offset:12288\n"
        "v_mfma_f32_16x16x32_f16 a[44:47], v[36:39], v[60:63], a[44:47]\n"
        "v_cvt_pk_f16_f32 v54, v73, v75\n"
        "s_waitcnt lgkmcnt(5)\n"
        "v_mfma_f32_16x16x32_f16 a[48:51], v[40:43], v[56:59], a[48:51]\n"
        "ds_read_b128 v[36:39], v0 offset:13312\n"
        "s_lshr_b32 s63, s68, 2\n"
        "v_mfma_f32_16x16x32_f16 a[52:55], v[40:43], v[60:63], a[52:55]\n"
        "s_waitcnt vmcnt(42)\n"
        "v_cvt_pk_f16_f32 v51, v76, v78\n"
        "s_and_b32 s63, s63, 28\n"
        "v_mfma_f32_16x16x32_f16 a[56:59], v[44:47], v[56:59], a[56:59]\n"
        "ds_read_b128 v[40:43], v0 offset:14336\n"
        "s_lshr_b32 s63, s79, s63\n"
        "v_mfma_f32_16x16x32_f16 a[60:63], v[44:47], v[60:63], a[60:63]\n"
        "v_cvt_pk_f16_f32 v55, v77, v79\n"
        "s_and_b32 s63, s63, 7\n"
        "s_waitcnt lgkmcnt(5)\n"
        "v_mfma_f32_16x16x32_f16 a[64:67], v[16:19], v[56:59], a[64:67]\n"
        "ds_read_b128 v[44:47], v0 offset:15360\n"
        "s_lshl_b32 s63, s63, 4\n"
        "v_mfma_f32_16x16x32_f16 a[68:71], v[16:19], v[60:63], a[68:71]\n"
        "s_and_b32 s22, s68, 15\n"
        "v_mfma_f32_16x16x32_f16 a[72:75], v[20:23], v[56:59], a[72:75]\n"
        "ds_read_b128 v[16:19], v0 offset:16384\n"
        "s_or_b32 s63, s63, s22\n"
        "v_mfma_f32_16x16x32_f16 a[76:79], v[20:23], v[60:63], a[76:79]\n"
        "s_lshl_b32 s63, s63, 15\n"
        "s_waitcnt lgkmcnt(5)\n"
        "v_mfma_f32_16x16x32_f16 a[80:83], v[24:27], v[56:59], a[80:83]\n"
        "ds_read_b128 v[20:23], v0 offset:17408\n"
        "s_add_u32 s63, s63, s78\n"
        "v_mfma_f32_16x16x32_f16 a[84:87], v[24:27], v[60:63], a[84:87]\n"
        "s_add_u32 s68, s68, 1\n"
        "v_mfma_f32_16x16x32_f16 a[88:91], v[28:31], v[56:59], a[88:91]\n"
        "ds_read_b128 v[24:27], v0 offset:18432\n"
        "s_mov_b32 m0, s64\n"
        "v_mfma_f32_16x16x32_f16 a[92:95], v[28:31], v[60:63], a[92:95]\n"
        "s_waitcnt lgkmcnt(5)\n"
        "v_mfma_f32_16x16x32_f16 a[96:99], v[32:35], v[56:59], a[96:99]\n"
        "ds_read_b128 v[28:31], v0 offset:19456\n"
        "buffer_load_dwordx4 v2, s[40:43], s63 offen offset:0 sc1 lds\n"
        "v_mfma_f32_16x16x32_f16 a[100:103], v[32:35], v[60:63], a[100:103]\n"
        "v_mfma_f32_16x16x32_f16 a[104:107], v[36:39], v[56:59], a[104:107]\n"
        "ds_read_b128 v[32:35], v0 offset:20480\n"
        "v_mfma_f32_16x16x32_f16 a[108:111], v[36:39], v[60:63], a[108:111]\n"
        "s_waitcnt lgkmcnt(5)\n"
        "v_mfma_f32_16x16x32_f16 a[112:115], v[40:43], v[56:59], a[112:115]\n"
        "ds_read_b128 v[36:39], v0 offset:21504\n"
        "buffer_load_dwordx4 v2, s[40:43], s63 offen offset:1024 sc1 lds\n"
        "v_mfma_f32_16x16x32_f16 a[116:119], v[40:43], v[60:63], a[116:119]\n"
        "v_mfma_f32_16x16x32_f16 a[120:123], v[44:47], v[56:59], a[120:123]\n"
        "ds_read_b128 v[40:43], v0 offset:22528\n"
        "v_mfma_f32_16x16x32_f16 a[124:127], v[44:47], v[60:63], a[124:127]\n"
        "s_waitcnt lgkmcnt(5)\n"
        "v_mfma_f32_16x16x32_f16 a[128:131], v[16:19], v[56:59], a[128:131]\n"
        "ds_read_b128 v[44:47], v0 offset:23552\n"
        "buffer_load_dwordx4 v2, s[40:43], s63 offen offset:2048 sc1 lds\n"
        "v_mfma_f32_16x16x32_f16 a[132:135], v[16:19], v[60:63], a[132:135]\n"
        "v_mfma_f32_16x16x32_f16 a[136:139], v[20:23], v[56:59], a[136:139]\n"
        "ds_read_b128 v[16:19], v0 offset:24576\n"
        "v_mfma_f32_16x16x32_f16 a[140:143], v[20:23], v[60:63], a[140:143]\n"
        "s_waitcnt lgkmcnt(5)\n"
        "v_mfma_f32_16x16x32_f16 a[144:147], v[24:27], v[56:59], a[144:147]\n"
        "ds_read_b128 v[20:23], v0 offset:25600\n"
        "buffer_load_dwordx4 v2, s[40:43], s63 offen offset:3072 sc1 lds\n"
        "v_mfma_f32_16x16x32_f16 a[148:151], v[24:27], v[60:63], a[148:151]\n"
        "v_mfma_f32_16x16x32_f16 a[152:155], v[28:31], v[56:59], a[152:155]\n"
        "ds_read_b128 v[24:27], v0 offset:26624\n"
        "v_mfma_f32_16x16x32_f16 a[156:159], v[28:31], v[60:63], a[156:159]\n"
        "s_waitcnt lgkmcnt(5)\n"
        "v_mfma_f32_16x16x32_f16 a[160:163], v[32:35], v[56:59], a[160:163]\n"
        "ds_read_b128 v[28:31], v0 offset:27648\n"
        "s_add_u32 m0, s64, 4096\n"
        "s_add_u32 s63, s63, 4096\n"
        "s_nop 0\n"
        "buffer_load_dwordx4 v2, s[40:43], s63 offen offset:0 sc1 lds\n"
        "v_mfma_f32_16x16x32_f16 a[164:167], v[32:35], v[60:63], a[164:167]\n"
        "v_mfma_f32_16x16x32_f16 a[168:171], v[36:39], v[56:59], a[168:171]\n"
        "ds_read_b128 v[32:35], v0 offset:28672\n"
        "v_mfma_f32_16x16x32_f16 a[172:175], v[36:39], v[60:63], a[172:175]\n"
        "s_waitcnt lgkmcnt(5)\n"
        "v_mfma_f32_16x16x32_f16 a[176:179], v[40:43], v[56:59], a[176:179]\n"
        "ds_read_b128 v[36:39], v0 offset:29696\n"
        "buffer_load_dwordx4 v2, s[40:43], s63 offen offset:1024 sc1 lds\n"
        "v_mfma_f32_16x16x32_f16 a[180:183], v[40:43], v[60:63], a[180:183]\n"
        "v_mfma_f32_16x16x32_f16 a[184:187], v[44:47], v[56:59], a[184:187]\n"
        "ds_read_b128 v[40:43], v0 offset:30720\n"
        "v_mfma_f32_16x16x32_f16 a[188:191], v[44:47], v[60:63], a[188:191]\n"
        "s_waitcnt lgkmcnt(5)\n"
        "v_mfma_f32_16x16x32_f16 a[192:195], v[16:19], v[56:59], a[192:195]\n"
        "ds_read_b128 v[44:47], v0 offset:31744\n"
        "buffer_load_dwordx4 v2, s[40:43], s63 offen offset:2048 sc1 lds\n"
        "v_mfma_f32_16x16x32_f16 a[196:199], v[16:19], v[60:63], a[196:199]\n"
        "v_mfma_f32_16x16x32_f16 a[200:203], v[20:23], v[56:59], a[200:203]\n"
        "s_waitcnt vmcnt(23)\n"
        "s_and_b32 s70, s95, 15\n"
        "s_cmp_lg_u32 s70, 11\n"
        "s_cbranch_scc1 L_d2u21_%=\n"
        "s_cmp_lg_u32 s83, 0\n"
        "s_cbranch_scc1 L_d2u21_%=\n"
        "s_memrealtime s[74:75]\n"
        "L_d2ru21_%=:\n"
        "v_min_u32 v144, v144, v145\n"
        "v_min3_u32 v144, v144, v146, v147\n"
        "v_min_u32 v148, v148, v149\n"
        "v_min3_u32 v148, v148, v150, v151\n"
        "v_cmp_le_u32 s[32:33], s60, v144\n"
        "v_cmp_le_u32 s[20:21], s60, v148\n"
        "s_nop 3\n"
        "s_mov_b32 s71, 0\n"
        "s_bfe_u32 s72, s32, 0x80000\n"
        "s_cmp_eq_u32 s72, 0xff\n"
        "s_cselect_b32 s72, 1, 0\n"
        "s_or_b32 s71, s71, s72\n"
        "s_bfe_u32 s72, s32, 0x80010\n"
        "s_cmp_eq_u32 s72, 0xff\n"
        "s_cselect_b32 s72, 2, 0\n"
        "s_or_b32 s71, s71, s72\n"
        "s_bfe_u32 s72, s33, 0x80000\n"
        "s_cmp_eq_u32 s72, 0xff\n"
        "s_cselect_b32 s72, 4, 0\n"
        "s_or_b32 s71, s71, s72\n"
        "s_bfe_u32 s72, s33, 0x80010\n"
        "s_cmp_eq_u32 s72, 0xff\n"
        "s_cselect_b32 s72, 8, 0\n"
        "s_or_b32 s71, s71, s72\n"
        "s_bfe_u32 s72, s20, 0x80000\n"
        "s_cmp_eq_u32 s72, 0xff\n"
        "s_cselect_b32 s72, 16, 0\n"
        "s_or_b32 s71, s71, s72\n"
        "s_bfe_u32 s72, s20, 0x80010\n"
        "s_cmp_eq_u32 s72, 0xff\n"
        "s_cselect_b32 s72, 32, 0\n"
        "s_or_b32 s71, s71, s72\n"
        "s_bfe_u32 s72, s21, 0x80000\n"
        "s_cmp_eq_u32 s72, 0xff\n"
        "s_cselect_b32 s72, 64, 0\n"
        "s_or_b32 s71, s71, s72\n"
        "s_bfe_u32 s72, s21, 0x80010\n"
        "s_cmp_eq_u32 s72, 0xff\n"
        "s_cselect_b32 s72, 128, 0\n"
        "s_or_b32 s71, s71, s72\n"
        "s_andn2_b32 s71, s71, s29\n"
        "s_not_b32 s72, s29\n"
        "s_and_b32 s72, s72, 0xff\n"
        "s_cmp_lg_u32 s71, 0\n"
        "s_cbranch_scc1 L_d2pu21_%=\n"
        "s_cmp_eq_u32 s72, 0\n"
        "s_cbranch_scc1 L_d2pu21_%=\n"
        "s_cmp_lg_u32 s69, 0\n"
        "s_cbranch_scc1 L_d2pu21_%=\n"
        "s_memrealtime s[76:77]\n"
        "s_waitcnt lgkmcnt(0)\n"
        "s_sub_u32 s73, s76, s74\n"
        "s_cmp_gt_u32 s73, 50000000\n"
        "s_cbranch_scc1 L_d2pu21_%=\n"
        "s_sleep 1\n"
        "buffer_load_dwordx4 v[144:147], v1, s[56:59], 0 offen sc0 sc1\n"
        "buffer_load_dwordx4 v[148:151], v1, s[56:59], 0 offen offset:1024 sc0 sc1\n"
        "v_mov_b32 v14, 0\n"
        "s_mov_b32 s73, 16384\n"
        "buffer_load_dword v13, v14, s[56:59], s73 offen sc1\n"
        "s_waitcnt vmcnt(0)\n"
        "v_readfirstlane_b32 s73, v13\n"
        "s_nop 3\n"
        "s_cmp_eq_u32 s73, s60\n"
        "s_cbranch_scc0 L_d2ru21_%=\n"
        "L_d2pu21_%=:\n"
        "s_cmp_lg_u32 s71, 0\n"
        "s_cselect_b32 s71, s71, s72\n"
        "s_lshl_b32 s72, s71, 8\n"
        "s_or_b32 s71, s71, s72\n"
        "s_add_u32 s72, s84, 1\n"
        "s_lshr_b32 s71, s71, s72\n"
        "s_ff1_i32_b32 s71, s71\n"
        "s_add_u32 s71, s71, s72\n"
        "s_and_b32 s71, s71, 7\n"
        "v_mov_b32 v14, s71\n"
        "v_mov_b32 v13, s87\n"
        "v_add_u32 v13, 131072, v13\n"
        "ds_write_b32 v13, v14\n"
        "s_waitcnt lgkmcnt(0)\n"
        "L_d2u21_%=:\n"
        "s_barrier\n"
        "v_add_u32 v0, s65, v1\n"
        "s_add_u32 s65, s65, 32768\n"
        "s_cmp_ge_u32 s65, s94\n"
        "s_cselect_b32 s65, s87, s65\n"
        "ds_read_b128 v[16:19], v0 offset:0\n"
        "v_mfma_f32_16x16x32_f16 a[204:207], v[20:23], v[60:63], a[204:207]\n"
        "s_cmp_lg_u32 s95, s26\n"
        "s_cbranch_scc1 L_nfu22_%=\n"
        "v_mov_b32 v14, s60\n"
        "v_mov_b32 v13, 0\n"
        "s_mov_b64 exec, 1\n"
        "buffer_store_dword v14, v13, s[52:55], s28 offen offset:0 sc0 sc1\n"
        "s_mov_b64 exec, -1\n"
        "L_nfu22_%=:\n"
        "s_cmp_lg_u32 s95, s27\n"
        "s_cbranch_scc1 L_nfu23_%=\n"
        "v_mov_b32 v14, s60\n"
        "v_mov_b32 v13, 0\n"
        "s_mov_b64 exec, 1\n"
        "buffer_store_dword v14, v13, s[52:55], s28 offen offset:128 sc0 sc1\n"
        "s_mov_b64 exec, -1\n"
        "L_nfu23_%=:\n"
        "s_waitcnt lgkmcnt(5)\n"
        "v_mfma_f32_16x16x32_f16 a[208:211], v[24:27], v[56:59], a[208:211]\n"
        "ds_read_b128 v[20:23], v0 offset:1024\n"
        "buffer_load_dwordx4 v2, s[40:43], s63 offen offset:3072 sc1 lds\n"
        "s_add_u32 s64, s64, 32768\n"
        "s_cmp_ge_u32 s64, s92\n"
        "s_cselect_b32 s64, s93, s64\n"
        "v_mfma_f32_16x16x32_f16 a[212:215], v[24:27], v[60:63], a[212:215]\n"
        "s_lshr_b32 s63, s68, 2\n"
        "s_and_b32 s63, s63, 28\n"
        "v_mfma_f32_16x16x32_f16 a[216:219], v[28:31], v[56:59], a[216:219]\n"
        "ds_read_b128 v[24:27], v0 offset:2048\n"
        "s_lshr_b32 s63, s79, s63\n"
        "s_and_b32 s63, s63, 7\n"
        "v_mfma_f32_16x16x32_f16 a[220:223], v[28:31], v[60:63], a[220:223]\n"
        "s_lshl_b32 s63, s63, 4\n"
        "s_and_b32 s22, s68, 15\n"
        "s_waitcnt lgkmcnt(5)\n"
        "v_mfma_f32_16x16x32_f16 a[224:227], v[32:35], v[56:59], a[224:227]\n"
        "ds_read_b128 v[28:31], v0 offset:3072\n"
        "s_or_b32 s63, s63, s22\n"
        "s_lshl_b32 s63, s63, 15\n"
        "v_mfma_f32_16x16x32_f16 a[228:231], v[32:35], v[60:63], a[228:231]\n"
        "s_add_u32 s63, s63, s78\n"
        "s_add_u32 s68, s68, 1\n"
        "v_mfma_f32_16x16x32_f16 a[232:235], v[36:39], v[56:59], a[232:235]\n"
        "ds_read_b128 v[32:35], v0 offset:4096\n"
        "s_mov_b32 m0, s64\n"
        "v_mfma_f32_16x16x32_f16 a[236:239], v[36:39], v[60:63], a[236:239]\n"
        "s_waitcnt lgkmcnt(5)\n"
        "v_mfma_f32_16x16x32_f16 a[240:243], v[40:43], v[56:59], a[240:243]\n"
        "ds_read_b128 v[36:39], v0 offset:5120\n"
        "v_mfma_f32_16x16x32_f16 a[244:247], v[40:43], v[60:63], a[244:247]\n"
        "v_mfma_f32_16x16x32_f16 a[248:251], v[44:47], v[56:59], a[248:251]\n"
        "ds_read_b128 v[40:43], v0 offset:6144\n"
        "v_mfma_f32_16x16x32_f16 a[252:255], v[44:47], v[60:63], a[252:255]\n"
        "s_add_u32 s95, s95, 1\n"
        "s_add_u32 s61, s61, 1\n"
        "s_cmp_lt_u32 s61, 4\n"
        "s_cbranch_scc1 L_loop1_%=\n"
        ".p2align 6\n"
        "L_loop2_%=:\n"
        "s_and_b32 s70, s95, 15\n"
        "s_cmp_lg_u32 s70, 12\n"
        "s_cbranch_scc1 L_d3u25_%=\n"
        "s_cmp_ge_u32 s95, 112\n"
        "s_cbranch_scc1 L_d3u25_%=\n"
        "v_mov_b32 v13, s87\n"
        "v_add_u32 v13, 131072, v13\n"
        "ds_read_b32 v14, v13\n"
        "s_waitcnt lgkmcnt(0)\n"
        "v_readfirstlane_b32 s71, v14\n"
        "s_nop 3\n"
        "s_lshr_b32 s72, s95, 4\n"
        "s_add_u32 s72, s72, 1\n"
        "s_lshl_b32 s72, s72, 2\n"
        "s_lshl_b32 s73, 15, s72\n"
        "s_andn2_b32 s79, s79, s73\n"
        "s_lshl_b32 s73, s71, s72\n"
        "s_or_b32 s79, s79, s73\n"
        "s_lshl_b32 s73, 1, s71\n"
        "s_or_b32 s29, s29, s73\n"
        "L_d3u25_%=:\n"
        "s_cmp_ge_u32 s61, 31\n"
        "s_cselect_b32 s38, 0, s38\n"
        "s_cmp_ge_u32 s61, 31\n"
        "s_cselect_b32 s35, 0, s35\n"
        "s_waitcnt lgkmcnt(5)\n"
        "v_mfma_f32_16x16x32_f16 a[0:3], v[16:19], v[48:51], a[0:3]\n"
        "s_waitcnt vmcnt(32)\n"
        "v_cmp_le_u32 vcc, s60, v4\n"
        "s_nop 1\n"
        "s_cmp_eq_u64 vcc, -1\n"
        "s_cbranch_scc1 L_oku24_%=\n"
        "s_memrealtime s[74:75]\n"
        "L_spinu24_%=:\n"
        "buffer_load_dword v4, v3, s[56:59], 0 offen sc0 sc1\n"
        "v_mov_b32 v14, 0\n"
        "s_mov_b32 s70, 16384\n"
        "buffer_load_dword v13, v14, s[56:59], s70 offen sc1\n"
        "s_waitcnt vmcnt(0) lgkmcnt(0)\n"
        "v_cmp_le_u32 vcc, s60, v4\n"
        "s_nop 1\n"
        "s_cmp_eq_u64 vcc, -1\n"
        "s_cbranch_scc1 L_oku24_%=\n"
        "v_readfirstlane_b32 s70, v13\n"
        "s_nop 3\n"
        "s_cmp_eq_u32 s70, s60\n"
        "s_cbranch_scc1 L_badu24_%=\n"
        "s_memrealtime s[76:77]\n"
        "s_waitcnt lgkmcnt(0)\n"
        "s_sub_u32 s70, s76, s74\n"
        "s_cmp_gt_u32 s70, 50000000\n"
        "s_cbranch_scc1 L_badu24_%=\n"
        "s_sleep 2\n"
        "s_branch L_spinu24_%=\n"
        "L_badu24_%=:\n"
        "v_mov_b32 v13, s60\n"
        "v_mov_b32 v14, 0\n"
        "s_mov_b32 s70, 16384\n"
        "buffer_store_dword v13, v14, s[56:59], s70 offen sc1\n"
        "s_waitcnt vmcnt(0)\n"
        "s_mov_b32 s69, 1\n"
        "s_mov_b64 vcc, -1\n"
        "L_oku24_%=:\n"
        "ds_read_b128 v[44:47], v0 offset:7168\n"
        "s_add_u32 s30, s68, 2\n"
        "s_lshr_b32 s66, s30, 2\n"
        "v_mfma_f32_16x16x32_f16 a[4:7], v[16:19], v[52:55], a[4:7]\n"
        "s_waitcnt vmcnt(48)\n"
        "v_cvt_pk_f16_f32 v56, v80, v82\n"
        "s_and_b32 s66, s66, 28\n"
        "s_lshr_b32 s66, s79, s66\n"
        "v_mfma_f32_16x16x32_f16 a[8:11], v[20:23], v[48:51], a[8:11]\n"
        "ds_read_b128 v[16:19], v0 offset:8192\n"
        "buffer_load_dwordx4 v2, s[40:43], s63 offen offset:0 sc1 lds\n"
        "s_and_b32 s66, s66, 7\n"
        "s_lshl_b32 s66, s66, 4\n"
        "v_mfma_f32_16x16x32_f16 a[12:15], v[20:23], v[52:55], a[12:15]\n"
        "v_cvt_pk_f16_f32 v60, v81, v83\n"
        "s_and_b32 s30, s30, 15\n"
        "s_or_b32 s66, s66, s30\n"
        "s_waitcnt lgkmcnt(5)\n"
        "v_mfma_f32_16x16x32_f16 a[16:19], v[24:27], v[48:51], a[16:19]\n"
        "ds_read_b128 v[20:23], v0 offset:9216\n"
        "s_and_b32 s66, s66, 120\n"
        "s_lshl_b32 s66, s66, 4\n"
        "v_mfma_f32_16x16x32_f16 a[20:23], v[24:27], v[52:55], a[20:23]\n"
        "s_waitcnt vmcnt(47)\n"
        "v_cvt_pk_f16_f32 v57, v84, v86\n"
        "v_add_u32 v3, s66, v153\n"
        "buffer_load_dword v4, v3, s[56:59], 0 offen sc0 sc1\n"
        "v_mfma_f32_16x16x32_f16 a[24:27], v[28:31], v[48:51], a[24:27]\n"
        "ds_read_b128 v[24:27], v0 offset:10240\n"
        "buffer_load_dwordx4 v2, s[40:43], s63 offen offset:1024 sc1 lds\n"
        "v_mfma_f32_16x16x32_f16 a[28:31], v[28:31], v[52:55], a[28:31]\n"
        "v_cvt_pk_f16_f32 v61, v85, v87\n"
        "s_waitcnt lgkmcnt(5)\n"
        "v_mfma_f32_16x16x32_f16 a[32:35], v[32:35], v[48:51], a[32:35]\n"
        "ds_read_b128 v[28:31], v0 offset:11264\n"
        "v_mfma_f32_16x16x32_f16 a[36:39], v[32:35], v[52:55], a[36:39]\n"
        "s_waitcnt vmcnt(47)\n"
        "v_cvt_pk_f16_f32 v58, v88, v90\n"
        "v_mfma_f32_16x16x32_f16 a[40:43], v[36:39], v[48:51], a[40:43]\n"
        "ds_read_b128 v[32:35], v0 offset:12288\n"
        "buffer_load_dwordx4 v2, s[40:43], s63 offen offset:2048 sc1 lds\n"
        "v_mfma_f32_16x16x32_f16 a[44:47], v[36:39], v[52:55], a[44:47]\n"
        "v_cvt_pk_f16_f32 v62, v89, v91\n"
        "s_waitcnt lgkmcnt(5)\n"
        "v_mfma_f32_16x16x32_f16 a[48:51], v[40:43], v[48:51], a[48:51]\n"
        "ds_read_b128 v[36:39], v0 offset:13312\n"
        "v_mfma_f32_16x16x32_f16 a[52:55], v[40:43], v[52:55], a[52:55]\n"
        "s_waitcnt vmcnt(46)\n"
        "v_cvt_pk_f16_f32 v59, v92, v94\n"
        "v_mfma_f32_16x16x32_f16 a[56:59], v[44:47], v[48:51], a[56:59]\n"
        "ds_read_b128 v[40:43], v0 offset:14336\n"
        "buffer_load_dwordx4 v2, s[40:43], s63 offen offset:3072 sc1 lds\n"
        "v_mfma_f32_16x16x32_f16 a[60:63], v[44:47], v[52:55], a[60:63]\n"
        "v_cvt_pk_f16_f32 v63, v93, v95\n"
        "s_waitcnt lgkmcnt(5)\n"
        "v_mfma_f32_16x16x32_f16 a[64:67], v[16:19], v[48:51], a[64:67]\n"
        "ds_read_b128 v[44:47], v0 offset:15360\n"
        "s_and_b32 s22, s88, 7\n"
        "v_mfma_f32_16x16x32_f16 a[68:71], v[16:19], v[52:55], a[68:71]\n"
        "s_cmp_eq_u32 s22, 0\n"
        "s_cselect_b32 s98, 0, s35\n"
        "v_mfma_f32_16x16x32_f16 a[72:75], v[20:23], v[48:51], a[72:75]\n"
        "ds_read_b128 v[16:19], v0 offset:16384\n"
        "s_add_u32 m0, s64, 4096\n"
        "s_add_u32 s63, s63, 4096\n"
        "s_nop 0\n"
        "buffer_load_dwordx4 v2, s[40:43], s63 offen offset:0 sc1 lds\n"
        "s_lshr_b32 s89, s88, 2\n"
        "v_mfma_f32_16x16x32_f16 a[76:79], v[20:23], v[52:55], a[76:79]\n"
        "s_and_b32 s89, s89, 28\n"
        "s_waitcnt lgkmcnt(5)\n"
        "v_mfma_f32_16x16x32_f16 a[80:83], v[24:27], v[48:51], a[80:83]\n"
        "ds_read_b128 v[20:23], v0 offset:17408\n"
        "s_lshr_b32 s89, s79, s89\n"
        "v_mfma_f32_16x16x32_f16 a[84:87], v[24:27], v[52:55], a[84:87]\n"
        "s_and_b32 s89, s89, 7\n"
        "v_mfma_f32_16x16x32_f16 a[88:91], v[28:31], v[48:51], a[88:91]\n"
        "ds_read_b128 v[24:27], v0 offset:18432\n"
        "buffer_load_dwordx4 v2, s[40:43], s63 offen offset:1024 sc1 lds\n"
        "s_lshl_b32 s89, s89, 4\n"
        "v_mfma_f32_16x16x32_f16 a[92:95], v[28:31], v[52:55], a[92:95]\n"
        "s_lshr_b32 s62, s67, 2\n"
        "s_and_b32 s22, s88, 15\n"
        "s_waitcnt lgkmcnt(5)\n"
        "v_mfma_f32_16x16x32_f16 a[96:99], v[32:35], v[48:51], a[96:99]\n"
        "ds_read_b128 v[28:31], v0 offset:19456\n"
        "s_and_b32 s62, s62, 28\n"
        "s_or_b32 s89, s89, s22\n"
        "v_mfma_f32_16x16x32_f16 a[100:103], v[32:35], v[52:55], a[100:103]\n"
        "s_lshr_b32 s62, s79, s62\n"
        "s_lshl_b32 s89, s89, 15\n"
        "v_mfma_f32_16x16x32_f16 a[104:107], v[36:39], v[48:51], a[104:107]\n"
        "ds_read_b128 v[32:35], v0 offset:20480\n"
        "buffer_load_dwordx4 v2, s[40:43], s63 offen offset:2048 sc1 lds\n"
        "s_and_b32 s62, s62, 7\n"
        "s_add_u32 s89, s89, s78\n"
        "v_mfma_f32_16x16x32_f16 a[108:111], v[36:39], v[52:55], a[108:111]\n"
        "s_lshl_b32 s62, s62, 4\n"
        "s_waitcnt lgkmcnt(5)\n"
        "v_mfma_f32_16x16x32_f16 a[112:115], v[40:43], v[48:51], a[112:115]\n"
        "ds_read_b128 v[36:39], v0 offset:21504\n"
        "s_and_b32 s23, s67, 15\n"
        "buffer_load_dword v152, v15, s[96:99], s89 offen\n"
        "s_add_u32 s88, s88, 2\n"
        "v_mfma_f32_16x16x32_f16 a[116:119], v[40:43], v[52:55], a[116:119]\n"
        "s_or_b32 s62, s62, s23\n"
        "v_mfma_f32_16x16x32_f16 a[120:123], v[44:47], v[48:51], a[120:123]\n"
        "ds_read_b128 v[40:43], v0 offset:22528\n"
        "buffer_load_dwordx4 v2, s[40:43], s63 offen offset:3072 sc1 lds\n"
        "s_add_u32 s64, s64, 32768\n"
        "s_cmp_ge_u32 s64, s92\n"
        "s_cselect_b32 s64, s93, s64\n"
        "s_lshl_b32 s62, s62, 22\n"
        "v_mfma_f32_16x16x32_f16 a[124:127], v[44:47], v[52:55], a[124:127]\n"
        "s_add_u32 s67, s67, 1\n"
        "s_waitcnt lgkmcnt(5)\n"
        "v_mfma_f32_16x16x32_f16 a[128:131], v[16:19], v[48:51], a[128:131]\n"
        "ds_read_b128 v[44:47], v0 offset:23552\n"
        "buffer_load_dwordx2 v[64:65], v5, s[36:39], s62 offen nt\n"
        "v_mfma_f32_16x16x32_f16 a[132:135], v[16:19], v[52:55], a[132:135]\n"
        "v_mfma_f32_16x16x32_f16 a[136:139], v[20:23], v[48:51], a[136:139]\n"
        "ds_read_b128 v[16:19], v0 offset:24576\n"
        "buffer_load_dwordx2 v[66:67], v6, s[36:39], s62 offen nt\n"
        "v_mfma_f32_16x16x32_f16 a[140:143], v[20:23], v[52:55], a[140:143]\n"
        "s_waitcnt lgkmcnt(5)\n"
        "v_mfma_f32_16x16x32_f16 a[144:147], v[24:27], v[48:51], a[144:147]\n"
        "ds_read_b128 v[20:23], v0 offset:25600\n"
        "buffer_load_dwordx2 v[68:69], v7, s[36:39], s62 offen nt\n"
        "v_mfma_f32_16x16x32_f16 a[148:151], v[24:27], v[52:55], a[148:151]\n"
        "v_mfma_f32_16x16x32_f16 a[152:155], v[28:31], v[48:51], a[152:155]\n"
        "ds_read_b128 v[24:27], v0 offset:26624\n"
        "buffer_load_dwordx2 v[70:71], v8, s[36:39], s62 offen nt\n"
        "v_mfma_f32_16x16x32_f16 a[156:159], v[28:31], v[52:55], a[156:159]\n"
        "s_waitcnt lgkmcnt(5)\n"
        "v_mfma_f32_16x16x32_f16 a[160:163], v[32:35], v[48:51], a[160:163]\n"
        "ds_read_b128 v[28:31], v0 offset:27648\n"
        "buffer_load_dwordx2 v[72:73], v9, s[36:39], s62 offen nt\n"
        "v_mfma_f32_16x16x32_f16 a[164:167], v[32:35], v[52:55], a[164:167]\n"
        "v_mfma_f32_16x16x32_f16 a[168:171], v[36:39], v[48:51], a[168:171]\n"
        "ds_read_b128 v[32:35], v0 offset:28672\n"
        "buffer_load_dwordx2 v[74:75], v10, s[36:39], s62 offen nt\n"
        "v_mfma_f32_16x16x32_f16 a[172:175], v[36:39], v[52:55], a[172:175]\n"
        "s_waitcnt lgkmcnt(5)\n"
        "v_mfma_f32_16x16x32_f16 a[176:179], v[40:43], v[48:51], a[176:179]\n"
        "ds_read_b128 v[36:39], v0 offset:29696\n"
        "buffer_load_dwordx2 v[76:77], v11, s[36:39], s62 offen nt\n"
        "v_mfma_f32_16x16x32_f16 a[180:183], v[40:43], v[52:55], a[180:183]\n"
        "v_mfma_f32_16x16x32_f16 a[184:187], v[44:47], v[48:51], a[184:187]\n"
        "ds_read_b128 v[40:43], v0 offset:30720\n"
        "buffer_load_dwordx2 v[78:79], v12, s[36:39], s62 offen nt\n"
        "v_mfma_f32_16x16x32_f16 a[188:191], v[44:47], v[52:55], a[188:191]\n"
        "s_waitcnt lgkmcnt(5)\n"
        "v_mfma_f32_16x16x32_f16 a[192:195], v[16:19], v[48:51], a[192:195]\n"
        "ds_read_b128 v[44:47], v0 offset:31744\n"
        "s_lshr_b32 s62, s67, 2\n"
        "s_and_b32 s62, s62, 28\n"
        "s_lshr_b32 s62, s79, s62\n"
        "s_and_b32 s62, s62, 7\n"
        "s_lshl_b32 s62, s62, 4\n"
        "s_and_b32 s23, s67, 15\n"
        "s_or_b32 s62, s62, s23\n"
        "s_lshl_b32 s62, s62, 22\n"
        "s_add_u32 s67, s67, 1\n"
        "buffer_load_dwordx2 v[80:81], v5, s[36:39], s62 offen nt\n"
        "v_mfma_f32_16x16x32_f16 a[196:199], v[16:19], v[52:55], a[196:199]\n"
        "v_mfma_f32_16x16x32_f16 a[200:203], v[20:23], v[48:51], a[200:203]\n"
        "s_waitcnt vmcnt(19)\n"
        "s_barrier\n"
        "v_add_u32 v0, s65, v1\n"
        "s_add_u32 s65, s65, 32768\n"
        "s_cmp_ge_u32 s65, s94\n"
        "s_cselect_b32 s65, s87, s65\n"
        "ds_read_b128 v[16:19], v0 offset:0\n"
        "buffer_load_dwordx2 v[82:83], v6, s[36:39], s62 offen nt\n"
        "v_mfma_f32_16x16x32_f16 a[204:207], v[20:23], v[52:55], a[204:207]\n"
        "s_waitcnt lgkmcnt(5)\n"
        "v_mfma_f32_16x16x32_f16 a[208:211], v[24:27], v[48:51], a[208:211]\n"
        "ds_read_b128 v[20:23], v0 offset:1024\n"
        "buffer_load_dwordx2 v[84:85], v7, s[36:39], s62 offen nt\n"
        "v_mfma_f32_16x16x32_f16 a[212:215], v[24:27], v[52:55], a[212:215]\n"
        "v_mfma_f32_16x16x32_f16 a[216:219], v[28:31], v[48:51], a[216:219]\n"
        "ds_read_b128 v[24:27], v0 offset:2048\n"
        "buffer_load_dwordx2 v[86:87], v8, s[36:39], s62 offen nt\n"
        "v_mfma_f32_16x16x32_f16 a[220:223], v[28:31], v[52:55], a[220:223]\n"
        "s_waitcnt lgkmcnt(5)\n"
        "v_mfma_f32_16x16x32_f16 a[224:227], v[32:35], v[48:51], a[224:227]\n"
        "ds_read_b128 v[28:31], v0 offset:3072\n"
        "buffer_load_dwordx2 v[88:89], v9, s[36:39], s62 offen nt\n"
        "v_mfma_f32_16x16x32_f16 a[228:231], v[32:35], v[52:55], a[228:231]\n"
        "v_mfma_f32_16x16x32_f16 a[232:235], v[36:39], v[48:51], a[232:235]\n"
        "ds_read_b128 v[32:35], v0 offset:4096\n"
        "buffer_load_dwordx2 v[90:91], v10, s[36:39], s62 offen nt\n"
        "v_mfma_f32_16x16x32_f16 a[236:239], v[36:39], v[52:55], a[236:239]\n"
        "s_waitcnt lgkmcnt(5)\n"
        "v_mfma_f32_16x16x32_f16 a[240:243], v[40:43], v[48:51], a[240:243]\n"
        "ds_read_b128 v[36:39], v0 offset:5120\n"
        "buffer_load_dwordx2 v[92:93], v11, s[36:39], s62 offen nt\n"
        "v_mfma_f32_16x16x32_f16 a[244:247], v[40:43], v[52:55], a[244:247]\n"
        "v_mfma_f32_16x16x32_f16 a[248:251], v[44:47], v[48:51], a[248:251]\n"
        "ds_read_b128 v[40:43], v0 offset:6144\n"
        "buffer_load_dwordx2 v[94:95], v12, s[36:39], s62 offen nt\n"
        "v_mfma_f32_16x16x32_f16 a[252:255], v[44:47], v[52:55], a[252:255]\n"
        "s_add_u32 s95, s95, 1\n"
        "s_waitcnt lgkmcnt(5)\n"
        "v_mfma_f32_16x16x32_f16 a[0:3], v[16:19], v[56:59], a[0:3]\n"
        "ds_read_b128 v[44:47], v0 offset:7168\n"
        "v_mfma_f32_16x16x32_f16 a[4:7], v[16:19], v[60:63], a[4:7]\n"
        "s_waitcnt vmcnt(48)\n"
        "v_cvt_pk_f16_f32 v48, v96, v98\n"
        "v_mfma_f32_16x16x32_f16 a[8:11], v[20:23], v[56:59], a[8:11]\n"
        "ds_read_b128 v[16:19], v0 offset:8192\n"
        "v_mfma_f32_16x16x32_f16 a[12:15], v[20:23], v[60:63], a[12:15]\n"
        "v_cvt_pk_f16_f32 v52, v97, v99\n"
        "s_waitcnt lgkmcnt(5)\n"
        "v_mfma_f32_16x16x32_f16 a[16:19], v[24:27], v[56:59], a[16:19]\n"
        "ds_read_b128 v[20:23], v0 offset:9216\n"
        "v_mfma_f32_16x16x32_f16 a[20:23], v[24:27], v[60:63], a[20:23]\n"
        "s_waitcnt vmcnt(46)\n"
        "v_cvt_pk_f16_f32 v49, v100, v102\n"
        "v_mfma_f32_16x16x32_f16 a[24:27], v[28:31], v[56:59], a[24:27]\n"
        "ds_read_b128 v[24:27], v0 offset:10240\n"
        "v_mfma_f32_16x16x32_f16 a[28:31], v[28:31], v[60:63], a[28:31]\n"
        "v_cvt_pk_f16_f32 v53, v101, v103\n"
        "s_waitcnt lgkmcnt(5)\n"
        "v_mfma_f32_16x16x32_f16 a[32:35], v[32:35], v[56:59], a[32:35]\n"
        "ds_read_b128 v[28:31], v0 offset:11264\n"
        "v_mfma_f32_16x16x32_f16 a[36:39], v[32:35], v[60:63], a[36:39]\n"
        "s_waitcnt vmcnt(44)\n"
        "v_cvt_pk_f16_f32 v50, v104, v106\n"
        "v_mfma_f32_16x16x32_f16 a[40:43], v[36:39], v[56:59], a[40:43]\n"
        "ds_read_b128 v[32:35], v0 offset:12288\n"
        "v_mfma_f32_16x16x32_f16 a[44:47], v[36:39], v[60:63], a[44:47]\n"
        "v_cvt_pk_f16_f32 v54, v105, v107\n"
        "s_waitcnt lgkmcnt(5)\n"
        "v_mfma_f32_16x16x32_f16 a[48:51], v[40:43], v[56:59], a[48:51]\n"
        "ds_read_b128 v[36:39], v0 offset:13312\n"
        "s_lshr_b32 s63, s68, 2\n"
        "v_mfma_f32_16x16x32_f16 a[52:55], v[40:43], v[60:63], a[52:55]\n"
        "s_waitcnt vmcnt(42)\n"
        "v_cvt_pk_f16_f32 v51, v108, v110\n"
        "s_and_b32 s63, s63, 28\n"
        "v_mfma_f32_16x16x32_f16 a[56:59], v[44:47], v[56:59], a[56:59]\n"
        "ds_read_b128 v[40:43], v0 offset:14336\n"
        "s_lshr_b32 s63, s79, s63\n"
        "v_mfma_f32_16x16x32_f16 a[60:63], v[44:47], v[60:63], a[60:63]\n"
        "v_cvt_pk_f16_f32 v55, v109, v111\n"
        "s_and_b32 s63, s63, 7\n"
        "s_waitcnt lgkmcnt(5)\n"
        "v_mfma_f32_16x16x32_f16 a[64:67], v[16:19], v[56:59], a[64:67]\n"
        "ds_read_b128 v[44:47], v0 offset:15360\n"
        "s_lshl_b32 s63, s63, 4\n"
        "v_mfma_f32_16x16x32_f16 a[68:71], v[16:19], v[60:63], a[68:71]\n"
        "s_and_b32 s22, s68, 15\n"
        "v_mfma_f32_16x16x32_f16 a[72:75], v[20:23], v[56:59], a[72:75]\n"
        "ds_read_b128 v[16:19], v0 offset:16384\n"
        "s_or_b32 s63, s63, s22\n"
        "v_mfma_f32_16x16x32_f16 a[76:79], v[20:23], v[60:63], a[76:79]\n"
        "s_lshl_b32 s63, s63, 15\n"
        "s_waitcnt lgkmcnt(5)\n"
        "v_mfma_f32_16x16x32_f16 a[80:83], v[24:27], v[56:59], a[80:83]\n"
        "ds_read_b128 v[20:23], v0 offset:17408\n"
        "s_add_u32 s63, s63, s78\n"
        "v_mfma_f32_16x16x32_f16 a[84:87], v[24:27], v[60:63], a[84:87]\n"
        "s_add_u32 s68, s68, 1\n"
        "v_mfma_f32_16x16x32_f16 a[88:91], v[28:31], v[56:59], a[88:91]\n"
        "ds_read_b128 v[24:27], v0 offset:18432\n"
        "s_mov_b32 m0, s64\n"
        "v_mfma_f32_16x16x32_f16 a[92:95], v[28:31], v[60:63], a[92:95]\n"
        "s_waitcnt lgkmcnt(5)\n"
        "v_mfma_f32_16x16x32_f16 a[96:99], v[32:35], v[56:59], a[96:99]\n"
        "ds_read_b128 v[28:31], v0 offset:19456\n"
        "buffer_load_dwordx4 v2, s[40:43], s63 offen offset:0 sc1 lds\n"
        "v_mfma_f32_16x16x32_f16 a[100:103], v[32:35], v[60:63], a[100:103]\n"
        "v_mfma_f32_16x16x32_f16 a[104:107], v[36:39], v[56:59], a[104:107]\n"
        "ds_read_b128 v[32:35], v0 offset:20480\n"
        "v_mfma_f32_16x16x32_f16 a[108:111], v[36:39], v[60:63], a[108:111]\n"
        "s_waitcnt lgkmcnt(5)\n"
        "v_mfma_f32_16x16x32_f16 a[112:115], v[40:43], v[56:59], a[112:115]\n"
        "ds_read_b128 v[36:39], v0 offset:21504\n"
        "buffer_load_dwordx4 v2, s[40:43], s63 offen offset:1024 sc1 lds\n"
        "v_mfma_f32_16x16x32_f16 a[116:119], v[40:43], v[60:63], a[116:119]\n"
        "v_mfma_f32_16x16x32_f16 a[120:123], v[44:47], v[56:59], a[120:123]\n"
        "ds_read_b128 v[40:43], v0 offset:22528\n"
        "v_mfma_f32_16x16x32_f16 a[124:127], v[44:47], v[60:63], a[124:127]\n"
        "s_waitcnt lgkmcnt(5)\n"
        "v_mfma_f32_16x16x32_f16 a[128:131], v[16:19], v[56:59], a[128:131]\n"
        "ds_read_b128 v[44:47], v0 offset:23552\n"
        "buffer_load_dwordx4 v2, s[40:43], s63 offen offset:2048 sc1 lds\n"
        "v_mfma_f32_16x16x32_f16 a[132:135], v[16:19], v[60:63], a[132:135]\n"
        "v_mfma_f32_16x16x32_f16 a[136:139], v[20:23], v[56:59], a[136:139]\n"
        "ds_read_b128 v[16:19], v0 offset:24576\n"
        "v_mfma_f32_16x16x32_f16 a[140:143], v[20:23], v[60:63], a[140:143]\n"
        "s_waitcnt lgkmcnt(5)\n"
        "v_mfma_f32_16x16x32_f16 a[144:147], v[24:27], v[56:59], a[144:147]\n"
        "ds_read_b128 v[20:23], v0 offset:25600\n"
        "buffer_load_dwordx4 v2, s[40:43], s63 offen offset:3072 sc1 lds\n"
        "v_mfma_f32_16x16x32_f16 a[148:151], v[24:27], v[60:63], a[148:151]\n"
        "v_mfma_f32_16x16x32_f16 a[152:155], v[28:31], v[56:59], a[152:155]\n"
        "ds_read_b128 v[24:27], v0 offset:26624\n"
        "v_mfma_f32_16x16x32_f16 a[156:159], v[28:31], v[60:63], a[156:159]\n"
        "s_waitcnt lgkmcnt(5)\n"
        "v_mfma_f32_16x16x32_f16 a[160:163], v[32:35], v[56:59], a[160:163]\n"
        "ds_read_b128 v[28:31], v0 offset:27648\n"
        "s_add_u32 m0, s64, 4096\n"
        "s_add_u32 s63, s63, 4096\n"
        "s_nop 0\n"
        "buffer_load_dwordx4 v2, s[40:43], s63 offen offset:0 sc1 lds\n"
        "v_mfma_f32_16x16x32_f16 a[164:167], v[32:35], v[60:63], a[164:167]\n"
        "v_mfma_f32_16x16x32_f16 a[168:171], v[36:39], v[56:59], a[168:171]\n"
        "ds_read_b128 v[32:35], v0 offset:28672\n"
        "v_mfma_f32_16x16x32_f16 a[172:175], v[36:39], v[60:63], a[172:175]\n"
        "s_waitcnt lgkmcnt(5)\n"
        "v_mfma_f32_16x16x32_f16 a[176:179], v[40:43], v[56:59], a[176:179]\n"
        "ds_read_b128 v[36:39], v0 offset:29696\n"
        "buffer_load_dwordx4 v2, s[40:43], s63 offen offset:1024 sc1 lds\n"
        "v_mfma_f32_16x16x32_f16 a[180:183], v[40:43], v[60:63], a[180:183]\n"
        "v_mfma_f32_16x16x32_f16 a[184:187], v[44:47], v[56:59], a[184:187]\n"
        "ds_read_b128 v[40:43], v0 offset:30720\n"
        "v_mfma_f32_16x16x32_f16 a[188:191], v[44:47], v[60:63], a[188:191]\n"
        "s_waitcnt lgkmcnt(5)\n"
        "v_mfma_f32_16x16x32_f16 a[192:195], v[16:19], v[56:59], a[192:195]\n"
        "ds_read_b128 v[44:47], v0 offset:31744\n"
        "buffer_load_dwordx4 v2, s[40:43], s63 offen offset:2048 sc1 lds\n"
        "v_mfma_f32_16x16x32_f16 a[196:199], v[16:19], v[60:63], a[196:199]\n"
        "v_mfma_f32_16x16x32_f16 a[200:203], v[20:23], v[56:59], a[200:203]\n"
        "s_waitcnt vmcnt(23)\n"
        "s_barrier\n"
        "v_add_u32 v0, s65, v1\n"
        "s_add_u32 s65, s65, 32768\n"
        "s_cmp_ge_u32 s65, s94\n"
        "s_cselect_b32 s65, s87, s65\n"
        "ds_read_b128 v[16:19], v0 offset:0\n"
        "v_mfma_f32_16x16x32_f16 a[204:207], v[20:23], v[60:63], a[204:207]\n"
        "s_waitcnt lgkmcnt(5)\n"
        "v_mfma_f32_16x16x32_f16 a[208:211], v[24:27], v[56:59], a[208:211]\n"
        "ds_read_b128 v[20:23], v0 offset:1024\n"
        "buffer_load_dwordx4 v2, s[40:43], s63 offen offset:3072 sc1 lds\n"
        "s_add_u32 s64, s64, 32768\n"
        "s_cmp_ge_u32 s64, s92\n"
        "s_cselect_b32 s64, s93, s64\n"
        "v_mfma_f32_16x16x32_f16 a[212:215], v[24:27], v[60:63], a[212:215]\n"
        "s_lshr_b32 s63, s68, 2\n"
        "s_and_b32 s63, s63, 28\n"
        "v_mfma_f32_16x16x32_f16 a[216:219], v[28:31], v[56:59], a[216:219]\n"
        "ds_read_b128 v[24:27], v0 offset:2048\n"
        "s_lshr_b32 s63, s79, s63\n"
        "s_and_b32 s63, s63, 7\n"
        "v_mfma_f32_16x16x32_f16 a[220:223], v[28:31], v[60:63], a[220:223]\n"
        "s_lshl_b32 s63, s63, 4\n"
        "s_and_b32 s22, s68, 15\n"
        "s_waitcnt lgkmcnt(5)\n"
        "v_mfma_f32_16x16x32_f16 a[224:227], v[32:35], v[56:59], a[224:227]\n"
        "ds_read_b128 v[28:31], v0 offset:3072\n"
        "s_or_b32 s63, s63, s22\n"
        "s_lshl_b32 s63, s63, 15\n"
        "v_mfma_f32_16x16x32_f16 a[228:231], v[32:35], v[60:63], a[228:231]\n"
        "s_add_u32 s63, s63, s78\n"
        "s_add_u32 s68, s68, 1\n"
        "v_mfma_f32_16x16x32_f16 a[232:235], v[36:39], v[56:59], a[232:235]\n"
        "ds_read_b128 v[32:35], v0 offset:4096\n"
        "s_mov_b32 m0, s64\n"
        "v_mfma_f32_16x16x32_f16 a[236:239], v[36:39], v[60:63], a[236:239]\n"
        "s_waitcnt lgkmcnt(5)\n"
        "v_mfma_f32_16x16x32_f16 a[240:243], v[40:43], v[56:59], a[240:243]\n"
        "ds_read_b128 v[36:39], v0 offset:5120\n"
        "v_mfma_f32_16x16x32_f16 a[244:247], v[40:43], v[60:63], a[244:247]\n"
        "v_mfma_f32_16x16x32_f16 a[248:251], v[44:47], v[56:59], a[248:251]\n"
        "ds_read_b128 v[40:43], v0 offset:6144\n"
        "s_and_b32 s70, s95, 15\n"
        "s_cmp_lg_u32 s70, 9\n"
        "s_cbranch_scc1 L_d1u26_%=\n"
        "buffer_load_dwordx4 v[144:147], v1, s[56:59], 0 offen sc0 sc1\n"
        "buffer_load_dwordx4 v[148:151], v1, s[56:59], 0 offen offset:1024 sc0 sc1\n"
        "L_d1u26_%=:\n"
        "v_mfma_f32_16x16x32_f16 a[252:255], v[44:47], v[60:63], a[252:255]\n"
        "s_add_u32 s95, s95, 1\n"
        "s_cmp_ge_u32 s61, 31\n"
        "s_cselect_b32 s42, 0, s42\n"
        "s_waitcnt lgkmcnt(5)\n"
        "v_mfma_f32_16x16x32_f16 a[0:3], v[16:19], v[48:51], a[0:3]\n"
        "s_waitcnt vmcnt(32)\n"
        "v_cmp_le_u32 vcc, s60, v4\n"
        "s_nop 1\n"
        "s_cmp_eq_u64 vcc, -1\n"
        "s_cbranch_scc1 L_oku27_%=\n"
        "s_memrealtime s[74:75]\n"
        "L_spinu27_%=:\n"
        "buffer_load_dword v4, v3, s[56:59], 0 offen sc0 sc1\n"
        "v_mov_b32 v14, 0\n"
        "s_mov_b32 s70, 16384\n"
        "buffer_load_dword v13, v14, s[56:59], s70 offen sc1\n"
        "s_waitcnt vmcnt(0) lgkmcnt(0)\n"
        "v_cmp_le_u32 vcc, s60, v4\n"
        "s_nop 1\n"
        "s_cmp_eq_u64 vcc, -1\n"
        "s_cbranch_scc1 L_oku27_%=\n"
        "v_readfirstlane_b32 s70, v13\n"
        "s_nop 3\n"
        "s_cmp_eq_u32 s70, s60\n"
        "s_cbranch_scc1 L_badu27_%=\n"
        "s_memrealtime s[76:77]\n"
        "s_waitcnt lgkmcnt(0)\n"
        "s_sub_u32 s70, s76, s74\n"
        "s_cmp_gt_u32 s70, 50000000\n"
        "s_cbranch_scc1 L_badu27_%=\n"
        "s_sleep 2\n"
        "s_branch L_spinu27_%=\n"
        "L_badu27_%=:\n"
        "v_mov_b32 v13, s60\n"
        "v_mov_b32 v14, 0\n"
        "s_mov_b32 s70, 16384\n"
        "buffer_store_dword v13, v14, s[56:59], s70 offen sc1\n"
        "s_waitcnt vmcnt(0)\n"
        "s_mov_b32 s69, 1\n"
        "s_mov_b64 vcc, -1\n"
        "L_oku27_%=:\n"
        "ds_read_b128 v[44:47], v0 offset:7168\n"
        "s_add_u32 s30, s68, 2\n"
        "s_lshr_b32 s66, s30, 2\n"
        "v_mfma_f32_16x16x32_f16 a[4:7], v[16:19], v[52:55], a[4:7]\n"
        "s_waitcnt vmcnt(48)\n"
        "v_cvt_pk_f16_f32 v56, v112, v114\n"
        "s_and_b32 s66, s66, 28\n"
        "s_lshr_b32 s66, s79, s66\n"
        "v_mfma_f32_16x16x32_f16 a[8:11], v[20:23], v[48:51], a[8:11]\n"
        "ds_read_b128 v[16:19], v0 offset:8192\n"
        "buffer_load_dwordx4 v2, s[40:43], s63 offen offset:0 sc1 lds\n"
        "s_and_b32 s66, s66, 7\n"
        "s_lshl_b32 s66, s66, 4\n"
        "v_mfma_f32_16x16x32_f16 a[12:15], v[20:23], v[52:55], a[12:15]\n"
        "v_cvt_pk_f16_f32 v60, v113, v115\n"
        "s_and_b32 s30, s30, 15\n"
        "s_or_b32 s66, s66, s30\n"
        "s_waitcnt lgkmcnt(5)\n"
        "v_mfma_f32_16x16x32_f16 a[16:19], v[24:27], v[48:51], a[16:19]\n"
        "ds_read_b128 v[20:23], v0 offset:9216\n"
        "s_and_b32 s66, s66, 120\n"
        "s_lshl_b32 s66, s66, 4\n"
        "v_mfma_f32_16x16x32_f16 a[20:23], v[24:27], v[52:55], a[20:23]\n"
        "s_waitcnt vmcnt(47)\n"
        "v_cvt_pk_f16_f32 v57, v116, v118\n"
        "v_add_u32 v3, s66, v153\n"
        "buffer_load_dword v4, v3, s[56:59], 0 offen sc0 sc1\n"
        "v_mfma_f32_16x16x32_f16 a[24:27], v[28:31], v[48:51], a[24:27]\n"
        "ds_read_b128 v[24:27], v0 offset:10240\n"
        "buffer_load_dwordx4 v2, s[40:43], s63 offen offset:1024 sc1 lds\n"
        "v_mfma_f32_16x16x32_f16 a[28:31], v[28:31], v[52:55], a[28:31]\n"
        "v_cvt_pk_f16_f32 v61, v117, v119\n"
        "s_waitcnt lgkmcnt(5)\n"
        "v_mfma_f32_16x16x32_f16 a[32:35], v[32:35], v[48:51], a[32:35]\n"
        "ds_read_b128 v[28:31], v0 offset:11264\n"
        "v_mfma_f32_16x16x32_f16 a[36:39], v[32:35], v[52:55], a[36:39]\n"
        "s_waitcnt vmcnt(47)\n"
        "v_cvt_pk_f16_f32 v58, v120, v122\n"
        "v_mfma_f32_16x16x32_f16 a[40:43], v[36:39], v[48:51], a[40:43]\n"
        "ds_read_b128 v[32:35], v0 offset:12288\n"
        "buffer_load_dwordx4 v2, s[40:43], s63 offen offset:2048 sc1 lds\n"
        "v_mfma_f32_16x16x32_f16 a[44:47], v[36:39], v[52:55], a[44:47]\n"
        "v_cvt_pk_f16_f32 v62, v121, v123\n"
        "s_waitcnt lgkmcnt(5)\n"
        "v_mfma_f32_16x16x32_f16 a[48:51], v[40:43], v[48:51], a[48:51]\n"
        "ds_read_b128 v[36:39], v0 offset:13312\n"
        "v_mfma_f32_16x16x32_f16 a[52:55], v[40:43], v[52:55], a[52:55]\n"
        "s_waitcnt vmcnt(46)\n"
        "v_cvt_pk_f16_f32 v59, v124, v126\n"
        "v_mfma_f32_16x16x32_f16 a[56:59], v[44:47], v[48:51], a[56:59]\n"
        "ds_read_b128 v[40:43], v0 offset:14336\n"
        "buffer_load_dwordx4 v2, s[40:43], s63 offen offset:3072 sc1 lds\n"
        "v_mfma_f32_16x16x32_f16 a[60:63], v[44:47], v[52:55], a[60:63]\n"
        "v_cvt_pk_f16_f32 v63, v125, v127\n"
        "s_waitcnt lgkmcnt(5)\n"
        "v_mfma_f32_16x16x32_f16 a[64:67], v[16:19], v[48:51], a[64:67]\n"
        "ds_read_b128 v[44:47], v0 offset:15360\n"
        "s_and_b32 s22, s88, 7\n"
        "v_mfma_f32_16x16x32_f16 a[68:71], v[16:19], v[52:55], a[68:71]\n"
        "s_cmp_eq_u32 s22, 0\n"
        "s_cselect_b32 s98, 0, s35\n"
        "v_mfma_f32_16x16x32_f16 a[72:75], v[20:23], v[48:51], a[72:75]\n"
        "ds_read_b128 v[16:19], v0 offset:16384\n"
        "s_add_u32 m0, s64, 4096\n"
        "s_add_u32 s63, s63, 4096\n"
        "s_nop 0\n"
        "buffer_load_dwordx4 v2, s[40:43], s63 offen offset:0 sc1 lds\n"
        "s_lshr_b32 s89, s88, 2\n"
        "v_mfma_f32_16x16x32_f16 a[76:79], v[20:23], v[52:55], a[76:79]\n"
        "s_and_b32 s89, s89, 28\n"
        "s_waitcnt lgkmcnt(5)\n"
        "v_mfma_f32_16x16x32_f16 a[80:83], v[24:27], v[48:51], a[80:83]\n"
        "ds_read_b128 v[20:23], v0 offset:17408\n"
        "s_lshr_b32 s89, s79, s89\n"
        "v_mfma_f32_16x16x32_f16 a[84:87], v[24:27], v[52:55], a[84:87]\n"
        "s_and_b32 s89, s89, 7\n"
        "v_mfma_f32_16x16x32_f16 a[88:91], v[28:31], v[48:51], a[88:91]\n"
        "ds_read_b128 v[24:27], v0 offset:18432\n"
        "buffer_load_dwordx4 v2, s[40:43], s63 offen offset:1024 sc1 lds\n"
        "s_lshl_b32 s89, s89, 4\n"
        "v_mfma_f32_16x16x32_f16 a[92:95], v[28:31], v[52:55], a[92:95]\n"
        "s_lshr_b32 s62, s67, 2\n"
        "s_and_b32 s22, s88, 15\n"
        "s_waitcnt lgkmcnt(5)\n"
        "v_mfma_f32_16x16x32_f16 a[96:99], v[32:35], v[48:51], a[96:99]\n"
        "ds_read_b128 v[28:31], v0 offset:19456\n"
        "s_and_b32 s62, s62, 28\n"
        "s_or_b32 s89, s89, s22\n"
        "v_mfma_f32_16x16x32_f16 a[100:103], v[32:35], v[52:55], a[100:103]\n"
        "s_lshr_b32 s62, s79, s62\n"
        "s_lshl_b32 s89, s89, 15\n"
        "v_mfma_f32_16x16x32_f16 a[104:107], v[36:39], v[48:51], a[104:107]\n"
        "ds_read_b128 v[32:35], v0 offset:20480\n"
        "buffer_load_dwordx4 v2, s[40:43], s63 offen offset:2048 sc1 lds\n"
        "s_and_b32 s62, s62, 7\n"
        "s_add_u32 s89, s89, s78\n"
        "v_mfma_f32_16x16x32_f16 a[108:111], v[36:39], v[52:55], a[108:111]\n"
        "s_lshl_b32 s62, s62, 4\n"
        "s_waitcnt lgkmcnt(5)\n"
        "v_mfma_f32_16x16x32_f16 a[112:115], v[40:43], v[48:51], a[112:115]\n"
        "ds_read_b128 v[36:39], v0 offset:21504\n"
        "s_and_b32 s23, s67, 15\n"
        "buffer_load_dword v152, v15, s[96:99], s89 offen\n"
        "s_add_u32 s88, s88, 2\n"
        "v_mfma_f32_16x16x32_f16 a[116:119], v[40:43], v[52:55], a[116:119]\n"
        "s_or_b32 s62, s62, s23\n"
        "v_mfma_f32_16x16x32_f16 a[120:123], v[44:47], v[48:51], a[120:123]\n"
        "ds_read_b128 v[40:43], v0 offset:22528\n"
        "buffer_load_dwordx4 v2, s[40:43], s63 offen offset:3072 sc1 lds\n"
        "s_add_u32 s64, s64, 32768\n"
        "s_cmp_ge_u32 s64, s92\n"
        "s_cselect_b32 s64, s93, s64\n"
        "s_lshl_b32 s62, s62, 22\n"
        "v_mfma_f32_16x16x32_f16 a[124:127], v[44:47], v[52:55], a[124:127]\n"
        "s_add_u32 s67, s67, 1\n"
        "s_waitcnt lgkmcnt(5)\n"
        "v_mfma_f32_16x16x32_f16 a[128:131], v[16:19], v[48:51], a[128:131]\n"
        "ds_read_b128 v[44:47], v0 offset:23552\n"
        "buffer_load_dwordx2 v[96:97], v5, s[36:39], s62 offen nt\n"
        "v_mfma_f32_16x16x32_f16 a[132:135], v[16:19], v[52:55], a[132:135]\n"
        "v_mfma_f32_16x16x32_f16 a[136:139], v[20:23], v[48:51], a[136:139]\n"
        "ds_read_b128 v[16:19], v0 offset:24576\n"
        "buffer_load_dwordx2 v[98:99], v6, s[36:39], s62 offen nt\n"
        "v_mfma_f32_16x16x32_f16 a[140:143], v[20:23], v[52:55], a[140:143]\n"
        "s_waitcnt lgkmcnt(5)\n"
        "v_mfma_f32_16x16x32_f16 a[144:147], v[24:27], v[48:51], a[144:147]\n"
        "ds_read_b128 v[20:23], v0 offset:25600\n"
        "buffer_load_dwordx2 v[100:101], v7, s[36:39], s62 offen nt\n"
        "v_mfma_f32_16x16x32_f16 a[148:151], v[24:27], v[52:55], a[148:151]\n"
        "v_mfma_f32_16x16x32_f16 a[152:155], v[28:31], v[48:51], a[152:155]\n"
        "ds_read_b128 v[24:27], v0 offset:26624\n"
        "buffer_load_dwordx2 v[102:103], v8, s[36:39], s62 offen nt\n"
        "v_mfma_f32_16x16x32_f16 a[156:159], v[28:31], v[52:55], a[156:159]\n"
        "s_waitcnt lgkmcnt(5)\n"
        "v_mfma_f32_16x16x32_f16 a[160:163], v[32:35], v[48:51], a[160:163]\n"
        "ds_read_b128 v[28:31], v0 offset:27648\n"
        "buffer_load_dwordx2 v[104:105], v9, s[36:39], s62 offen nt\n"
        "v_mfma_f32_16x16x32_f16 a[164:167], v[32:35], v[52:55], a[164:167]\n"
        "v_mfma_f32_16x16x32_f16 a[168:171], v[36:39], v[48:51], a[168:171]\n"
        "ds_read_b128 v[32:35], v0 offset:28672\n"
        "buffer_load_dwordx2 v[106:107], v10, s[36:39], s62 offen nt\n"
        "v_mfma_f32_16x16x32_f16 a[172:175], v[36:39], v[52:55], a[172:175]\n"
        "s_waitcnt lgkmcnt(5)\n"
        "v_mfma_f32_16x16x32_f16 a[176:179], v[40:43], v[48:51], a[176:179]\n"
        "ds_read_b128 v[36:39], v0 offset:29696\n"
        "buffer_load_dwordx2 v[108:109], v11, s[36:39], s62 offen nt\n"
        "v_mfma_f32_16x16x32_f16 a[180:183], v[40:43], v[52:55], a[180:183]\n"
        "v_mfma_f32_16x16x32_f16 a[184:187], v[44:47], v[48:51], a[184:187]\n"
        "ds_read_b128 v[40:43], v0 offset:30720\n"
        "buffer_load_dwordx2 v[110:111], v12, s[36:39], s62 offen nt\n"
        "v_mfma_f32_16x16x32_f16 a[188:191], v[44:47], v[52:55], a[188:191]\n"
        "s_waitcnt lgkmcnt(5)\n"
        "v_mfma_f32_16x16x32_f16 a[192:195], v[16:19], v[48:51], a[192:195]\n"
        "ds_read_b128 v[44:47], v0 offset:31744\n"
        "s_lshr_b32 s62, s67, 2\n"
        "s_and_b32 s62, s62, 28\n"
        "s_lshr_b32 s62, s79, s62\n"
        "s_and_b32 s62, s62, 7\n"
        "s_lshl_b32 s62, s62, 4\n"
        "s_and_b32 s23, s67, 15\n"
        "s_or_b32 s62, s62, s23\n"
        "s_lshl_b32 s62, s62, 22\n"
        "s_add_u32 s67, s67, 1\n"
        "buffer_load_dwordx2 v[112:113], v5, s[36:39], s62 offen nt\n"
        "v_mfma_f32_16x16x32_f16 a[196:199], v[16:19], v[52:55], a[196:199]\n"
        "v_mfma_f32_16x16x32_f16 a[200:203], v[20:23], v[48:51], a[200:203]\n"
        "s_waitcnt vmcnt(19)\n"
        "s_barrier\n"
        "v_add_u32 v0, s65, v1\n"
        "s_add_u32 s65, s65, 32768\n"
        "s_cmp_ge_u32 s65, s94\n"
        "s_cselect_b32 s65, s87, s65\n"
        "ds_read_b128 v[16:19], v0 offset:0\n"
        "buffer_load_dwordx2 v[114:115], v6, s[36:39], s62 offen nt\n"
        "v_mfma_f32_16x16x32_f16 a[204:207], v[20:23], v[52:55], a[204:207]\n"
        "s_waitcnt lgkmcnt(5)\n"
        "v_mfma_f32_16x16x32_f16 a[208:211], v[24:27], v[48:51], a[208:211]\n"
        "ds_read_b128 v[20:23], v0 offset:1024\n"
        "buffer_load_dwordx2 v[116:117], v7, s[36:39], s62 offen nt\n"
        "v_mfma_f32_16x16x32_f16 a[212:215], v[24:27], v[52:55], a[212:215]\n"
        "v_mfma_f32_16x16x32_f16 a[216:219], v[28:31], v[48:51], a[216:219]\n"
        "ds_read_b128 v[24:27], v0 offset:2048\n"
        "buffer_load_dwordx2 v[118:119], v8, s[36:39], s62 offen nt\n"
        "v_mfma_f32_16x16x32_f16 a[220:223], v[28:31], v[52:55], a[220:223]\n"
        "s_waitcnt lgkmcnt(5)\n"
        "v_mfma_f32_16x16x32_f16 a[224:227], v[32:35], v[48:51], a[224:227]\n"
        "ds_read_b128 v[28:31], v0 offset:3072\n"
        "buffer_load_dwordx2 v[120:121], v9, s[36:39], s62 offen nt\n"
        "v_mfma_f32_16x16x32_f16 a[228:231], v[32:35], v[52:55], a[228:231]\n"
        "v_mfma_f32_16x16x32_f16 a[232:235], v[36:39], v[48:51], a[232:235]\n"
        "ds_read_b128 v[32:35], v0 offset:4096\n"
        "buffer_load_dwordx2 v[122:123], v10, s[36:39], s62 offen nt\n"
        "v_mfma_f32_16x16x32_f16 a[236:239], v[36:39], v[52:55], a[236:239]\n"
        "s_waitcnt lgkmcnt(5)\n"
        "v_mfma_f32_16x16x32_f16 a[240:243], v[40:43], v[48:51], a[240:243]\n"
        "ds_read_b128 v[36:39], v0 offset:5120\n"
        "buffer_load_dwordx2 v[124:125], v11, s[36:39], s62 offen nt\n"
        "v_mfma_f32_16x16x32_f16 a[244:247], v[40:43], v[52:55], a[244:247]\n"
        "v_mfma_f32_16x16x32_f16 a[248:251], v[44:47], v[48:51], a[248:251]\n"
        "ds_read_b128 v[40:43], v0 offset:6144\n"
        "buffer_load_dwordx2 v[126:127], v12, s[36:39], s62 offen nt\n"
        "v_mfma_f32_16x16x32_f16 a[252:255], v[44:47], v[52:55], a[252:255]\n"
        "s_add_u32 s95, s95, 1\n"
        "s_waitcnt lgkmcnt(5)\n"
        "v_mfma_f32_16x16x32_f16 a[0:3], v[16:19], v[56:59], a[0:3]\n"
        "ds_read_b128 v[44:47], v0 offset:7168\n"
        "v_mfma_f32_16x16x32_f16 a[4:7], v[16:19], v[60:63], a[4:7]\n"
        "s_waitcnt vmcnt(48)\n"
        "v_cvt_pk_f16_f32 v48, v64, v66\n"
        "v_mfma_f32_16x16x32_f16 a[8:11], v[20:23], v[56:59], a[8:11]\n"
        "ds_read_b128 v[16:19], v0 offset:8192\n"
        "v_mfma_f32_16x16x32_f16 a[12:15], v[20:23], v[60:63], a[12:15]\n"
        "v_cvt_pk_f16_f32 v52, v65, v67\n"
        "s_waitcnt lgkmcnt(5)\n"
        "v_mfma_f32_16x16x32_f16 a[16:19], v[24:27], v[56:59], a[16:19]\n"
        "ds_read_b128 v[20:23], v0 offset:9216\n"
        "v_mfma_f32_16x16x32_f16 a[20:23], v[24:27], v[60:63], a[20:23]\n"
        "s_waitcnt vmcnt(46)\n"
        "v_cvt_pk_f16_f32 v49, v68, v70\n"
        "v_mfma_f32_16x16x32_f16 a[24:27], v[28:31], v[56:59], a[24:27]\n"
        "ds_read_b128 v[24:27], v0 offset:10240\n"
        "v_mfma_f32_16x16x32_f16 a[28:31], v[28:31], v[60:63], a[28:31]\n"
        "v_cvt_pk_f16_f32 v53, v69, v71\n"
        "s_waitcnt lgkmcnt(5)\n"
        "v_mfma_f32_16x16x32_f16 a[32:35], v[32:35], v[56:59], a[32:35]\n"
        "ds_read_b128 v[28:31], v0 offset:11264\n"
        "v_mfma_f32_16x16x32_f16 a[36:39], v[32:35], v[60:63], a[36:39]\n"
        "s_waitcnt vmcnt(44)\n"
        "v_cvt_pk_f16_f32 v50, v72, v74\n"
        "v_mfma_f32_16x16x32_f16 a[40:43], v[36:39], v[56:59], a[40:43]\n"
        "ds_read_b128 v[32:35], v0 offset:12288\n"
        "v_mfma_f32_16x16x32_f16 a[44:47], v[36:39], v[60:63], a[44:47]\n"
        "v_cvt_pk_f16_f32 v54, v73, v75\n"
        "s_waitcnt lgkmcnt(5)\n"
        "v_mfma_f32_16x16x32_f16 a[48:51], v[40:43], v[56:59], a[48:51]\n"
        "ds_read_b128 v[36:39], v0 offset:13312\n"
        "s_lshr_b32 s63, s68, 2\n"
        "v_mfma_f32_16x16x32_f16 a[52:55], v[40:43], v[60:63], a[52:55]\n"
        "s_waitcnt vmcnt(42)\n"
        "v_cvt_pk_f16_f32 v51, v76, v78\n"
        "s_and_b32 s63, s63, 28\n"
        "v_mfma_f32_16x16x32_f16 a[56:59], v[44:47], v[56:59], a[56:59]\n"
        "ds_read_b128 v[40:43], v0 offset:14336\n"
        "s_lshr_b32 s63, s79, s63\n"
        "v_mfma_f32_16x16x32_f16 a[60:63], v[44:47], v[60:63], a[60:63]\n"
        "v_cvt_pk_f16_f32 v55, v77, v79\n"
        "s_and_b32 s63, s63, 7\n"
        "s_waitcnt lgkmcnt(5)\n"
        "v_mfma_f32_16x16x32_f16 a[64:67], v[16:19], v[56:59], a[64:67]\n"
        "ds_read_b128 v[44:47], v0 offset:15360\n"
        "s_lshl_b32 s63, s63, 4\n"
        "v_mfma_f32_16x16x32_f16 a[68:71], v[16:19], v[60:63], a[68:71]\n"
        "s_and_b32 s22, s68, 15\n"
        "v_mfma_f32_16x16x32_f16 a[72:75], v[20:23], v[56:59], a[72:75]\n"
        "ds_read_b128 v[16:19], v0 offset:16384\n"
        "s_or_b32 s63, s63, s22\n"
        "v_mfma_f32_16x16x32_f16 a[76:79], v[20:23], v[60:63], a[76:79]\n"
        "s_lshl_b32 s63, s63, 15\n"
        "s_waitcnt lgkmcnt(5)\n"
        "v_mfma_f32_16x16x32_f16 a[80:83], v[24:27], v[56:59], a[80:83]\n"
        "ds_read_b128 v[20:23], v0 offset:17408\n"
        "s_add_u32 s63, s63, s78\n"
        "v_mfma_f32_16x16x32_f16 a[84:87], v[24:27], v[60:63], a[84:87]\n"
        "s_add_u32 s68, s68, 1\n"
        "v_mfma_f32_16x16x32_f16 a[88:91], v[28:31], v[56:59], a[88:91]\n"
        "ds_read_b128 v[24:27], v0 offset:18432\n"
        "s_mov_b32 m0, s64\n"
        "v_mfma_f32_16x16x32_f16 a[92:95], v[28:31], v[60:63], a[92:95]\n"
        "s_waitcnt lgkmcnt(5)\n"
        "v_mfma_f32_16x16x32_f16 a[96:99], v[32:35], v[56:59], a[96:99]\n"
        "ds_read_b128 v[28:31], v0 offset:19456\n"
        "buffer_load_dwordx4 v2, s[40:43], s63 offen offset:0 sc1 lds\n"
        "v_mfma_f32_16x16x32_f16 a[100:103], v[32:35], v[60:63], a[100:103]\n"
        "v_mfma_f32_16x16x32_f16 a[104:107], v[36:39], v[56:59], a[104:107]\n"
        "ds_read_b128 v[32:35], v0 offset:20480\n"
        "v_mfma_f32_16x16x32_f16 a[108:111], v[36:39], v[60:63], a[108:111]\n"
        "s_waitcnt lgkmcnt(5)\n"
        "v_mfma_f32_16x16x32_f16 a[112:115], v[40:43], v[56:59], a[112:115]\n"
        "ds_read_b128 v[36:39], v0 offset:21504\n"
        "buffer_load_dwordx4 v2, s[40:43], s63 offen offset:1024 sc1 lds\n"
        "v_mfma_f32_16x16x32_f16 a[116:119], v[40:43], v[60:63], a[116:119]\n"
        "v_mfma_f32_16x16x32_f16 a[120:123], v[44:47], v[56:59], a[120:123]\n"
        "ds_read_b128 v[40:43], v0 offset:22528\n"
        "v_mfma_f32_16x16x32_f16 a[124:127], v[44:47], v[60:63], a[124:127]\n"
        "s_waitcnt lgkmcnt(5)\n"
        "v_mfma_f32_16x16x32_f16 a[128:131], v[16:19], v[56:59], a[128:131]\n"
        "ds_read_b128 v[44:47], v0 offset:23552\n"
        "buffer_load_dwordx4 v2, s[40:43], s63 offen offset:2048 sc1 lds\n"
        "v_mfma_f32_16x16x32_f16 a[132:135], v[16:19], v[60:63], a[132:135]\n"
        "v_mfma_f32_16x16x32_f16 a[136:139], v[20:23], v[56:59], a[136:139]\n"
        "ds_read_b128 v[16:19], v0 offset:24576\n"
        "v_mfma_f32_16x16x32_f16 a[140:143], v[20:23], v[60:63], a[140:143]\n"
        "s_waitcnt lgkmcnt(5)\n"
        "v_mfma_f32_16x16x32_f16 a[144:147], v[24:27], v[56:59], a[144:147]\n"
        "ds_read_b128 v[20:23], v0 offset:25600\n"
        "buffer_load_dwordx4 v2, s[40:43], s63 offen offset:3072 sc1 lds\n"
        "v_mfma_f32_16x16x32_f16 a[148:151], v[24:27], v[60:63], a[148:151]\n"
        "v_mfma_f32_16x16x32_f16 a[152:155], v[28:31], v[56:59], a[152:155]\n"
        "ds_read_b128 v[24:27], v0 offset:26624\n"
        "v_mfma_f32_16x16x32_f16 a[156:159], v[28:31], v[60:63], a[156:159]\n"
        "s_waitcnt lgkmcnt(5)\n"
        "v_mfma_f32_16x16x32_f16 a[160:163], v[32:35], v[56:59], a[160:163]\n"
        "ds_read_b128 v[28:31], v0 offset:27648\n"
        "s_add_u32 m0, s64, 4096\n"
        "s_add_u32 s63, s63, 4096\n"
        "s_nop 0\n"
        "buffer_load_dwordx4 v2, s[40:43], s63 offen offset:0 sc1 lds\n"
        "v_mfma_f32_16x16x32_f16 a[164:167], v[32:35], v[60:63], a[164:167]\n"
        "v_mfma_f32_16x16x32_f16 a[168:171], v[36:39], v[56:59], a[168:171]\n"
        "ds_read_b128 v[32:35], v0 offset:28672\n"
        "v_mfma_f32_16x16x32_f16 a[172:175], v[36:39], v[60:63], a[172:175]\n"
        "s_waitcnt lgkmcnt(5)\n"
        "v_mfma_f32_16x16x32_f16 a[176:179], v[40:43], v[56:59], a[176:179]\n"
        "ds_read_b128 v[36:39], v0 offset:29696\n"
        "buffer_load_dwordx4 v2, s[40:43], s63 offen offset:1024 sc1 lds\n"
        "v_mfma_f32_16x16x32_f16 a[180:183], v[40:43], v[60:63], a[180:183]\n"
        "v_mfma_f32_16x16x32_f16 a[184:187], v[44:47], v[56:59], a[184:187]\n"
        "ds_read_b128 v[40:43], v0 offset:30720\n"
        "v_mfma_f32_16x16x32_f16 a[188:191], v[44:47], v[60:63], a[188:191]\n"
        "s_waitcnt lgkmcnt(5)\n"
        "v_mfma_f32_16x16x32_f16 a[192:195], v[16:19], v[56:59], a[192:195]\n"
        "ds_read_b128 v[44:47], v0 offset:31744\n"
        "buffer_load_dwordx4 v2, s[40:43], s63 offen offset:2048 sc1 lds\n"
        "v_mfma_f32_16x16x32_f16 a[196:199], v[16:19], v[60:63], a[196:199]\n"
        "v_mfma_f32_16x16x32_f16 a[200:203], v[20:23], v[56:59], a[200:203]\n"
        "s_waitcnt vmcnt(23)\n"
        "s_and_b32 s70, s95, 15\n"
        "s_cmp_lg_u32 s70, 11\n"
        "s_cbranch_scc1 L_d2u28_%=\n"
        "s_cmp_lg_u32 s83, 0\n"
        "s_cbranch_scc1 L_d2u28_%=\n"
        "s_memrealtime s[74:75]\n"
        "L_d2ru28_%=:\n"
        "v_min_u32 v144, v144, v145\n"
        "v_min3_u32 v144, v144, v146, v147\n"
        "v_min_u32 v148, v148, v149\n"
        "v_min3_u32 v148, v148, v150, v151\n"
        "v_cmp_le_u32 s[32:33], s60, v144\n"
        "v_cmp_le_u32 s[20:21], s60, v148\n"
        "s_nop 3\n"
        "s_mov_b32 s71, 0\n"
        "s_bfe_u32 s72, s32, 0x80000\n"
        "s_cmp_eq_u32 s72, 0xff\n"
        "s_cselect_b32 s72, 1, 0\n"
        "s_or_b32 s71, s71, s72\n"
        "s_bfe_u32 s72, s32, 0x80010\n"
        "s_cmp_eq_u32 s72, 0xff\n"
        "s_cselect_b32 s72, 2, 0\n"
        "s_or_b32 s71, s71, s72\n"
        "s_bfe_u32 s72, s33, 0x80000\n"
        "s_cmp_eq_u32 s72, 0xff\n"
        "s_cselect_b32 s72, 4, 0\n"
        "s_or_b32 s71, s71, s72\n"
        "s_bfe_u32 s72, s33, 0x80010\n"
        "s_cmp_eq_u32 s72, 0xff\n"
        "s_cselect_b32 s72, 8, 0\n"
        "s_or_b32 s71, s71, s72\n"
        "s_bfe_u32 s72, s20, 0x80000\n"
        "s_cmp_eq_u32 s72, 0xff\n"
        "s_cselect_b32 s72, 16, 0\n"
        "s_or_b32 s71, s71, s72\n"
        "s_bfe_u32 s72, s20, 0x80010\n"
        "s_cmp_eq_u32 s72, 0xff\n"
        "s_cselect_b32 s72, 32, 0\n"
        "s_or_b32 s71, s71, s72\n"
        "s_bfe_u32 s72, s21, 0x80000\n"
        "s_cmp_eq_u32 s72, 0xff\n"
        "s_cselect_b32 s72, 64, 0\n"
        "s_or_b32 s71, s71, s72\n"
        "s_bfe_u32 s72, s21, 0x80010\n"
        "s_cmp_eq_u32 s72, 0xff\n"
        "s_cselect_b32 s72, 128, 0\n"
        "s_or_b32 s71, s71, s72\n"
        "s_andn2_b32 s71, s71, s29\n"
        "s_not_b32 s72, s29\n"
        "s_and_b32 s72, s72, 0xff\n"
        "s_cmp_lg_u32 s71, 0\n"
        "s_cbranch_scc1 L_d2pu28_%=\n"
        "s_cmp_eq_u32 s72, 0\n"
        "s_cbranch_scc1 L_d2pu28_%=\n"
        "s_cmp_lg_u32 s69, 0\n"
        "s_cbranch_scc1 L_d2pu28_%=\n"
        "s_memrealtime s[76:77]\n"
        "s_waitcnt lgkmcnt(0)\n"
        "s_sub_u32 s73, s76, s74\n"
        "s_cmp_gt_u32 s73, 50000000\n"
        "s_cbranch_scc1 L_d2pu28_%=\n"
        "s_sleep 1\n"
        "buffer_load_dwordx4 v[144:147], v1, s[56:59], 0 offen sc0 sc1\n"
        "buffer_load_dwordx4 v[148:151], v1, s[56:59], 0 offen offset:1024 sc0 sc1\n"
        "v_mov_b32 v14, 0\n"
        "s_mov_b32 s73, 16384\n"
        "buffer_load_dword v13, v14, s[56:59], s73 offen sc1\n"
        "s_waitcnt vmcnt(0)\n"
        "v_readfirstlane_b32 s73, v13\n"
        "s_nop 3\n"
        "s_cmp_eq_u32 s73, s60\n"
        "s_cbranch_scc0 L_d2ru28_%=\n"
        "L_d2pu28_%=:\n"
        "s_cmp_lg_u32 s71, 0\n"
        "s_cselect_b32 s71, s71, s72\n"
        "s_lshl_b32 s72, s71, 8\n"
        "s_or_b32 s71, s71, s72\n"
        "s_add_u32 s72, s84, 1\n"
        "s_lshr_b32 s71, s71, s72\n"
        "s_ff1_i32_b32 s71, s71\n"
        "s_add_u32 s71, s71, s72\n"
        "s_and_b32 s71, s71, 7\n"
        "v_mov_b32 v14, s71\n"
        "v_mov_b32 v13, s87\n"
        "v_add_u32 v13, 131072, v13\n"
        "ds_write_b32 v13, v14\n"
        "s_waitcnt lgkmcnt(0)\n"
        "L_d2u28_%=:\n"
        "s_barrier\n"
        "v_add_u32 v0, s65, v1\n"
        "s_add_u32 s65, s65, 32768\n"
        "s_cmp_ge_u32 s65, s94\n"
        "s_cselect_b32 s65, s87, s65\n"
        "ds_read_b128 v[16:19], v0 offset:0\n"
        "v_mfma_f32_16x16x32_f16 a[204:207], v[20:23], v[60:63], a[204:207]\n"
        "s_waitcnt lgkmcnt(5)\n"
        "v_mfma_f32_16x16x32_f16 a[208:211], v[24:27], v[56:59], a[208:211]\n"
        "ds_read_b128 v[20:23], v0 offset:1024\n"
        "buffer_load_dwordx4 v2, s[40:43], s63 offen offset:3072 sc1 lds\n"
        "s_add_u32 s64, s64, 32768\n"
        "s_cmp_ge_u32 s64, s92\n"
        "s_cselect_b32 s64, s93, s64\n"
        "v_mfma_f32_16x16x32_f16 a[212:215], v[24:27], v[60:63], a[212:215]\n"
        "s_lshr_b32 s63, s68, 2\n"
        "s_and_b32 s63, s63, 28\n"
        "v_mfma_f32_16x16x32_f16 a[216:219], v[28:31], v[56:59], a[216:219]\n"
        "ds_read_b128 v[24:27], v0 offset:2048\n"
        "s_lshr_b32 s63, s79, s63\n"
        "s_and_b32 s63, s63, 7\n"
        "v_mfma_f32_16x16x32_f16 a[220:223], v[28:31], v[60:63], a[220:223]\n"
        "s_lshl_b32 s63, s63, 4\n"
        "s_and_b32 s22, s68, 15\n"
        "s_waitcnt lgkmcnt(5)\n"
        "v_mfma_f32_16x16x32_f16 a[224:227], v[32:35], v[56:59], a[224:227]\n"
        "ds_read_b128 v[28:31], v0 offset:3072\n"
        "s_or_b32 s63, s63, s22\n"
        "s_lshl_b32 s63, s63, 15\n"
        "v_mfma_f32_16x16x32_f16 a[228:231], v[32:35], v[60:63], a[228:231]\n"
        "s_add_u32 s63, s63, s78\n"
        "s_add_u32 s68, s68, 1\n"
        "v_mfma_f32_16x16x32_f16 a[232:235], v[36:39], v[56:59], a[232:235]\n"
        "ds_read_b128 v[32:35], v0 offset:4096\n"
        "s_mov_b32 m0, s64\n"
        "v_mfma_f32_16x16x32_f16 a[236:239], v[36:39], v[60:63], a[236:239]\n"
        "s_waitcnt lgkmcnt(5)\n"
        "v_mfma_f32_16x16x32_f16 a[240:243], v[40:43], v[56:59], a[240:243]\n"
        "ds_read_b128 v[36:39], v0 offset:5120\n"
        "v_mfma_f32_16x16x32_f16 a[244:247], v[40:43], v[60:63], a[244:247]\n"
        "v_mfma_f32_16x16x32_f16 a[248:251], v[44:47], v[56:59], a[248:251]\n"
        "ds_read_b128 v[40:43], v0 offset:6144\n"
        "v_mfma_f32_16x16x32_f16 a[252:255], v[44:47], v[60:63], a[252:255]\n"
        "s_add_u32 s95, s95, 1\n"
        "s_add_u32 s61, s61, 1\n"
        "s_cmp_lt_u32 s61, 32\n"
        "s_cbranch_scc1 L_loop2_%=\n"
        "s_waitcnt vmcnt(0) lgkmcnt(0)\n"
        "v_mov_b32 v14, 0\n"
        "s_mov_b32 s70, 16384\n"
        "buffer_load_dword v13, v14, s[56:59], s70 offen sc1\n"
        "v_mov_b32 v2, s60\n"
        "s_mov_b64 exec, 1\n"
        "buffer_store_dword v2, v14, s[56:59], s86 offen sc1\n"
        "s_mov_b64 exec, -1\n"
        "s_waitcnt vmcnt(1)\n"
        "v_readfirstlane_b32 s70, v13\n"
        "s_nop 3\n"
        "s_cmp_eq_u32 s70, s60\n"
        "s_cselect_b32 s69, 1, s69\n"
        "s_cmp_eq_u32 s69, 0\n"
        "s_cbranch_scc1 L_good_%=\n"
        "v_mov_b32 v13, 0x7fc00000\n"
        "s_nop 7\n"
        "v_accvgpr_write_b32 a0, v13\n"
        "v_accvgpr_write_b32 a1, v13\n"
        "v_accvgpr_write_b32 a2, v13\n"
        "v_accvgpr_write_b32 a3, v13\n"
        "v_accvgpr_write_b32 a4, v13\n"
        "v_accvgpr_write_b32 a5, v13\n"
        "v_accvgpr_write_b32 a6, v13\n"
        "v_accvgpr_write_b32 a7, v13\n"
        "v_accvgpr_write_b32 a8, v13\n"
        "v_accvgpr_write_b32 a9, v13\n"
        "v_accvgpr_write_b32 a10, v13\n"
        "v_accvgpr_write_b32 a11, v13\n"
        "v_accvgpr_write_b32 a12, v13\n"
        "v_accvgpr_write_b32 a13, v13\n"
        "v_accvgpr_write_b32 a14, v13\n"
        "v_accvgpr_write_b32 a15, v13\n"
        "v_accvgpr_write_b32 a16, v13\n"
        "v_accvgpr_write_b32 a17, v13\n"
        "v_accvgpr_write_b32 a18, v13\n"
        "v_accvgpr_write_b32 a19, v13\n"
        "v_accvgpr_write_b32 a20, v13\n"
        "v_accvgpr_write_b32 a21, v13\n"
        "v_accvgpr_write_b32 a22, v13\n"
        "v_accvgpr_write_b32 a23, v13\n"
        "v_accvgpr_write_b32 a24, v13\n"
        "v_accvgpr_write_b32 a25, v13\n"
        "v_accvgpr_write_b32 a26, v13\n"
        "v_accvgpr_write_b32 a27, v13\n"
        "v_accvgpr_write_b32 a28, v13\n"
        "v_accvgpr_write_b32 a29, v13\n"
        "v_accvgpr_write_b32 a30, v13\n"
        "v_accvgpr_write_b32 a31, v13\n"
        "v_accvgpr_write_b32 a32, v13\n"
        "v_accvgpr_write_b32 a33, v13\n"
        "v_accvgpr_write_b32 a34, v13\n"
        "v_accvgpr_write_b32 a35, v13\n"
        "v_accvgpr_write_b32 a36, v13\n"
        "v_accvgpr_write_b32 a37, v13\n"
        "v_accvgpr_write_b32 a38, v13\n"
        "v_accvgpr_write_b32 a39, v13\n"
        "v_accvgpr_write_b32 a40, v13\n"
        "v_accvgpr_write_b32 a41, v13\n"
        "v_accvgpr_write_b32 a42, v13\n"
        "v_accvgpr_write_b32 a43, v13\n"
        "v_accvgpr_write_b32 a44, v13\n"
        "v_accvgpr_write_b32 a45, v13\n"
        "v_accvgpr_write_b32 a46, v13\n"
        "v_accvgpr_write_b32 a47, v13\n"
        "v_accvgpr_write_b32 a48, v13\n"
        "v_accvgpr_write_b32 a49, v13\n"
        "v_accvgpr_write_b32 a50, v13\n"
        "v_accvgpr_write_b32 a51, v13\n"
        "v_accvgpr_write_b32 a52, v13\n"
        "v_accvgpr_write_b32 a53, v13\n"
        "v_accvgpr_write_b32 a54, v13\n"
        "v_accvgpr_write_b32 a55, v13\n"
        "v_accvgpr_write_b32 a56, v13\n"
        "v_accvgpr_write_b32 a57, v13\n"
        "v_accvgpr_write_b32 a58, v13\n"
        "v_accvgpr_write_b32 a59, v13\n"
        "v_accvgpr_write_b32 a60, v13\n"
        "v_accvgpr_write_b32 a61, v13\n"
        "v_accvgpr_write_b32 a62, v13\n"
        "v_accvgpr_write_b32 a63, v13\n"
        "v_accvgpr_write_b32 a64, v13\n"
        "v_accvgpr_write_b32 a65, v13\n"
        "v_accvgpr_write_b32 a66, v13\n"
        "v_accvgpr_write_b32 a67, v13\n"
        "v_accvgpr_write_b32 a68, v13\n"
        "v_accvgpr_write_b32 a69, v13\n"
        "v_accvgpr_write_b32 a70, v13\n"
        "v_accvgpr_write_b32 a71, v13\n"
        "v_accvgpr_write_b32 a72, v13\n"
        "v_accvgpr_write_b32 a73, v13\n"
        "v_accvgpr_write_b32 a74, v13\n"
        "v_accvgpr_write_b32 a75, v13\n"
        "v_accvgpr_write_b32 a76, v13\n"
        "v_accvgpr_write_b32 a77, v13\n"
        "v_accvgpr_write_b32 a78, v13\n"
        "v_accvgpr_write_b32 a79, v13\n"
        "v_accvgpr_write_b32 a80, v13\n"
        "v_accvgpr_write_b32 a81, v13\n"
        "v_accvgpr_write_b32 a82, v13\n"
        "v_accvgpr_write_b32 a83, v13\n"
        "v_accvgpr_write_b32 a84, v13\n"
        "v_accvgpr_write_b32 a85, v13\n"
        "v_accvgpr_write_b32 a86, v13\n"
        "v_accvgpr_write_b32 a87, v13\n"
        "v_accvgpr_write_b32 a88, v13\n"
        "v_accvgpr_write_b32 a89, v13\n"
        "v_accvgpr_write_b32 a90, v13\n"
        "v_accvgpr_write_b32 a91, v13\n"
        "v_accvgpr_write_b32 a92, v13\n"
        "v_accvgpr_write_b32 a93, v13\n"
        "v_accvgpr_write_b32 a94, v13\n"
        "v_accvgpr_write_b32 a95, v13\n"
        "v_accvgpr_write_b32 a96, v13\n"
        "v_accvgpr_write_b32 a97, v13\n"
        "v_accvgpr_write_b32 a98, v13\n"
        "v_accvgpr_write_b32 a99, v13\n"
        "v_accvgpr_write_b32 a100, v13\n"
        "v_accvgpr_write_b32 a101, v13\n"
        "v_accvgpr_write_b32 a102, v13\n"
        "v_accvgpr_write_b32 a103, v13\n"
        "v_accvgpr_write_b32 a104, v13\n"
        "v_accvgpr_write_b32 a105, v13\n"
        "v_accvgpr_write_b32 a106, v13\n"
        "v_accvgpr_write_b32 a107, v13\n"
        "v_accvgpr_write_b32 a108, v13\n"
        "v_accvgpr_write_b32 a109, v13\n"
        "v_accvgpr_write_b32 a110, v13\n"
        "v_accvgpr_write_b32 a111, v13\n"
        "v_accvgpr_write_b32 a112, v13\n"
        "v_accvgpr_write_b32 a113, v13\n"
        "v_accvgpr_write_b32 a114, v13\n"
        "v_accvgpr_write_b32 a115, v13\n"
        "v_accvgpr_write_b32 a116, v13\n"
        "v_accvgpr_write_b32 a117, v13\n"
        "v_accvgpr_write_b32 a118, v13\n"
        "v_accvgpr_write_b32 a119, v13\n"
        "v_accvgpr_write_b32 a120, v13\n"
        "v_accvgpr_write_b32 a121, v13\n"
        "v_accvgpr_write_b32 a122, v13\n"
        "v_accvgpr_write_b32 a123, v13\n"
        "v_accvgpr_write_b32 a124, v13\n"
        "v_accvgpr_write_b32 a125, v13\n"
        "v_accvgpr_write_b32 a126, v13\n"
        "v_accvgpr_write_b32 a127, v13\n"
        "v_accvgpr_write_b32 a128, v13\n"
        "v_accvgpr_write_b32 a129, v13\n"
        "v_accvgpr_write_b32 a130, v13\n"
        "v_accvgpr_write_b32 a131, v13\n"
        "v_accvgpr_write_b32 a132, v13\n"
        "v_accvgpr_write_b32 a133, v13\n"
        "v_accvgpr_write_b32 a134, v13\n"
        "v_accvgpr_write_b32 a135, v13\n"
        "v_accvgpr_write_b32 a136, v13\n"
        "v_accvgpr_write_b32 a137, v13\n"
        "v_accvgpr_write_b32 a138, v13\n"
        "v_accvgpr_write_b32 a139, v13\n"
        "v_accvgpr_write_b32 a140, v13\n"
        "v_accvgpr_write_b32 a141, v13\n"
        "v_accvgpr_write_b32 a142, v13\n"
        "v_accvgpr_write_b32 a143, v13\n"
        "v_accvgpr_write_b32 a144, v13\n"
        "v_accvgpr_write_b32 a145, v13\n"
        "v_accvgpr_write_b32 a146, v13\n"
        "v_accvgpr_write_b32 a147, v13\n"
        "v_accvgpr_write_b32 a148, v13\n"
        "v_accvgpr_write_b32 a149, v13\n"
        "v_accvgpr_write_b32 a150, v13\n"
        "v_accvgpr_write_b32 a151, v13\n"
        "v_accvgpr_write_b32 a152, v13\n"
        "v_accvgpr_write_b32 a153, v13\n"
        "v_accvgpr_write_b32 a154, v13\n"
        "v_accvgpr_write_b32 a155, v13\n"
        "v_accvgpr_write_b32 a156, v13\n"
        "v_accvgpr_write_b32 a157, v13\n"
        "v_accvgpr_write_b32 a158, v13\n"
        "v_accvgpr_write_b32 a159, v13\n"
        "v_accvgpr_write_b32 a160, v13\n"
        "v_accvgpr_write_b32 a161, v13\n"
        "v_accvgpr_write_b32 a162, v13\n"
        "v_accvgpr_write_b32 a163, v13\n"
        "v_accvgpr_write_b32 a164, v13\n"
        "v_accvgpr_write_b32 a165, v13\n"
        "v_accvgpr_write_b32 a166, v13\n"
        "v_accvgpr_write_b32 a167, v13\n"
        "v_accvgpr_write_b32 a168, v13\n"
        "v_accvgpr_write_b32 a169, v13\n"
        "v_accvgpr_write_b32 a170, v13\n"
        "v_accvgpr_write_b32 a171, v13\n"
        "v_accvgpr_write_b32 a172, v13\n"
        "v_accvgpr_write_b32 a173, v13\n"
        "v_accvgpr_write_b32 a174, v13\n"
        "v_accvgpr_write_b32 a175, v13\n"
        "v_accvgpr_write_b32 a176, v13\n"
        "v_accvgpr_write_b32 a177, v13\n"
        "v_accvgpr_write_b32 a178, v13\n"
        "v_accvgpr_write_b32 a179, v13\n"
        "v_accvgpr_write_b32 a180, v13\n"
        "v_accvgpr_write_b32 a181, v13\n"
        "v_accvgpr_write_b32 a182, v13\n"
        "v_accvgpr_write_b32 a183, v13\n"
        "v_accvgpr_write_b32 a184, v13\n"
        "v_accvgpr_write_b32 a185, v13\n"
        "v_accvgpr_write_b32 a186, v13\n"
        "v_accvgpr_write_b32 a187, v13\n"
        "v_accvgpr_write_b32 a188, v13\n"
        "v_accvgpr_write_b32 a189, v13\n"
        "v_accvgpr_write_b32 a190, v13\n"
        "v_accvgpr_write_b32 a191, v13\n"
        "v_accvgpr_write_b32 a192, v13\n"
        "v_accvgpr_write_b32 a193, v13\n"
        "v_accvgpr_write_b32 a194, v13\n"
        "v_accvgpr_write_b32 a195, v13\n"
        "v_accvgpr_write_b32 a196, v13\n"
        "v_accvgpr_write_b32 a197, v13\n"
        "v_accvgpr_write_b32 a198, v13\n"
        "v_accvgpr_write_b32 a199, v13\n"
        "v_accvgpr_write_b32 a200, v13\n"
        "v_accvgpr_write_b32 a201, v13\n"
        "v_accvgpr_write_b32 a202, v13\n"
        "v_accvgpr_write_b32 a203, v13\n"
        "v_accvgpr_write_b32 a204, v13\n"
        "v_accvgpr_write_b32 a205, v13\n"
        "v_accvgpr_write_b32 a206, v13\n"
        "v_accvgpr_write_b32 a207, v13\n"
        "v_accvgpr_write_b32 a208, v13\n"
        "v_accvgpr_write_b32 a209, v13\n"
        "v_accvgpr_write_b32 a210, v13\n"
        "v_accvgpr_write_b32 a211, v13\n"
        "v_accvgpr_write_b32 a212, v13\n"
        "v_accvgpr_write_b32 a213, v13\n"
        "v_accvgpr_write_b32 a214, v13\n"
        "v_accvgpr_write_b32 a215, v13\n"
        "v_accvgpr_write_b32 a216, v13\n"
        "v_accvgpr_write_b32 a217, v13\n"
        "v_accvgpr_write_b32 a218, v13\n"
        "v_accvgpr_write_b32 a219, v13\n"
        "v_accvgpr_write_b32 a220, v13\n"
        "v_accvgpr_write_b32 a221, v13\n"
        "v_accvgpr_write_b32 a222, v13\n"
        "v_accvgpr_write_b32 a223, v13\n"
        "v_accvgpr_write_b32 a224, v13\n"
        "v_accvgpr_write_b32 a225, v13\n"
        "v_accvgpr_write_b32 a226, v13\n"
        "v_accvgpr_write_b32 a227, v13\n"
        "v_accvgpr_write_b32 a228, v13\n"
        "v_accvgpr_write_b32 a229, v13\n"
        "v_accvgpr_write_b32 a230, v13\n"
        "v_accvgpr_write_b32 a231, v13\n"
        "v_accvgpr_write_b32 a232, v13\n"
        "v_accvgpr_write_b32 a233, v13\n"
        "v_accvgpr_write_b32 a234, v13\n"
        "v_accvgpr_write_b32 a235, v13\n"
        "v_accvgpr_write_b32 a236, v13\n"
        "v_accvgpr_write_b32 a237, v13\n"
        "v_accvgpr_write_b32 a238, v13\n"
        "v_accvgpr_write_b32 a239, v13\n"
        "v_accvgpr_write_b32 a240, v13\n"
        "v_accvgpr_write_b32 a241, v13\n"
        "v_accvgpr_write_b32 a242, v13\n"
        "v_accvgpr_write_b32 a243, v13\n"
        "v_accvgpr_write_b32 a244, v13\n"
        "v_accvgpr_write_b32 a245, v13\n"
        "v_accvgpr_write_b32 a246, v13\n"
        "v_accvgpr_write_b32 a247, v13\n"
        "v_accvgpr_write_b32 a248, v13\n"
        "v_accvgpr_write_b32 a249, v13\n"
        "v_accvgpr_write_b32 a250, v13\n"
        "v_accvgpr_write_b32 a251, v13\n"
        "v_accvgpr_write_b32 a252, v13\n"
        "v_accvgpr_write_b32 a253, v13\n"
        "v_accvgpr_write_b32 a254, v13\n"
        "v_accvgpr_write_b32 a255, v13\n"
        "L_good_%=:\n"
        "s_nop 7\n"
        "s_nop 7\n"
        "v_lshrrev_b32 v13, 8, v1\n"
        "v_and_b32 v14, 0xf0, v1\n"
        "v_lshrrev_b32 v14, 1, v14\n"
        "v_lshl_add_u32 v2, v13, 19, v14\n"
        "s_lshl_b32 s70, s80, 9\n"
        "s_lshl_b32 s71, s83, 7\n"
        "s_add_u32 s70, s70, s71\n"
        "v_add_u32 v2, s70, v2\n"
        "v_mov_b32 v16, 0xc0135761\n"
        "v_mov_b32 v17, 0xc0135761\n"
        "v_mov_b32 v18, 0xbdd2d3e8\n"
        "v_mov_b32 v19, 0xbdd2d3e8\n"
        "v_accvgpr_read_b32 v24, a0\n"
        "v_accvgpr_read_b32 v25, a4\n"
        "v_accvgpr_read_b32 v30, a1\n"
        "v_accvgpr_read_b32 v31, a5\n"
        "v_accvgpr_read_b32 v36, a2\n"
        "v_accvgpr_read_b32 v37, a6\n"
        "v_accvgpr_read_b32 v42, a3\n"
        "v_accvgpr_read_b32 v43, a7\n"
        "v_pk_mul_f32 v[26:27], v[24:25], v[24:25]\n"
        "v_pk_mul_f32 v[32:33], v[30:31], v[30:31]\n"
        "v_pk_mul_f32 v[38:39], v[36:37], v[36:37]\n"
        "v_pk_mul_f32 v[44:45], v[42:43], v[42:43]\n"
        "v_pk_fma_f32 v[26:27], v[26:27], v[18:19], v[16:17]\n"
        "v_pk_fma_f32 v[32:33], v[32:33], v[18:19], v[16:17]\n"
        "v_pk_fma_f32 v[38:39], v[38:39], v[18:19], v[16:17]\n"
        "v_pk_fma_f32 v[44:45], v[44:45], v[18:19], v[16:17]\n"
        "v_pk_mul_f32 v[26:27], v[26:27], v[24:25]\n"
        "v_pk_mul_f32 v[32:33], v[32:33], v[30:31]\n"
        "v_pk_mul_f32 v[38:39], v[38:39], v[36:37]\n"
        "v_pk_mul_f32 v[44:45], v[44:45], v[42:43]\n"
        "v_exp_f32 v26, v26\n"
        "v_exp_f32 v27, v27\n"
        "v_exp_f32 v32, v32\n"
        "v_exp_f32 v33, v33\n"
        "v_exp_f32 v38, v38\n"
        "v_exp_f32 v39, v39\n"
        "v_exp_f32 v44, v44\n"
        "v_exp_f32 v45, v45\n"
        "v_pk_add_f32 v[26:27], v[26:27], 1.0 op_sel_hi:[1,0]\n"
        "v_pk_add_f32 v[32:33], v[32:33], 1.0 op_sel_hi:[1,0]\n"
        "v_pk_add_f32 v[38:39], v[38:39], 1.0 op_sel_hi:[1,0]\n"
        "v_pk_add_f32 v[44:45], v[44:45], 1.0 op_sel_hi:[1,0]\n"
        "v_rcp_f32 v26, v26\n"
        "v_rcp_f32 v27, v27\n"
        "v_rcp_f32 v32, v32\n"
        "v_rcp_f32 v33, v33\n"
        "v_rcp_f32 v38, v38\n"
        "v_rcp_f32 v39, v39\n"
        "v_rcp_f32 v44, v44\n"
        "v_rcp_f32 v45, v45\n"
        "s_mov_b32 s70, 0\n"
        "s_mov_b32 s71, 131072\n"
        "s_mov_b32 s72, 262144\n"
        "s_mov_b32 s73, 393216\n"
        "v_pk_mul_f32 v[28:29], v[26:27], v[24:25]\n"
        "v_pk_mul_f32 v[34:35], v[32:33], v[30:31]\n"
        "v_pk_mul_f32 v[40:41], v[38:39], v[36:37]\n"
        "v_pk_mul_f32 v[46:47], v[44:45], v[42:43]\n"
        "buffer_store_dwordx2 v[28:29], v2, s[44:47], s70 offen nt\n"
        "buffer_store_dwordx2 v[34:35], v2, s[44:47], s71 offen nt\n"
        "buffer_store_dwordx2 v[40:41], v2, s[44:47], s72 offen nt\n"
        "buffer_store_dwordx2 v[46:47], v2, s[44:47], s73 offen nt\n"
        "v_accvgpr_read_b32 v48, a8\n"
        "v_accvgpr_read_b32 v49, a12\n"
        "v_accvgpr_read_b32 v54, a9\n"
        "v_accvgpr_read_b32 v55, a13\n"
        "v_accvgpr_read_b32 v60, a10\n"
        "v_accvgpr_read_b32 v61, a14\n"
        "v_accvgpr_read_b32 v66, a11\n"
        "v_accvgpr_read_b32 v67, a15\n"
        "v_pk_mul_f32 v[50:51], v[48:49], v[48:49]\n"
        "v_pk_mul_f32 v[56:57], v[54:55], v[54:55]\n"
        "v_pk_mul_f32 v[62:63], v[60:61], v[60:61]\n"
        "v_pk_mul_f32 v[68:69], v[66:67], v[66:67]\n"
        "v_pk_fma_f32 v[50:51], v[50:51], v[18:19], v[16:17]\n"
        "v_pk_fma_f32 v[56:57], v[56:57], v[18:19], v[16:17]\n"
        "v_pk_fma_f32 v[62:63], v[62:63], v[18:19], v[16:17]\n"
        "v_pk_fma_f32 v[68:69], v[68:69], v[18:19], v[16:17]\n"
        "v_pk_mul_f32 v[50:51], v[50:51], v[48:49]\n"
        "v_pk_mul_f32 v[56:57], v[56:57], v[54:55]\n"
        "v_pk_mul_f32 v[62:63], v[62:63], v[60:61]\n"
        "v_pk_mul_f32 v[68:69], v[68:69], v[66:67]\n"
        "v_exp_f32 v50, v50\n"
        "v_exp_f32 v51, v51\n"
        "v_exp_f32 v56, v56\n"
        "v_exp_f32 v57, v57\n"
        "v_exp_f32 v62, v62\n"
        "v_exp_f32 v63, v63\n"
        "v_exp_f32 v68, v68\n"
        "v_exp_f32 v69, v69\n"
        "v_pk_add_f32 v[50:51], v[50:51], 1.0 op_sel_hi:[1,0]\n"
        "v_pk_add_f32 v[56:57], v[56:57], 1.0 op_sel_hi:[1,0]\n"
        "v_pk_add_f32 v[62:63], v[62:63], 1.0 op_sel_hi:[1,0]\n"
        "v_pk_add_f32 v[68:69], v[68:69], 1.0 op_sel_hi:[1,0]\n"
        "v_rcp_f32 v50, v50\n"
        "v_rcp_f32 v51, v51\n"
        "v_rcp_f32 v56, v56\n"
        "v_rcp_f32 v57, v57\n"
        "v_rcp_f32 v62, v62\n"
        "v_rcp_f32 v63, v63\n"
        "v_rcp_f32 v68, v68\n"
        "v_rcp_f32 v69, v69\n"
        "s_mov_b32 s70, 2097152\n"
        "s_mov_b32 s71, 2228224\n"
        "s_mov_b32 s72, 2359296\n"
        "s_mov_b32 s73, 2490368\n"
        "v_pk_mul_f32 v[52:53], v[50:51], v[48:49]\n"
        "v_pk_mul_f32 v[58:59], v[56:57], v[54:55]\n"
        "v_pk_mul_f32 v[64:65], v[62:63], v[60:61]\n"
        "v_pk_mul_f32 v[70:71], v[68:69], v[66:67]\n"
        "buffer_store_dwordx2 v[52:53], v2, s[44:47], s70 offen nt\n"
        "buffer_store_dwordx2 v[58:59], v2, s[44:47], s71 offen nt\n"
        "buffer_store_dwordx2 v[64:65], v2, s[44:47], s72 offen nt\n"
        "buffer_store_dwordx2 v[70:71], v2, s[44:47], s73 offen nt\n"
        "v_accvgpr_read_b32 v24, a16\n"
        "v_accvgpr_read_b32 v25, a20\n"
        "v_accvgpr_read_b32 v30, a17\n"
        "v_accvgpr_read_b32 v31, a21\n"
        "v_accvgpr_read_b32 v36, a18\n"
        "v_accvgpr_read_b32 v37, a22\n"
        "v_accvgpr_read_b32 v42, a19\n"
        "v_accvgpr_read_b32 v43, a23\n"
        "v_pk_mul_f32 v[26:27], v[24:25], v[24:25]\n"
        "v_pk_mul_f32 v[32:33], v[30:31], v[30:31]\n"
        "v_pk_mul_f32 v[38:39], v[36:37], v[36:37]\n"
        "v_pk_mul_f32 v[44:45], v[42:43], v[42:43]\n"
        "v_pk_fma_f32 v[26:27], v[26:27], v[18:19], v[16:17]\n"
        "v_pk_fma_f32 v[32:33], v[32:33], v[18:19], v[16:17]\n"
        "v_pk_fma_f32 v[38:39], v[38:39], v[18:19], v[16:17]\n"
        "v_pk_fma_f32 v[44:45], v[44:45], v[18:19], v[16:17]\n"
        "v_pk_mul_f32 v[26:27], v[26:27], v[24:25]\n"
        "v_pk_mul_f32 v[32:33], v[32:33], v[30:31]\n"
        "v_pk_mul_f32 v[38:39], v[38:39], v[36:37]\n"
        "v_pk_mul_f32 v[44:45], v[44:45], v[42:43]\n"
        "v_exp_f32 v26, v26\n"
        "v_exp_f32 v27, v27\n"
        "v_exp_f32 v32, v32\n"
        "v_exp_f32 v33, v33\n"
        "v_exp_f32 v38, v38\n"
        "v_exp_f32 v39, v39\n"
        "v_exp_f32 v44, v44\n"
        "v_exp_f32 v45, v45\n"
        "v_pk_add_f32 v[26:27], v[26:27], 1.0 op_sel_hi:[1,0]\n"
        "v_pk_add_f32 v[32:33], v[32:33], 1.0 op_sel_hi:[1,0]\n"
        "v_pk_add_f32 v[38:39], v[38:39], 1.0 op_sel_hi:[1,0]\n"
        "v_pk_add_f32 v[44:45], v[44:45], 1.0 op_sel_hi:[1,0]\n"
        "v_rcp_f32 v26, v26\n"
        "v_rcp_f32 v27, v27\n"
        "v_rcp_f32 v32, v32\n"
        "v_rcp_f32 v33, v33\n"
        "v_rcp_f32 v38, v38\n"
        "v_rcp_f32 v39, v39\n"
        "v_rcp_f32 v44, v44\n"
        "v_rcp_f32 v45, v45\n"
        "s_mov_b32 s70, 4194304\n"
        "s_mov_b32 s71, 4325376\n"
        "s_mov_b32 s72, 4456448\n"
        "s_mov_b32 s73, 4587520\n"
        "v_pk_mul_f32 v[28:29], v[26:27], v[24:25]\n"
        "v_pk_mul_f32 v[34:35], v[32:33], v[30:31]\n"
        "v_pk_mul_f32 v[40:41], v[38:39], v[36:37]\n"
        "v_pk_mul_f32 v[46:47], v[44:45], v[42:43]\n"
        "buffer_store_dwordx2 v[28:29], v2, s[44:47], s70 offen nt\n"
        "buffer_store_dwordx2 v[34:35], v2, s[44:47], s71 offen nt\n"
        "buffer_store_dwordx2 v[40:41], v2, s[44:47], s72 offen nt\n"
        "buffer_store_dwordx2 v[46:47], v2, s[44:47], s73 offen nt\n"
        "v_accvgpr_read_b32 v48, a24\n"
        "v_accvgpr_read_b32 v49, a28\n"
        "v_accvgpr_read_b32 v54, a25\n"
        "v_accvgpr_read_b32 v55, a29\n"
        "v_accvgpr_read_b32 v60, a26\n"
        "v_accvgpr_read_b32 v61, a30\n"
        "v_accvgpr_read_b32 v66, a27\n"
        "v_accvgpr_read_b32 v67, a31\n"
        "v_pk_mul_f32 v[50:51], v[48:49], v[48:49]\n"
        "v_pk_mul_f32 v[56:57], v[54:55], v[54:55]\n"
        "v_pk_mul_f32 v[62:63], v[60:61], v[60:61]\n"
        "v_pk_mul_f32 v[68:69], v[66:67], v[66:67]\n"
        "v_pk_fma_f32 v[50:51], v[50:51], v[18:19], v[16:17]\n"
        "v_pk_fma_f32 v[56:57], v[56:57], v[18:19], v[16:17]\n"
        "v_pk_fma_f32 v[62:63], v[62:63], v[18:19], v[16:17]\n"
        "v_pk_fma_f32 v[68:69], v[68:69], v[18:19], v[16:17]\n"
        "v_pk_mul_f32 v[50:51], v[50:51], v[48:49]\n"
        "v_pk_mul_f32 v[56:57], v[56:57], v[54:55]\n"
        "v_pk_mul_f32 v[62:63], v[62:63], v[60:61]\n"
        "v_pk_mul_f32 v[68:69], v[68:69], v[66:67]\n"
        "v_exp_f32 v50, v50\n"
        "v_exp_f32 v51, v51\n"
        "v_exp_f32 v56, v56\n"
        "v_exp_f32 v57, v57\n"
        "v_exp_f32 v62, v62\n"
        "v_exp_f32 v63, v63\n"
        "v_exp_f32 v68, v68\n"
        "v_exp_f32 v69, v69\n"
        "v_pk_add_f32 v[50:51], v[50:51], 1.0 op_sel_hi:[1,0]\n"
        "v_pk_add_f32 v[56:57], v[56:57], 1.0 op_sel_hi:[1,0]\n"
        "v_pk_add_f32 v[62:63], v[62:63], 1.0 op_sel_hi:[1,0]\n"
        "v_pk_add_f32 v[68:69], v[68:69], 1.0 op_sel_hi:[1,0]\n"
        "v_rcp_f32 v50, v50\n"
        "v_rcp_f32 v51, v51\n"
        "v_rcp_f32 v56, v56\n"
        "v_rcp_f32 v57, v57\n"
        "v_rcp_f32 v62, v62\n"
        "v_rcp_f32 v63, v63\n"
        "v_rcp_f32 v68, v68\n"
        "v_rcp_f32 v69, v69\n"
        "s_mov_b32 s70, 6291456\n"
        "s_mov_b32 s71, 6422528\n"
        "s_mov_b32 s72, 6553600\n"
        "s_mov_b32 s73, 6684672\n"
        "v_pk_mul_f32 v[52:53], v[50:51], v[48:49]\n"
        "v_pk_mul_f32 v[58:59], v[56:57], v[54:55]\n"
        "v_pk_mul_f32 v[64:65], v[62:63], v[60:61]\n"
        "v_pk_mul_f32 v[70:71], v[68:69], v[66:67]\n"
        "buffer_store_dwordx2 v[52:53], v2, s[44:47], s70 offen nt\n"
        "buffer_store_dwordx2 v[58:59], v2, s[44:47], s71 offen nt\n"
        "buffer_store_dwordx2 v[64:65], v2, s[44:47], s72 offen nt\n"
        "buffer_store_dwordx2 v[70:71], v2, s[44:47], s73 offen nt\n"
        "v_accvgpr_read_b32 v24, a32\n"
        "v_accvgpr_read_b32 v25, a36\n"
        "v_accvgpr_read_b32 v30, a33\n"
        "v_accvgpr_read_b32 v31, a37\n"
        "v_accvgpr_read_b32 v36, a34\n"
        "v_accvgpr_read_b32 v37, a38\n"
        "v_accvgpr_read_b32 v42, a35\n"
        "v_accvgpr_read_b32 v43, a39\n"
        "v_pk_mul_f32 v[26:27], v[24:25], v[24:25]\n"
        "v_pk_mul_f32 v[32:33], v[30:31], v[30:31]\n"
        "v_pk_mul_f32 v[38:39], v[36:37], v[36:37]\n"
        "v_pk_mul_f32 v[44:45], v[42:43], v[42:43]\n"
        "v_pk_fma_f32 v[26:27], v[26:27], v[18:19], v[16:17]\n"
        "v_pk_fma_f32 v[32:33], v[32:33], v[18:19], v[16:17]\n"
        "v_pk_fma_f32 v[38:39], v[38:39], v[18:19], v[16:17]\n"
        "v_pk_fma_f32 v[44:45], v[44:45], v[18:19], v[16:17]\n"
        "v_pk_mul_f32 v[26:27], v[26:27], v[24:25]\n"
        "v_pk_mul_f32 v[32:33], v[32:33], v[30:31]\n"
        "v_pk_mul_f32 v[38:39], v[38:39], v[36:37]\n"
        "v_pk_mul_f32 v[44:45], v[44:45], v[42:43]\n"
        "v_exp_f32 v26, v26\n"
        "v_exp_f32 v27, v27\n"
        "v_exp_f32 v32, v32\n"
        "v_exp_f32 v33, v33\n"
        "v_exp_f32 v38, v38\n"
        "v_exp_f32 v39, v39\n"
        "v_exp_f32 v44, v44\n"
        "v_exp_f32 v45, v45\n"
        "v_pk_add_f32 v[26:27], v[26:27], 1.0 op_sel_hi:[1,0]\n"
        "v_pk_add_f32 v[32:33], v[32:33], 1.0 op_sel_hi:[1,0]\n"
        "v_pk_add_f32 v[38:39], v[38:39], 1.0 op_sel_hi:[1,0]\n"
        "v_pk_add_f32 v[44:45], v[44:45], 1.0 op_sel_hi:[1,0]\n"
        "v_rcp_f32 v26, v26\n"
        "v_rcp_f32 v27, v27\n"
        "v_rcp_f32 v32, v32\n"
        "v_rcp_f32 v33, v33\n"
        "v_rcp_f32 v38, v38\n"
        "v_rcp_f32 v39, v39\n"
        "v_rcp_f32 v44, v44\n"
        "v_rcp_f32 v45, v45\n"
        "s_mov_b32 s70, 8388608\n"
        "s_mov_b32 s71, 8519680\n"
        "s_mov_b32 s72, 8650752\n"
        "s_mov_b32 s73, 8781824\n"
        "v_pk_mul_f32 v[28:29], v[26:27], v[24:25]\n"
        "v_pk_mul_f32 v[34:35], v[32:33], v[30:31]\n"
        "v_pk_mul_f32 v[40:41], v[38:39], v[36:37]\n"
        "v_pk_mul_f32 v[46:47], v[44:45], v[42:43]\n"
        "buffer_store_dwordx2 v[28:29], v2, s[44:47], s70 offen nt\n"
        "buffer_store_dwordx2 v[34:35], v2, s[44:47], s71 offen nt\n"
        "buffer_store_dwordx2 v[40:41], v2, s[44:47], s72 offen nt\n"
        "buffer_store_dwordx2 v[46:47], v2, s[44:47], s73 offen nt\n"
        "v_accvgpr_read_b32 v48, a40\n"
        "v_accvgpr_read_b32 v49, a44\n"
        "v_accvgpr_read_b32 v54, a41\n"
        "v_accvgpr_read_b32 v55, a45\n"
        "v_accvgpr_read_b32 v60, a42\n"
        "v_accvgpr_read_b32 v61, a46\n"
        "v_accvgpr_read_b32 v66, a43\n"
        "v_accvgpr_read_b32 v67, a47\n"
        "v_pk_mul_f32 v[50:51], v[48:49], v[48:49]\n"
        "v_pk_mul_f32 v[56:57], v[54:55], v[54:55]\n"
        "v_pk_mul_f32 v[62:63], v[60:61], v[60:61]\n"
        "v_pk_mul_f32 v[68:69], v[66:67], v[66:67]\n"
        "v_pk_fma_f32 v[50:51], v[50:51], v[18:19], v[16:17]\n"
        "v_pk_fma_f32 v[56:57], v[56:57], v[18:19], v[16:17]\n"
        "v_pk_fma_f32 v[62:63], v[62:63], v[18:19], v[16:17]\n"
        "v_pk_fma_f32 v[68:69], v[68:69], v[18:19], v[16:17]\n"
        "v_pk_mul_f32 v[50:51], v[50:51], v[48:49]\n"
        "v_pk_mul_f32 v[56:57], v[56:57], v[54:55]\n"
        "v_pk_mul_f32 v[62:63], v[62:63], v[60:61]\n"
        "v_pk_mul_f32 v[68:69], v[68:69], v[66:67]\n"
        "v_exp_f32 v50, v50\n"
        "v_exp_f32 v51, v51\n"
        "v_exp_f32 v56, v56\n"
        "v_exp_f32 v57, v57\n"
        "v_exp_f32 v62, v62\n"
        "v_exp_f32 v63, v63\n"
        "v_exp_f32 v68, v68\n"
        "v_exp_f32 v69, v69\n"
        "v_pk_add_f32 v[50:51], v[50:51], 1.0 op_sel_hi:[1,0]\n"
        "v_pk_add_f32 v[56:57], v[56:57], 1.0 op_sel_hi:[1,0]\n"
        "v_pk_add_f32 v[62:63], v[62:63], 1.0 op_sel_hi:[1,0]\n"
        "v_pk_add_f32 v[68:69], v[68:69], 1.0 op_sel_hi:[1,0]\n"
        "v_rcp_f32 v50, v50\n"
        "v_rcp_f32 v51, v51\n"
        "v_rcp_f32 v56, v56\n"
        "v_rcp_f32 v57, v57\n"
        "v_rcp_f32 v62, v62\n"
        "v_rcp_f32 v63, v63\n"
        "v_rcp_f32 v68, v68\n"
        "v_rcp_f32 v69, v69\n"
        "s_mov_b32 s70, 10485760\n"
        "s_mov_b32 s71, 10616832\n"
        "s_mov_b32 s72, 10747904\n"
        "s_mov_b32 s73, 10878976\n"
        "v_pk_mul_f32 v[52:53], v[50:51], v[48:49]\n"
        "v_pk_mul_f32 v[58:59], v[56:57], v[54:55]\n"
        "v_pk_mul_f32 v[64:65], v[62:63], v[60:61]\n"
        "v_pk_mul_f32 v[70:71], v[68:69], v[66:67]\n"
        "buffer_store_dwordx2 v[52:53], v2, s[44:47], s70 offen nt\n"
        "buffer_store_dwordx2 v[58:59], v2, s[44:47], s71 offen nt\n"
        "buffer_store_dwordx2 v[64:65], v2, s[44:47], s72 offen nt\n"
        "buffer_store_dwordx2 v[70:71], v2, s[44:47], s73 offen nt\n"
        "v_accvgpr_read_b32 v24, a48\n"
        "v_accvgpr_read_b32 v25, a52\n"
        "v_accvgpr_read_b32 v30, a49\n"
        "v_accvgpr_read_b32 v31, a53\n"
        "v_accvgpr_read_b32 v36, a50\n"
        "v_accvgpr_read_b32 v37, a54\n"
        "v_accvgpr_read_b32 v42, a51\n"
        "v_accvgpr_read_b32 v43, a55\n"
        "v_pk_mul_f32 v[26:27], v[24:25], v[24:25]\n"
        "v_pk_mul_f32 v[32:33], v[30:31], v[30:31]\n"
        "v_pk_mul_f32 v[38:39], v[36:37], v[36:37]\n"
        "v_pk_mul_f32 v[44:45], v[42:43], v[42:43]\n"
        "v_pk_fma_f32 v[26:27], v[26:27], v[18:19], v[16:17]\n"
        "v_pk_fma_f32 v[32:33], v[32:33], v[18:19], v[16:17]\n"
        "v_pk_fma_f32 v[38:39], v[38:39], v[18:19], v[16:17]\n"
        "v_pk_fma_f32 v[44:45], v[44:45], v[18:19], v[16:17]\n"
        "v_pk_mul_f32 v[26:27], v[26:27], v[24:25]\n"
        "v_pk_mul_f32 v[32:33], v[32:33], v[30:31]\n"
        "v_pk_mul_f32 v[38:39], v[38:39], v[36:37]\n"
        "v_pk_mul_f32 v[44:45], v[44:45], v[42:43]\n"
        "v_exp_f32 v26, v26\n"
        "v_exp_f32 v27, v27\n"
        "v_exp_f32 v32, v32\n"
        "v_exp_f32 v33, v33\n"
        "v_exp_f32 v38, v38\n"
        "v_exp_f32 v39, v39\n"
        "v_exp_f32 v44, v44\n"
        "v_exp_f32 v45, v45\n"
        "v_pk_add_f32 v[26:27], v[26:27], 1.0 op_sel_hi:[1,0]\n"
        "v_pk_add_f32 v[32:33], v[32:33], 1.0 op_sel_hi:[1,0]\n"
        "v_pk_add_f32 v[38:39], v[38:39], 1.0 op_sel_hi:[1,0]\n"
        "v_pk_add_f32 v[44:45], v[44:45], 1.0 op_sel_hi:[1,0]\n"
        "v_rcp_f32 v26, v26\n"
        "v_rcp_f32 v27, v27\n"
        "v_rcp_f32 v32, v32\n"
        "v_rcp_f32 v33, v33\n"
        "v_rcp_f32 v38, v38\n"
        "v_rcp_f32 v39, v39\n"
        "v_rcp_f32 v44, v44\n"
        "v_rcp_f32 v45, v45\n"
        "s_mov_b32 s70, 12582912\n"
        "s_mov_b32 s71, 12713984\n"
        "s_mov_b32 s72, 12845056\n"
        "s_mov_b32 s73, 12976128\n"
        "v_pk_mul_f32 v[28:29], v[26:27], v[24:25]\n"
        "v_pk_mul_f32 v[34:35], v[32:33], v[30:31]\n"
        "v_pk_mul_f32 v[40:41], v[38:39], v[36:37]\n"
        "v_pk_mul_f32 v[46:47], v[44:45], v[42:43]\n"
        "buffer_store_dwordx2 v[28:29], v2, s[44:47], s70 offen nt\n"
        "buffer_store_dwordx2 v[34:35], v2, s[44:47], s71 offen nt\n"
        "buffer_store_dwordx2 v[40:41], v2, s[44:47], s72 offen nt\n"
        "buffer_store_dwordx2 v[46:47], v2, s[44:47], s73 offen nt\n"
        "v_accvgpr_read_b32 v48, a56\n"
        "v_accvgpr_read_b32 v49, a60\n"
        "v_accvgpr_read_b32 v54, a57\n"
        "v_accvgpr_read_b32 v55, a61\n"
        "v_accvgpr_read_b32 v60, a58\n"
        "v_accvgpr_read_b32 v61, a62\n"
        "v_accvgpr_read_b32 v66, a59\n"
        "v_accvgpr_read_b32 v67, a63\n"
        "v_pk_mul_f32 v[50:51], v[48:49], v[48:49]\n"
        "v_pk_mul_f32 v[56:57], v[54:55], v[54:55]\n"
        "v_pk_mul_f32 v[62:63], v[60:61], v[60:61]\n"
        "v_pk_mul_f32 v[68:69], v[66:67], v[66:67]\n"
        "v_pk_fma_f32 v[50:51], v[50:51], v[18:19], v[16:17]\n"
        "v_pk_fma_f32 v[56:57], v[56:57], v[18:19], v[16:17]\n"
        "v_pk_fma_f32 v[62:63], v[62:63], v[18:19], v[16:17]\n"
        "v_pk_fma_f32 v[68:69], v[68:69], v[18:19], v[16:17]\n"
        "v_pk_mul_f32 v[50:51], v[50:51], v[48:49]\n"
        "v_pk_mul_f32 v[56:57], v[56:57], v[54:55]\n"
        "v_pk_mul_f32 v[62:63], v[62:63], v[60:61]\n"
        "v_pk_mul_f32 v[68:69], v[68:69], v[66:67]\n"
        "v_exp_f32 v50, v50\n"
        "v_exp_f32 v51, v51\n"
        "v_exp_f32 v56, v56\n"
        "v_exp_f32 v57, v57\n"
        "v_exp_f32 v62, v62\n"
        "v_exp_f32 v63, v63\n"
        "v_exp_f32 v68, v68\n"
        "v_exp_f32 v69, v69\n"
        "v_pk_add_f32 v[50:51], v[50:51], 1.0 op_sel_hi:[1,0]\n"
        "v_pk_add_f32 v[56:57], v[56:57], 1.0 op_sel_hi:[1,0]\n"
        "v_pk_add_f32 v[62:63], v[62:63], 1.0 op_sel_hi:[1,0]\n"
        "v_pk_add_f32 v[68:69], v[68:69], 1.0 op_sel_hi:[1,0]\n"
        "v_rcp_f32 v50, v50\n"
        "v_rcp_f32 v51, v51\n"
        "v_rcp_f32 v56, v56\n"
        "v_rcp_f32 v57, v57\n"
        "v_rcp_f32 v62, v62\n"
        "v_rcp_f32 v63, v63\n"
        "v_rcp_f32 v68, v68\n"
        "v_rcp_f32 v69, v69\n"
        "s_mov_b32 s70, 14680064\n"
        "s_mov_b32 s71, 14811136\n"
        "s_mov_b32 s72, 14942208\n"
        "s_mov_b32 s73, 15073280\n"
        "v_pk_mul_f32 v[52:53], v[50:51], v[48:49]\n"
        "v_pk_mul_f32 v[58:59], v[56:57], v[54:55]\n"
        "v_pk_mul_f32 v[64:65], v[62:63], v[60:61]\n"
        "v_pk_mul_f32 v[70:71], v[68:69], v[66:67]\n"
        "buffer_store_dwordx2 v[52:53], v2, s[44:47], s70 offen nt\n"
        "buffer_store_dwordx2 v[58:59], v2, s[44:47], s71 offen nt\n"
        "buffer_store_dwordx2 v[64:65], v2, s[44:47], s72 offen nt\n"
        "buffer_store_dwordx2 v[70:71], v2, s[44:47], s73 offen nt\n"
        "v_accvgpr_read_b32 v24, a64\n"
        "v_accvgpr_read_b32 v25, a68\n"
        "v_accvgpr_read_b32 v30, a65\n"
        "v_accvgpr_read_b32 v31, a69\n"
        "v_accvgpr_read_b32 v36, a66\n"
        "v_accvgpr_read_b32 v37, a70\n"
        "v_accvgpr_read_b32 v42, a67\n"
        "v_accvgpr_read_b32 v43, a71\n"
        "v_pk_mul_f32 v[26:27], v[24:25], v[24:25]\n"
        "v_pk_mul_f32 v[32:33], v[30:31], v[30:31]\n"
        "v_pk_mul_f32 v[38:39], v[36:37], v[36:37]\n"
        "v_pk_mul_f32 v[44:45], v[42:43], v[42:43]\n"
        "v_pk_fma_f32 v[26:27], v[26:27], v[18:19], v[16:17]\n"
        "v_pk_fma_f32 v[32:33], v[32:33], v[18:19], v[16:17]\n"
        "v_pk_fma_f32 v[38:39], v[38:39], v[18:19], v[16:17]\n"
        "v_pk_fma_f32 v[44:45], v[44:45], v[18:19], v[16:17]\n"
        "v_pk_mul_f32 v[26:27], v[26:27], v[24:25]\n"
        "v_pk_mul_f32 v[32:33], v[32:33], v[30:31]\n"
        "v_pk_mul_f32 v[38:39], v[38:39], v[36:37]\n"
        "v_pk_mul_f32 v[44:45], v[44:45], v[42:43]\n"
        "v_exp_f32 v26, v26\n"
        "v_exp_f32 v27, v27\n"
        "v_exp_f32 v32, v32\n"
        "v_exp_f32 v33, v33\n"
        "v_exp_f32 v38, v38\n"
        "v_exp_f32 v39, v39\n"
        "v_exp_f32 v44, v44\n"
        "v_exp_f32 v45, v45\n"
        "v_pk_add_f32 v[26:27], v[26:27], 1.0 op_sel_hi:[1,0]\n"
        "v_pk_add_f32 v[32:33], v[32:33], 1.0 op_sel_hi:[1,0]\n"
        "v_pk_add_f32 v[38:39], v[38:39], 1.0 op_sel_hi:[1,0]\n"
        "v_pk_add_f32 v[44:45], v[44:45], 1.0 op_sel_hi:[1,0]\n"
        "v_rcp_f32 v26, v26\n"
        "v_rcp_f32 v27, v27\n"
        "v_rcp_f32 v32, v32\n"
        "v_rcp_f32 v33, v33\n"
        "v_rcp_f32 v38, v38\n"
        "v_rcp_f32 v39, v39\n"
        "v_rcp_f32 v44, v44\n"
        "v_rcp_f32 v45, v45\n"
        "s_mov_b32 s70, 16777216\n"
        "s_mov_b32 s71, 16908288\n"
        "s_mov_b32 s72, 17039360\n"
        "s_mov_b32 s73, 17170432\n"
        "v_pk_mul_f32 v[28:29], v[26:27], v[24:25]\n"
        "v_pk_mul_f32 v[34:35], v[32:33], v[30:31]\n"
        "v_pk_mul_f32 v[40:41], v[38:39], v[36:37]\n"
        "v_pk_mul_f32 v[46:47], v[44:45], v[42:43]\n"
        "buffer_store_dwordx2 v[28:29], v2, s[44:47], s70 offen nt\n"
        "buffer_store_dwordx2 v[34:35], v2, s[44:47], s71 offen nt\n"
        "buffer_store_dwordx2 v[40:41], v2, s[44:47], s72 offen nt\n"
        "buffer_store_dwordx2 v[46:47], v2, s[44:47], s73 offen nt\n"
        "v_accvgpr_read_b32 v48, a72\n"
        "v_accvgpr_read_b32 v49, a76\n"
        "v_accvgpr_read_b32 v54, a73\n"
        "v_accvgpr_read_b32 v55, a77\n"
        "v_accvgpr_read_b32 v60, a74\n"
        "v_accvgpr_read_b32 v61, a78\n"
        "v_accvgpr_read_b32 v66, a75\n"
        "v_accvgpr_read_b32 v67, a79\n"
        "v_pk_mul_f32 v[50:51], v[48:49], v[48:49]\n"
        "v_pk_mul_f32 v[56:57], v[54:55], v[54:55]\n"
        "v_pk_mul_f32 v[62:63], v[60:61], v[60:61]\n"
        "v_pk_mul_f32 v[68:69], v[66:67], v[66:67]\n"
        "v_pk_fma_f32 v[50:51], v[50:51], v[18:19], v[16:17]\n"
        "v_pk_fma_f32 v[56:57], v[56:57], v[18:19], v[16:17]\n"
        "v_pk_fma_f32 v[62:63], v[62:63], v[18:19], v[16:17]\n"
        "v_pk_fma_f32 v[68:69], v[68:69], v[18:19], v[16:17]\n"
        "v_pk_mul_f32 v[50:51], v[50:51], v[48:49]\n"
        "v_pk_mul_f32 v[56:57], v[56:57], v[54:55]\n"
        "v_pk_mul_f32 v[62:63], v[62:63], v[60:61]\n"
        "v_pk_mul_f32 v[68:69], v[68:69], v[66:67]\n"
        "v_exp_f32 v50, v50\n"
        "v_exp_f32 v51, v51\n"
        "v_exp_f32 v56, v56\n"
        "v_exp_f32 v57, v57\n"
        "v_exp_f32 v62, v62\n"
        "v_exp_f32 v63, v63\n"
        "v_exp_f32 v68, v68\n"
        "v_exp_f32 v69, v69\n"
        "v_pk_add_f32 v[50:51], v[50:51], 1.0 op_sel_hi:[1,0]\n"
        "v_pk_add_f32 v[56:57], v[56:57], 1.0 op_sel_hi:[1,0]\n"
        "v_pk_add_f32 v[62:63], v[62:63], 1.0 op_sel_hi:[1,0]\n"
        "v_pk_add_f32 v[68:69], v[68:69], 1.0 op_sel_hi:[1,0]\n"
        "v_rcp_f32 v50, v50\n"
        "v_rcp_f32 v51, v51\n"
        "v_rcp_f32 v56, v56\n"
        "v_rcp_f32 v57, v57\n"
        "v_rcp_f32 v62, v62\n"
        "v_rcp_f32 v63, v63\n"
        "v_rcp_f32 v68, v68\n"
        "v_rcp_f32 v69, v69\n"
        "s_mov_b32 s70, 18874368\n"
        "s_mov_b32 s71, 19005440\n"
        "s_mov_b32 s72, 19136512\n"
        "s_mov_b32 s73, 19267584\n"
        "v_pk_mul_f32 v[52:53], v[50:51], v[48:49]\n"
        "v_pk_mul_f32 v[58:59], v[56:57], v[54:55]\n"
        "v_pk_mul_f32 v[64:65], v[62:63], v[60:61]\n"
        "v_pk_mul_f32 v[70:71], v[68:69], v[66:67]\n"
        "buffer_store_dwordx2 v[52:53], v2, s[44:47], s70 offen nt\n"
        "buffer_store_dwordx2 v[58:59], v2, s[44:47], s71 offen nt\n"
        "buffer_store_dwordx2 v[64:65], v2, s[44:47], s72 offen nt\n"
        "buffer_store_dwordx2 v[70:71], v2, s[44:47], s73 offen nt\n"
        "v_accvgpr_read_b32 v24, a80\n"
        "v_accvgpr_read_b32 v25, a84\n"
        "v_accvgpr_read_b32 v30, a81\n"
        "v_accvgpr_read_b32 v31, a85\n"
        "v_accvgpr_read_b32 v36, a82\n"
        "v_accvgpr_read_b32 v37, a86\n"
        "v_accvgpr_read_b32 v42, a83\n"
        "v_accvgpr_read_b32 v43, a87\n"
        "v_pk_mul_f32 v[26:27], v[24:25], v[24:25]\n"
        "v_pk_mul_f32 v[32:33], v[30:31], v[30:31]\n"
        "v_pk_mul_f32 v[38:39], v[36:37], v[36:37]\n"
        "v_pk_mul_f32 v[44:45], v[42:43], v[42:43]\n"
        "v_pk_fma_f32 v[26:27], v[26:27], v[18:19], v[16:17]\n"
        "v_pk_fma_f32 v[32:33], v[32:33], v[18:19], v[16:17]\n"
        "v_pk_fma_f32 v[38:39], v[38:39], v[18:19], v[16:17]\n"
        "v_pk_fma_f32 v[44:45], v[44:45], v[18:19], v[16:17]\n"
        "v_pk_mul_f32 v[26:27], v[26:27], v[24:25]\n"
        "v_pk_mul_f32 v[32:33], v[32:33], v[30:31]\n"
        "v_pk_mul_f32 v[38:39], v[38:39], v[36:37]\n"
        "v_pk_mul_f32 v[44:45], v[44:45], v[42:43]\n"
        "v_exp_f32 v26, v26\n"
        "v_exp_f32 v27, v27\n"
        "v_exp_f32 v32, v32\n"
        "v_exp_f32 v33, v33\n"
        "v_exp_f32 v38, v38\n"
        "v_exp_f32 v39, v39\n"
        "v_exp_f32 v44, v44\n"
        "v_exp_f32 v45, v45\n"
        "v_pk_add_f32 v[26:27], v[26:27], 1.0 op_sel_hi:[1,0]\n"
        "v_pk_add_f32 v[32:33], v[32:33], 1.0 op_sel_hi:[1,0]\n"
        "v_pk_add_f32 v[38:39], v[38:39], 1.0 op_sel_hi:[1,0]\n"
        "v_pk_add_f32 v[44:45], v[44:45], 1.0 op_sel_hi:[1,0]\n"
        "v_rcp_f32 v26, v26\n"
        "v_rcp_f32 v27, v27\n"
        "v_rcp_f32 v32, v32\n"
        "v_rcp_f32 v33, v33\n"
        "v_rcp_f32 v38, v38\n"
        "v_rcp_f32 v39, v39\n"
        "v_rcp_f32 v44, v44\n"
        "v_rcp_f32 v45, v45\n"
        "s_mov_b32 s70, 20971520\n"
        "s_mov_b32 s71, 21102592\n"
        "s_mov_b32 s72, 21233664\n"
        "s_mov_b32 s73, 21364736\n"
        "v_pk_mul_f32 v[28:29], v[26:27], v[24:25]\n"
        "v_pk_mul_f32 v[34:35], v[32:33], v[30:31]\n"
        "v_pk_mul_f32 v[40:41], v[38:39], v[36:37]\n"
        "v_pk_mul_f32 v[46:47], v[44:45], v[42:43]\n"
        "buffer_store_dwordx2 v[28:29], v2, s[44:47], s70 offen nt\n"
        "buffer_store_dwordx2 v[34:35], v2, s[44:47], s71 offen nt\n"
        "buffer_store_dwordx2 v[40:41], v2, s[44:47], s72 offen nt\n"
        "buffer_store_dwordx2 v[46:47], v2, s[44:47], s73 offen nt\n"
        "v_accvgpr_read_b32 v48, a88\n"
        "v_accvgpr_read_b32 v49, a92\n"
        "v_accvgpr_read_b32 v54, a89\n"
        "v_accvgpr_read_b32 v55, a93\n"
        "v_accvgpr_read_b32 v60, a90\n"
        "v_accvgpr_read_b32 v61, a94\n"
        "v_accvgpr_read_b32 v66, a91\n"
        "v_accvgpr_read_b32 v67, a95\n"
        "v_pk_mul_f32 v[50:51], v[48:49], v[48:49]\n"
        "v_pk_mul_f32 v[56:57], v[54:55], v[54:55]\n"
        "v_pk_mul_f32 v[62:63], v[60:61], v[60:61]\n"
        "v_pk_mul_f32 v[68:69], v[66:67], v[66:67]\n"
        "v_pk_fma_f32 v[50:51], v[50:51], v[18:19], v[16:17]\n"
        "v_pk_fma_f32 v[56:57], v[56:57], v[18:19], v[16:17]\n"
        "v_pk_fma_f32 v[62:63], v[62:63], v[18:19], v[16:17]\n"
        "v_pk_fma_f32 v[68:69], v[68:69], v[18:19], v[16:17]\n"
        "v_pk_mul_f32 v[50:51], v[50:51], v[48:49]\n"
        "v_pk_mul_f32 v[56:57], v[56:57], v[54:55]\n"
        "v_pk_mul_f32 v[62:63], v[62:63], v[60:61]\n"
        "v_pk_mul_f32 v[68:69], v[68:69], v[66:67]\n"
        "v_exp_f32 v50, v50\n"
        "v_exp_f32 v51, v51\n"
        "v_exp_f32 v56, v56\n"
        "v_exp_f32 v57, v57\n"
        "v_exp_f32 v62, v62\n"
        "v_exp_f32 v63, v63\n"
        "v_exp_f32 v68, v68\n"
        "v_exp_f32 v69, v69\n"
        "v_pk_add_f32 v[50:51], v[50:51], 1.0 op_sel_hi:[1,0]\n"
        "v_pk_add_f32 v[56:57], v[56:57], 1.0 op_sel_hi:[1,0]\n"
        "v_pk_add_f32 v[62:63], v[62:63], 1.0 op_sel_hi:[1,0]\n"
        "v_pk_add_f32 v[68:69], v[68:69], 1.0 op_sel_hi:[1,0]\n"
        "v_rcp_f32 v50, v50\n"
        "v_rcp_f32 v51, v51\n"
        "v_rcp_f32 v56, v56\n"
        "v_rcp_f32 v57, v57\n"
        "v_rcp_f32 v62, v62\n"
        "v_rcp_f32 v63, v63\n"
        "v_rcp_f32 v68, v68\n"
        "v_rcp_f32 v69, v69\n"
        "s_mov_b32 s70, 23068672\n"
        "s_mov_b32 s71, 23199744\n"
        "s_mov_b32 s72, 23330816\n"
        "s_mov_b32 s73, 23461888\n"
        "v_pk_mul_f32 v[52:53], v[50:51], v[48:49]\n"
        "v_pk_mul_f32 v[58:59], v[56:57], v[54:55]\n"
        "v_pk_mul_f32 v[64:65], v[62:63], v[60:61]\n"
        "v_pk_mul_f32 v[70:71], v[68:69], v[66:67]\n"
        "buffer_store_dwordx2 v[52:53], v2, s[44:47], s70 offen nt\n"
        "buffer_store_dwordx2 v[58:59], v2, s[44:47], s71 offen nt\n"
        "buffer_store_dwordx2 v[64:65], v2, s[44:47], s72 offen nt\n"
        "buffer_store_dwordx2 v[70:71], v2, s[44:47], s73 offen nt\n"
        "v_accvgpr_read_b32 v24, a96\n"
        "v_accvgpr_read_b32 v25, a100\n"
        "v_accvgpr_read_b32 v30, a97\n"
        "v_accvgpr_read_b32 v31, a101\n"
        "v_accvgpr_read_b32 v36, a98\n"
        "v_accvgpr_read_b32 v37, a102\n"
        "v_accvgpr_read_b32 v42, a99\n"
        "v_accvgpr_read_b32 v43, a103\n"
        "v_pk_mul_f32 v[26:27], v[24:25], v[24:25]\n"
        "v_pk_mul_f32 v[32:33], v[30:31], v[30:31]\n"
        "v_pk_mul_f32 v[38:39], v[36:37], v[36:37]\n"
        "v_pk_mul_f32 v[44:45], v[42:43], v[42:43]\n"
        "v_pk_fma_f32 v[26:27], v[26:27], v[18:19], v[16:17]\n"
        "v_pk_fma_f32 v[32:33], v[32:33], v[18:19], v[16:17]\n"
        "v_pk_fma_f32 v[38:39], v[38:39], v[18:19], v[16:17]\n"
        "v_pk_fma_f32 v[44:45], v[44:45], v[18:19], v[16:17]\n"
        "v_pk_mul_f32 v[26:27], v[26:27], v[24:25]\n"
        "v_pk_mul_f32 v[32:33], v[32:33], v[30:31]\n"
        "v_pk_mul_f32 v[38:39], v[38:39], v[36:37]\n"
        "v_pk_mul_f32 v[44:45], v[44:45], v[42:43]\n"
        "v_exp_f32 v26, v26\n"
        "v_exp_f32 v27, v27\n"
        "v_exp_f32 v32, v32\n"
        "v_exp_f32 v33, v33\n"
        "v_exp_f32 v38, v38\n"
        "v_exp_f32 v39, v39\n"
        "v_exp_f32 v44, v44\n"
        "v_exp_f32 v45, v45\n"
        "v_pk_add_f32 v[26:27], v[26:27], 1.0 op_sel_hi:[1,0]\n"
        "v_pk_add_f32 v[32:33], v[32:33], 1.0 op_sel_hi:[1,0]\n"
        "v_pk_add_f32 v[38:39], v[38:39], 1.0 op_sel_hi:[1,0]\n"
        "v_pk_add_f32 v[44:45], v[44:45], 1.0 op_sel_hi:[1,0]\n"
        "v_rcp_f32 v26, v26\n"
        "v_rcp_f32 v27, v27\n"
        "v_rcp_f32 v32, v32\n"
        "v_rcp_f32 v33, v33\n"
        "v_rcp_f32 v38, v38\n"
        "v_rcp_f32 v39, v39\n"
        "v_rcp_f32 v44, v44\n"
        "v_rcp_f32 v45, v45\n"
        "s_mov_b32 s70, 25165824\n"
        "s_mov_b32 s71, 25296896\n"
        "s_mov_b32 s72, 25427968\n"
        "s_mov_b32 s73, 25559040\n"
        "v_pk_mul_f32 v[28:29], v[26:27], v[24:25]\n"
        "v_pk_mul_f32 v[34:35], v[32:33], v[30:31]\n"
        "v_pk_mul_f32 v[40:41], v[38:39], v[36:37]\n"
        "v_pk_mul_f32 v[46:47], v[44:45], v[42:43]\n"
        "buffer_store_dwordx2 v[28:29], v2, s[44:47], s70 offen nt\n"
        "buffer_store_dwordx2 v[34:35], v2, s[44:47], s71 offen nt\n"
        "buffer_store_dwordx2 v[40:41], v2, s[44:47], s72 offen nt\n"
        "buffer_store_dwordx2 v[46:47], v2, s[44:47], s73 offen nt\n"
        "v_accvgpr_read_b32 v48, a104\n"
        "v_accvgpr_read_b32 v49, a108\n"
        "v_accvgpr_read_b32 v54, a105\n"
        "v_accvgpr_read_b32 v55, a109\n"
        "v_accvgpr_read_b32 v60, a106\n"
        "v_accvgpr_read_b32 v61, a110\n"
        "v_accvgpr_read_b32 v66, a107\n"
        "v_accvgpr_read_b32 v67, a111\n"
        "v_pk_mul_f32 v[50:51], v[48:49], v[48:49]\n"
        "v_pk_mul_f32 v[56:57], v[54:55], v[54:55]\n"
        "v_pk_mul_f32 v[62:63], v[60:61], v[60:61]\n"
        "v_pk_mul_f32 v[68:69], v[66:67], v[66:67]\n"
        "v_pk_fma_f32 v[50:51], v[50:51], v[18:19], v[16:17]\n"
        "v_pk_fma_f32 v[56:57], v[56:57], v[18:19], v[16:17]\n"
        "v_pk_fma_f32 v[62:63], v[62:63], v[18:19], v[16:17]\n"
        "v_pk_fma_f32 v[68:69], v[68:69], v[18:19], v[16:17]\n"
        "v_pk_mul_f32 v[50:51], v[50:51], v[48:49]\n"
        "v_pk_mul_f32 v[56:57], v[56:57], v[54:55]\n"
        "v_pk_mul_f32 v[62:63], v[62:63], v[60:61]\n"
        "v_pk_mul_f32 v[68:69], v[68:69], v[66:67]\n"
        "v_exp_f32 v50, v50\n"
        "v_exp_f32 v51, v51\n"
        "v_exp_f32 v56, v56\n"
        "v_exp_f32 v57, v57\n"
        "v_exp_f32 v62, v62\n"
        "v_exp_f32 v63, v63\n"
        "v_exp_f32 v68, v68\n"
        "v_exp_f32 v69, v69\n"
        "v_pk_add_f32 v[50:51], v[50:51], 1.0 op_sel_hi:[1,0]\n"
        "v_pk_add_f32 v[56:57], v[56:57], 1.0 op_sel_hi:[1,0]\n"
        "v_pk_add_f32 v[62:63], v[62:63], 1.0 op_sel_hi:[1,0]\n"
        "v_pk_add_f32 v[68:69], v[68:69], 1.0 op_sel_hi:[1,0]\n"
        "v_rcp_f32 v50, v50\n"
        "v_rcp_f32 v51, v51\n"
        "v_rcp_f32 v56, v56\n"
        "v_rcp_f32 v57, v57\n"
        "v_rcp_f32 v62, v62\n"
        "v_rcp_f32 v63, v63\n"
        "v_rcp_f32 v68, v68\n"
        "v_rcp_f32 v69, v69\n"
        "s_mov_b32 s70, 27262976\n"
        "s_mov_b32 s71, 27394048\n"
        "s_mov_b32 s72, 27525120\n"
        "s_mov_b32 s73, 27656192\n"
        "v_pk_mul_f32 v[52:53], v[50:51], v[48:49]\n"
        "v_pk_mul_f32 v[58:59], v[56:57], v[54:55]\n"
        "v_pk_mul_f32 v[64:65], v[62:63], v[60:61]\n"
        "v_pk_mul_f32 v[70:71], v[68:69], v[66:67]\n"
        "buffer_store_dwordx2 v[52:53], v2, s[44:47], s70 offen nt\n"
        "buffer_store_dwordx2 v[58:59], v2, s[44:47], s71 offen nt\n"
        "buffer_store_dwordx2 v[64:65], v2, s[44:47], s72 offen nt\n"
        "buffer_store_dwordx2 v[70:71], v2, s[44:47], s73 offen nt\n"
        "v_accvgpr_read_b32 v24, a112\n"
        "v_accvgpr_read_b32 v25, a116\n"
        "v_accvgpr_read_b32 v30, a113\n"
        "v_accvgpr_read_b32 v31, a117\n"
        "v_accvgpr_read_b32 v36, a114\n"
        "v_accvgpr_read_b32 v37, a118\n"
        "v_accvgpr_read_b32 v42, a115\n"
        "v_accvgpr_read_b32 v43, a119\n"
        "v_pk_mul_f32 v[26:27], v[24:25], v[24:25]\n"
        "v_pk_mul_f32 v[32:33], v[30:31], v[30:31]\n"
        "v_pk_mul_f32 v[38:39], v[36:37], v[36:37]\n"
        "v_pk_mul_f32 v[44:45], v[42:43], v[42:43]\n"
        "v_pk_fma_f32 v[26:27], v[26:27], v[18:19], v[16:17]\n"
        "v_pk_fma_f32 v[32:33], v[32:33], v[18:19], v[16:17]\n"
        "v_pk_fma_f32 v[38:39], v[38:39], v[18:19], v[16:17]\n"
        "v_pk_fma_f32 v[44:45], v[44:45], v[18:19], v[16:17]\n"
        "v_pk_mul_f32 v[26:27], v[26:27], v[24:25]\n"
        "v_pk_mul_f32 v[32:33], v[32:33], v[30:31]\n"
        "v_pk_mul_f32 v[38:39], v[38:39], v[36:37]\n"
        "v_pk_mul_f32 v[44:45], v[44:45], v[42:43]\n"
        "v_exp_f32 v26, v26\n"
        "v_exp_f32 v27, v27\n"
        "v_exp_f32 v32, v32\n"
        "v_exp_f32 v33, v33\n"
        "v_exp_f32 v38, v38\n"
        "v_exp_f32 v39, v39\n"
        "v_exp_f32 v44, v44\n"
        "v_exp_f32 v45, v45\n"
        "v_pk_add_f32 v[26:27], v[26:27], 1.0 op_sel_hi:[1,0]\n"
        "v_pk_add_f32 v[32:33], v[32:33], 1.0 op_sel_hi:[1,0]\n"
        "v_pk_add_f32 v[38:39], v[38:39], 1.0 op_sel_hi:[1,0]\n"
        "v_pk_add_f32 v[44:45], v[44:45], 1.0 op_sel_hi:[1,0]\n"
        "v_rcp_f32 v26, v26\n"
        "v_rcp_f32 v27, v27\n"
        "v_rcp_f32 v32, v32\n"
        "v_rcp_f32 v33, v33\n"
        "v_rcp_f32 v38, v38\n"
        "v_rcp_f32 v39, v39\n"
        "v_rcp_f32 v44, v44\n"
        "v_rcp_f32 v45, v45\n"
        "s_mov_b32 s70, 29360128\n"
        "s_mov_b32 s71, 29491200\n"
        "s_mov_b32 s72, 29622272\n"
        "s_mov_b32 s73, 29753344\n"
        "v_pk_mul_f32 v[28:29], v[26:27], v[24:25]\n"
        "v_pk_mul_f32 v[34:35], v[32:33], v[30:31]\n"
        "v_pk_mul_f32 v[40:41], v[38:39], v[36:37]\n"
        "v_pk_mul_f32 v[46:47], v[44:45], v[42:43]\n"
        "buffer_store_dwordx2 v[28:29], v2, s[44:47], s70 offen nt\n"
        "buffer_store_dwordx2 v[34:35], v2, s[44:47], s71 offen nt\n"
        "buffer_store_dwordx2 v[40:41], v2, s[44:47], s72 offen nt\n"
        "buffer_store_dwordx2 v[46:47], v2, s[44:47], s73 offen nt\n"
        "v_accvgpr_read_b32 v48, a120\n"
        "v_accvgpr_read_b32 v49, a124\n"
        "v_accvgpr_read_b32 v54, a121\n"
        "v_accvgpr_read_b32 v55, a125\n"
        "v_accvgpr_read_b32 v60, a122\n"
        "v_accvgpr_read_b32 v61, a126\n"
        "v_accvgpr_read_b32 v66, a123\n"
        "v_accvgpr_read_b32 v67, a127\n"
        "v_pk_mul_f32 v[50:51], v[48:49], v[48:49]\n"
        "v_pk_mul_f32 v[56:57], v[54:55], v[54:55]\n"
        "v_pk_mul_f32 v[62:63], v[60:61], v[60:61]\n"
        "v_pk_mul_f32 v[68:69], v[66:67], v[66:67]\n"
        "v_pk_fma_f32 v[50:51], v[50:51], v[18:19], v[16:17]\n"
        "v_pk_fma_f32 v[56:57], v[56:57], v[18:19], v[16:17]\n"
        "v_pk_fma_f32 v[62:63], v[62:63], v[18:19], v[16:17]\n"
        "v_pk_fma_f32 v[68:69], v[68:69], v[18:19], v[16:17]\n"
        "v_pk_mul_f32 v[50:51], v[50:51], v[48:49]\n"
        "v_pk_mul_f32 v[56:57], v[56:57], v[54:55]\n"
        "v_pk_mul_f32 v[62:63], v[62:63], v[60:61]\n"
        "v_pk_mul_f32 v[68:69], v[68:69], v[66:67]\n"
        "v_exp_f32 v50, v50\n"
        "v_exp_f32 v51, v51\n"
        "v_exp_f32 v56, v56\n"
        "v_exp_f32 v57, v57\n"
        "v_exp_f32 v62, v62\n"
        "v_exp_f32 v63, v63\n"
        "v_exp_f32 v68, v68\n"
        "v_exp_f32 v69, v69\n"
        "v_pk_add_f32 v[50:51], v[50:51], 1.0 op_sel_hi:[1,0]\n"
        "v_pk_add_f32 v[56:57], v[56:57], 1.0 op_sel_hi:[1,0]\n"
        "v_pk_add_f32 v[62:63], v[62:63], 1.0 op_sel_hi:[1,0]\n"
        "v_pk_add_f32 v[68:69], v[68:69], 1.0 op_sel_hi:[1,0]\n"
        "v_rcp_f32 v50, v50\n"
        "v_rcp_f32 v51, v51\n"
        "v_rcp_f32 v56, v56\n"
        "v_rcp_f32 v57, v57\n"
        "v_rcp_f32 v62, v62\n"
        "v_rcp_f32 v63, v63\n"
        "v_rcp_f32 v68, v68\n"
        "v_rcp_f32 v69, v69\n"
        "s_mov_b32 s70, 31457280\n"
        "s_mov_b32 s71, 31588352\n"
        "s_mov_b32 s72, 31719424\n"
        "s_mov_b32 s73, 31850496\n"
        "v_pk_mul_f32 v[52:53], v[50:51], v[48:49]\n"
        "v_pk_mul_f32 v[58:59], v[56:57], v[54:55]\n"
        "v_pk_mul_f32 v[64:65], v[62:63], v[60:61]\n"
        "v_pk_mul_f32 v[70:71], v[68:69], v[66:67]\n"
        "buffer_store_dwordx2 v[52:53], v2, s[44:47], s70 offen nt\n"
        "buffer_store_dwordx2 v[58:59], v2, s[44:47], s71 offen nt\n"
        "buffer_store_dwordx2 v[64:65], v2, s[44:47], s72 offen nt\n"
        "buffer_store_dwordx2 v[70:71], v2, s[44:47], s73 offen nt\n"
        "v_accvgpr_read_b32 v24, a128\n"
        "v_accvgpr_read_b32 v25, a132\n"
        "v_accvgpr_read_b32 v30, a129\n"
        "v_accvgpr_read_b32 v31, a133\n"
        "v_accvgpr_read_b32 v36, a130\n"
        "v_accvgpr_read_b32 v37, a134\n"
        "v_accvgpr_read_b32 v42, a131\n"
        "v_accvgpr_read_b32 v43, a135\n"
        "v_pk_mul_f32 v[26:27], v[24:25], v[24:25]\n"
        "v_pk_mul_f32 v[32:33], v[30:31], v[30:31]\n"
        "v_pk_mul_f32 v[38:39], v[36:37], v[36:37]\n"
        "v_pk_mul_f32 v[44:45], v[42:43], v[42:43]\n"
        "v_pk_fma_f32 v[26:27], v[26:27], v[18:19], v[16:17]\n"
        "v_pk_fma_f32 v[32:33], v[32:33], v[18:19], v[16:17]\n"
        "v_pk_fma_f32 v[38:39], v[38:39], v[18:19], v[16:17]\n"
        "v_pk_fma_f32 v[44:45], v[44:45], v[18:19], v[16:17]\n"
        "v_pk_mul_f32 v[26:27], v[26:27], v[24:25]\n"
        "v_pk_mul_f32 v[32:33], v[32:33], v[30:31]\n"
        "v_pk_mul_f32 v[38:39], v[38:39], v[36:37]\n"
        "v_pk_mul_f32 v[44:45], v[44:45], v[42:43]\n"
        "v_exp_f32 v26, v26\n"
        "v_exp_f32 v27, v27\n"
        "v_exp_f32 v32, v32\n"
        "v_exp_f32 v33, v33\n"
        "v_exp_f32 v38, v38\n"
        "v_exp_f32 v39, v39\n"
        "v_exp_f32 v44, v44\n"
        "v_exp_f32 v45, v45\n"
        "v_pk_add_f32 v[26:27], v[26:27], 1.0 op_sel_hi:[1,0]\n"
        "v_pk_add_f32 v[32:33], v[32:33], 1.0 op_sel_hi:[1,0]\n"
        "v_pk_add_f32 v[38:39], v[38:39], 1.0 op_sel_hi:[1,0]\n"
        "v_pk_add_f32 v[44:45], v[44:45], 1.0 op_sel_hi:[1,0]\n"
        "v_rcp_f32 v26, v26\n"
        "v_rcp_f32 v27, v27\n"
        "v_rcp_f32 v32, v32\n"
        "v_rcp_f32 v33, v33\n"
        "v_rcp_f32 v38, v38\n"
        "v_rcp_f32 v39, v39\n"
        "v_rcp_f32 v44, v44\n"
        "v_rcp_f32 v45, v45\n"
        "s_mov_b32 s70, 33554432\n"
        "s_mov_b32 s71, 33685504\n"
        "s_mov_b32 s72, 33816576\n"
        "s_mov_b32 s73, 33947648\n"
        "v_pk_mul_f32 v[28:29], v[26:27], v[24:25]\n"
        "v_pk_mul_f32 v[34:35], v[32:33], v[30:31]\n"
        "v_pk_mul_f32 v[40:41], v[38:39], v[36:37]\n"
        "v_pk_mul_f32 v[46:47], v[44:45], v[42:43]\n"
        "buffer_store_dwordx2 v[28:29], v2, s[44:47], s70 offen nt\n"
        "buffer_store_dwordx2 v[34:35], v2, s[44:47], s71 offen nt\n"
        "buffer_store_dwordx2 v[40:41], v2, s[44:47], s72 offen nt\n"
        "buffer_store_dwordx2 v[46:47], v2, s[44:47], s73 offen nt\n"
        "v_accvgpr_read_b32 v48, a136\n"
        "v_accvgpr_read_b32 v49, a140\n"
        "v_accvgpr_read_b32 v54, a137\n"
        "v_accvgpr_read_b32 v55, a141\n"
        "v_accvgpr_read_b32 v60, a138\n"
        "v_accvgpr_read_b32 v61, a142\n"
        "v_accvgpr_read_b32 v66, a139\n"
        "v_accvgpr_read_b32 v67, a143\n"
        "v_pk_mul_f32 v[50:51], v[48:49], v[48:49]\n"
        "v_pk_mul_f32 v[56:57], v[54:55], v[54:55]\n"
        "v_pk_mul_f32 v[62:63], v[60:61], v[60:61]\n"
        "v_pk_mul_f32 v[68:69], v[66:67], v[66:67]\n"
        "v_pk_fma_f32 v[50:51], v[50:51], v[18:19], v[16:17]\n"
        "v_pk_fma_f32 v[56:57], v[56:57], v[18:19], v[16:17]\n"
        "v_pk_fma_f32 v[62:63], v[62:63], v[18:19], v[16:17]\n"
        "v_pk_fma_f32 v[68:69], v[68:69], v[18:19], v[16:17]\n"
        "v_pk_mul_f32 v[50:51], v[50:51], v[48:49]\n"
        "v_pk_mul_f32 v[56:57], v[56:57], v[54:55]\n"
        "v_pk_mul_f32 v[62:63], v[62:63], v[60:61]\n"
        "v_pk_mul_f32 v[68:69], v[68:69], v[66:67]\n"
        "v_exp_f32 v50, v50\n"
        "v_exp_f32 v51, v51\n"
        "v_exp_f32 v56, v56\n"
        "v_exp_f32 v57, v57\n"
        "v_exp_f32 v62, v62\n"
        "v_exp_f32 v63, v63\n"
        "v_exp_f32 v68, v68\n"
        "v_exp_f32 v69, v69\n"
        "v_pk_add_f32 v[50:51], v[50:51], 1.0 op_sel_hi:[1,0]\n"
        "v_pk_add_f32 v[56:57], v[56:57], 1.0 op_sel_hi:[1,0]\n"
        "v_pk_add_f32 v[62:63], v[62:63], 1.0 op_sel_hi:[1,0]\n"
        "v_pk_add_f32 v[68:69], v[68:69], 1.0 op_sel_hi:[1,0]\n"
        "v_rcp_f32 v50, v50\n"
        "v_rcp_f32 v51, v51\n"
        "v_rcp_f32 v56, v56\n"
        "v_rcp_f32 v57, v57\n"
        "v_rcp_f32 v62, v62\n"
        "v_rcp_f32 v63, v63\n"
        "v_rcp_f32 v68, v68\n"
        "v_rcp_f32 v69, v69\n"
        "s_mov_b32 s70, 35651584\n"
        "s_mov_b32 s71, 35782656\n"
        "s_mov_b32 s72, 35913728\n"
        "s_mov_b32 s73, 36044800\n"
        "v_pk_mul_f32 v[52:53], v[50:51], v[48:49]\n"
        "v_pk_mul_f32 v[58:59], v[56:57], v[54:55]\n"
        "v_pk_mul_f32 v[64:65], v[62:63], v[60:61]\n"
        "v_pk_mul_f32 v[70:71], v[68:69], v[66:67]\n"
        "buffer_store_dwordx2 v[52:53], v2, s[44:47], s70 offen nt\n"
        "buffer_store_dwordx2 v[58:59], v2, s[44:47], s71 offen nt\n"
        "buffer_store_dwordx2 v[64:65], v2, s[44:47], s72 offen nt\n"
        "buffer_store_dwordx2 v[70:71], v2, s[44:47], s73 offen nt\n"
        "v_accvgpr_read_b32 v24, a144\n"
        "v_accvgpr_read_b32 v25, a148\n"
        "v_accvgpr_read_b32 v30, a145\n"
        "v_accvgpr_read_b32 v31, a149\n"
        "v_accvgpr_read_b32 v36, a146\n"
        "v_accvgpr_read_b32 v37, a150\n"
        "v_accvgpr_read_b32 v42, a147\n"
        "v_accvgpr_read_b32 v43, a151\n"
        "v_pk_mul_f32 v[26:27], v[24:25], v[24:25]\n"
        "v_pk_mul_f32 v[32:33], v[30:31], v[30:31]\n"
        "v_pk_mul_f32 v[38:39], v[36:37], v[36:37]\n"
        "v_pk_mul_f32 v[44:45], v[42:43], v[42:43]\n"
        "v_pk_fma_f32 v[26:27], v[26:27], v[18:19], v[16:17]\n"
        "v_pk_fma_f32 v[32:33], v[32:33], v[18:19], v[16:17]\n"
        "v_pk_fma_f32 v[38:39], v[38:39], v[18:19], v[16:17]\n"
        "v_pk_fma_f32 v[44:45], v[44:45], v[18:19], v[16:17]\n"
        "v_pk_mul_f32 v[26:27], v[26:27], v[24:25]\n"
        "v_pk_mul_f32 v[32:33], v[32:33], v[30:31]\n"
        "v_pk_mul_f32 v[38:39], v[38:39], v[36:37]\n"
        "v_pk_mul_f32 v[44:45], v[44:45], v[42:43]\n"
        "v_exp_f32 v26, v26\n"
        "v_exp_f32 v27, v27\n"
        "v_exp_f32 v32, v32\n"
        "v_exp_f32 v33, v33\n"
        "v_exp_f32 v38, v38\n"
        "v_exp_f32 v39, v39\n"
        "v_exp_f32 v44, v44\n"
        "v_exp_f32 v45, v45\n"
        "v_pk_add_f32 v[26:27], v[26:27], 1.0 op_sel_hi:[1,0]\n"
        "v_pk_add_f32 v[32:33], v[32:33], 1.0 op_sel_hi:[1,0]\n"
        "v_pk_add_f32 v[38:39], v[38:39], 1.0 op_sel_hi:[1,0]\n"
        "v_pk_add_f32 v[44:45], v[44:45], 1.0 op_sel_hi:[1,0]\n"
        "v_rcp_f32 v26, v26\n"
        "v_rcp_f32 v27, v27\n"
        "v_rcp_f32 v32, v32\n"
        "v_rcp_f32 v33, v33\n"
        "v_rcp_f32 v38, v38\n"
        "v_rcp_f32 v39, v39\n"
        "v_rcp_f32 v44, v44\n"
        "v_rcp_f32 v45, v45\n"
        "s_mov_b32 s70, 37748736\n"
        "s_mov_b32 s71, 37879808\n"
        "s_mov_b32 s72, 38010880\n"
        "s_mov_b32 s73, 38141952\n"
        "v_pk_mul_f32 v[28:29], v[26:27], v[24:25]\n"
        "v_pk_mul_f32 v[34:35], v[32:33], v[30:31]\n"
        "v_pk_mul_f32 v[40:41], v[38:39], v[36:37]\n"
        "v_pk_mul_f32 v[46:47], v[44:45], v[42:43]\n"
        "buffer_store_dwordx2 v[28:29], v2, s[44:47], s70 offen nt\n"
        "buffer_store_dwordx2 v[34:35], v2, s[44:47], s71 offen nt\n"
        "buffer_store_dwordx2 v[40:41], v2, s[44:47], s72 offen nt\n"
        "buffer_store_dwordx2 v[46:47], v2, s[44:47], s73 offen nt\n"
        "v_accvgpr_read_b32 v48, a152\n"
        "v_accvgpr_read_b32 v49, a156\n"
        "v_accvgpr_read_b32 v54, a153\n"
        "v_accvgpr_read_b32 v55, a157\n"
        "v_accvgpr_read_b32 v60, a154\n"
        "v_accvgpr_read_b32 v61, a158\n"
        "v_accvgpr_read_b32 v66, a155\n"
        "v_accvgpr_read_b32 v67, a159\n"
        "v_pk_mul_f32 v[50:51], v[48:49], v[48:49]\n"
        "v_pk_mul_f32 v[56:57], v[54:55], v[54:55]\n"
        "v_pk_mul_f32 v[62:63], v[60:61], v[60:61]\n"
        "v_pk_mul_f32 v[68:69], v[66:67], v[66:67]\n"
        "v_pk_fma_f32 v[50:51], v[50:51], v[18:19], v[16:17]\n"
        "v_pk_fma_f32 v[56:57], v[56:57], v[18:19], v[16:17]\n"
        "v_pk_fma_f32 v[62:63], v[62:63], v[18:19], v[16:17]\n"
        "v_pk_fma_f32 v[68:69], v[68:69], v[18:19], v[16:17]\n"
        "v_pk_mul_f32 v[50:51], v[50:51], v[48:49]\n"
        "v_pk_mul_f32 v[56:57], v[56:57], v[54:55]\n"
        "v_pk_mul_f32 v[62:63], v[62:63], v[60:61]\n"
        "v_pk_mul_f32 v[68:69], v[68:69], v[66:67]\n"
        "v_exp_f32 v50, v50\n"
        "v_exp_f32 v51, v51\n"
        "v_exp_f32 v56, v56\n"
        "v_exp_f32 v57, v57\n"
        "v_exp_f32 v62, v62\n"
        "v_exp_f32 v63, v63\n"
        "v_exp_f32 v68, v68\n"
        "v_exp_f32 v69, v69\n"
        "v_pk_add_f32 v[50:51], v[50:51], 1.0 op_sel_hi:[1,0]\n"
        "v_pk_add_f32 v[56:57], v[56:57], 1.0 op_sel_hi:[1,0]\n"
        "v_pk_add_f32 v[62:63], v[62:63], 1.0 op_sel_hi:[1,0]\n"
        "v_pk_add_f32 v[68:69], v[68:69], 1.0 op_sel_hi:[1,0]\n"
        "v_rcp_f32 v50, v50\n"
        "v_rcp_f32 v51, v51\n"
        "v_rcp_f32 v56, v56\n"
        "v_rcp_f32 v57, v57\n"
        "v_rcp_f32 v62, v62\n"
        "v_rcp_f32 v63, v63\n"
        "v_rcp_f32 v68, v68\n"
        "v_rcp_f32 v69, v69\n"
        "s_mov_b32 s70, 39845888\n"
        "s_mov_b32 s71, 39976960\n"
        "s_mov_b32 s72, 40108032\n"
        "s_mov_b32 s73, 40239104\n"
        "v_pk_mul_f32 v[52:53], v[50:51], v[48:49]\n"
        "v_pk_mul_f32 v[58:59], v[56:57], v[54:55]\n"
        "v_pk_mul_f32 v[64:65], v[62:63], v[60:61]\n"
        "v_pk_mul_f32 v[70:71], v[68:69], v[66:67]\n"
        "buffer_store_dwordx2 v[52:53], v2, s[44:47], s70 offen nt\n"
        "buffer_store_dwordx2 v[58:59], v2, s[44:47], s71 offen nt\n"
        "buffer_store_dwordx2 v[64:65], v2, s[44:47], s72 offen nt\n"
        "buffer_store_dwordx2 v[70:71], v2, s[44:47], s73 offen nt\n"
        "v_accvgpr_read_b32 v24, a160\n"
        "v_accvgpr_read_b32 v25, a164\n"
        "v_accvgpr_read_b32 v30, a161\n"
        "v_accvgpr_read_b32 v31, a165\n"
        "v_accvgpr_read_b32 v36, a162\n"
        "v_accvgpr_read_b32 v37, a166\n"
        "v_accvgpr_read_b32 v42, a163\n"
        "v_accvgpr_read_b32 v43, a167\n"
        "v_pk_mul_f32 v[26:27], v[24:25], v[24:25]\n"
        "v_pk_mul_f32 v[32:33], v[30:31], v[30:31]\n"
        "v_pk_mul_f32 v[38:39], v[36:37], v[36:37]\n"
        "v_pk_mul_f32 v[44:45], v[42:43], v[42:43]\n"
        "v_pk_fma_f32 v[26:27], v[26:27], v[18:19], v[16:17]\n"
        "v_pk_fma_f32 v[32:33], v[32:33], v[18:19], v[16:17]\n"
        "v_pk_fma_f32 v[38:39], v[38:39], v[18:19], v[16:17]\n"
        "v_pk_fma_f32 v[44:45], v[44:45], v[18:19], v[16:17]\n"
        "v_pk_mul_f32 v[26:27], v[26:27], v[24:25]\n"
        "v_pk_mul_f32 v[32:33], v[32:33], v[30:31]\n"
        "v_pk_mul_f32 v[38:39], v[38:39], v[36:37]\n"
        "v_pk_mul_f32 v[44:45], v[44:45], v[42:43]\n"
        "v_exp_f32 v26, v26\n"
        "v_exp_f32 v27, v27\n"
        "v_exp_f32 v32, v32\n"
        "v_exp_f32 v33, v33\n"
        "v_exp_f32 v38, v38\n"
        "v_exp_f32 v39, v39\n"
        "v_exp_f32 v44, v44\n"
        "v_exp_f32 v45, v45\n"
        "v_pk_add_f32 v[26:27], v[26:27], 1.0 op_sel_hi:[1,0]\n"
        "v_pk_add_f32 v[32:33], v[32:33], 1.0 op_sel_hi:[1,0]\n"
        "v_pk_add_f32 v[38:39], v[38:39], 1.0 op_sel_hi:[1,0]\n"
        "v_pk_add_f32 v[44:45], v[44:45], 1.0 op_sel_hi:[1,0]\n"
        "v_rcp_f32 v26, v26\n"
        "v_rcp_f32 v27, v27\n"
        "v_rcp_f32 v32, v32\n"
        "v_rcp_f32 v33, v33\n"
        "v_rcp_f32 v38, v38\n"
        "v_rcp_f32 v39, v39\n"
        "v_rcp_f32 v44, v44\n"
        "v_rcp_f32 v45, v45\n"
        "s_mov_b32 s70, 41943040\n"
        "s_mov_b32 s71, 42074112\n"
        "s_mov_b32 s72, 42205184\n"
        "s_mov_b32 s73, 42336256\n"
        "v_pk_mul_f32 v[28:29], v[26:27], v[24:25]\n"
        "v_pk_mul_f32 v[34:35], v[32:33], v[30:31]\n"
        "v_pk_mul_f32 v[40:41], v[38:39], v[36:37]\n"
        "v_pk_mul_f32 v[46:47], v[44:45], v[42:43]\n"
        "buffer_store_dwordx2 v[28:29], v2, s[44:47], s70 offen nt\n"
        "buffer_store_dwordx2 v[34:35], v2, s[44:47], s71 offen nt\n"
        "buffer_store_dwordx2 v[40:41], v2, s[44:47], s72 offen nt\n"
        "buffer_store_dwordx2 v[46:47], v2, s[44:47], s73 offen nt\n"
        "v_accvgpr_read_b32 v48, a168\n"
        "v_accvgpr_read_b32 v49, a172\n"
        "v_accvgpr_read_b32 v54, a169\n"
        "v_accvgpr_read_b32 v55, a173\n"
        "v_accvgpr_read_b32 v60, a170\n"
        "v_accvgpr_read_b32 v61, a174\n"
        "v_accvgpr_read_b32 v66, a171\n"
        "v_accvgpr_read_b32 v67, a175\n"
        "v_pk_mul_f32 v[50:51], v[48:49], v[48:49]\n"
        "v_pk_mul_f32 v[56:57], v[54:55], v[54:55]\n"
        "v_pk_mul_f32 v[62:63], v[60:61], v[60:61]\n"
        "v_pk_mul_f32 v[68:69], v[66:67], v[66:67]\n"
        "v_pk_fma_f32 v[50:51], v[50:51], v[18:19], v[16:17]\n"
        "v_pk_fma_f32 v[56:57], v[56:57], v[18:19], v[16:17]\n"
        "v_pk_fma_f32 v[62:63], v[62:63], v[18:19], v[16:17]\n"
        "v_pk_fma_f32 v[68:69], v[68:69], v[18:19], v[16:17]\n"
        "v_pk_mul_f32 v[50:51], v[50:51], v[48:49]\n"
        "v_pk_mul_f32 v[56:57], v[56:57], v[54:55]\n"
        "v_pk_mul_f32 v[62:63], v[62:63], v[60:61]\n"
        "v_pk_mul_f32 v[68:69], v[68:69], v[66:67]\n"
        "v_exp_f32 v50, v50\n"
        "v_exp_f32 v51, v51\n"
        "v_exp_f32 v56, v56\n"
        "v_exp_f32 v57, v57\n"
        "v_exp_f32 v62, v62\n"
        "v_exp_f32 v63, v63\n"
        "v_exp_f32 v68, v68\n"
        "v_exp_f32 v69, v69\n"
        "v_pk_add_f32 v[50:51], v[50:51], 1.0 op_sel_hi:[1,0]\n"
        "v_pk_add_f32 v[56:57], v[56:57], 1.0 op_sel_hi:[1,0]\n"
        "v_pk_add_f32 v[62:63], v[62:63], 1.0 op_sel_hi:[1,0]\n"
        "v_pk_add_f32 v[68:69], v[68:69], 1.0 op_sel_hi:[1,0]\n"
        "v_rcp_f32 v50, v50\n"
        "v_rcp_f32 v51, v51\n"
        "v_rcp_f32 v56, v56\n"
        "v_rcp_f32 v57, v57\n"
        "v_rcp_f32 v62, v62\n"
        "v_rcp_f32 v63, v63\n"
        "v_rcp_f32 v68, v68\n"
        "v_rcp_f32 v69, v69\n"
        "s_mov_b32 s70, 44040192\n"
        "s_mov_b32 s71, 44171264\n"
        "s_mov_b32 s72, 44302336\n"
        "s_mov_b32 s73, 44433408\n"
        "v_pk_mul_f32 v[52:53], v[50:51], v[48:49]\n"
        "v_pk_mul_f32 v[58:59], v[56:57], v[54:55]\n"
        "v_pk_mul_f32 v[64:65], v[62:63], v[60:61]\n"
        "v_pk_mul_f32 v[70:71], v[68:69], v[66:67]\n"
        "buffer_store_dwordx2 v[52:53], v2, s[44:47], s70 offen nt\n"
        "buffer_store_dwordx2 v[58:59], v2, s[44:47], s71 offen nt\n"
        "buffer_store_dwordx2 v[64:65], v2, s[44:47], s72 offen nt\n"
        "buffer_store_dwordx2 v[70:71], v2, s[44:47], s73 offen nt\n"
        "v_accvgpr_read_b32 v24, a176\n"
        "v_accvgpr_read_b32 v25, a180\n"
        "v_accvgpr_read_b32 v30, a177\n"
        "v_accvgpr_read_b32 v31, a181\n"
        "v_accvgpr_read_b32 v36, a178\n"
        "v_accvgpr_read_b32 v37, a182\n"
        "v_accvgpr_read_b32 v42, a179\n"
        "v_accvgpr_read_b32 v43, a183\n"
        "v_pk_mul_f32 v[26:27], v[24:25], v[24:25]\n"
        "v_pk_mul_f32 v[32:33], v[30:31], v[30:31]\n"
        "v_pk_mul_f32 v[38:39], v[36:37], v[36:37]\n"
        "v_pk_mul_f32 v[44:45], v[42:43], v[42:43]\n"
        "v_pk_fma_f32 v[26:27], v[26:27], v[18:19], v[16:17]\n"
        "v_pk_fma_f32 v[32:33], v[32:33], v[18:19], v[16:17]\n"
        "v_pk_fma_f32 v[38:39], v[38:39], v[18:19], v[16:17]\n"
        "v_pk_fma_f32 v[44:45], v[44:45], v[18:19], v[16:17]\n"
        "v_pk_mul_f32 v[26:27], v[26:27], v[24:25]\n"
        "v_pk_mul_f32 v[32:33], v[32:33], v[30:31]\n"
        "v_pk_mul_f32 v[38:39], v[38:39], v[36:37]\n"
        "v_pk_mul_f32 v[44:45], v[44:45], v[42:43]\n"
        "v_exp_f32 v26, v26\n"
        "v_exp_f32 v27, v27\n"
        "v_exp_f32 v32, v32\n"
        "v_exp_f32 v33, v33\n"
        "v_exp_f32 v38, v38\n"
        "v_exp_f32 v39, v39\n"
        "v_exp_f32 v44, v44\n"
        "v_exp_f32 v45, v45\n"
        "v_pk_add_f32 v[26:27], v[26:27], 1.0 op_sel_hi:[1,0]\n"
        "v_pk_add_f32 v[32:33], v[32:33], 1.0 op_sel_hi:[1,0]\n"
        "v_pk_add_f32 v[38:39], v[38:39], 1.0 op_sel_hi:[1,0]\n"
        "v_pk_add_f32 v[44:45], v[44:45], 1.0 op_sel_hi:[1,0]\n"
        "v_rcp_f32 v26, v26\n"
        "v_rcp_f32 v27, v27\n"
        "v_rcp_f32 v32, v32\n"
        "v_rcp_f32 v33, v33\n"
        "v_rcp_f32 v38, v38\n"
        "v_rcp_f32 v39, v39\n"
        "v_rcp_f32 v44, v44\n"
        "v_rcp_f32 v45, v45\n"
        "s_mov_b32 s70, 46137344\n"
        "s_mov_b32 s71, 46268416\n"
        "s_mov_b32 s72, 46399488\n"
        "s_mov_b32 s73, 46530560\n"
        "v_pk_mul_f32 v[28:29], v[26:27], v[24:25]\n"
        "v_pk_mul_f32 v[34:35], v[32:33], v[30:31]\n"
        "v_pk_mul_f32 v[40:41], v[38:39], v[36:37]\n"
        "v_pk_mul_f32 v[46:47], v[44:45], v[42:43]\n"
        "buffer_store_dwordx2 v[28:29], v2, s[44:47], s70 offen nt\n"
        "buffer_store_dwordx2 v[34:35], v2, s[44:47], s71 offen nt\n"
        "buffer_store_dwordx2 v[40:41], v2, s[44:47], s72 offen nt\n"
        "buffer_store_dwordx2 v[46:47], v2, s[44:47], s73 offen nt\n"
        "v_accvgpr_read_b32 v48, a184\n"
        "v_accvgpr_read_b32 v49, a188\n"
        "v_accvgpr_read_b32 v54, a185\n"
        "v_accvgpr_read_b32 v55, a189\n"
        "v_accvgpr_read_b32 v60, a186\n"
        "v_accvgpr_read_b32 v61, a190\n"
        "v_accvgpr_read_b32 v66, a187\n"
        "v_accvgpr_read_b32 v67, a191\n"
        "v_pk_mul_f32 v[50:51], v[48:49], v[48:49]\n"
        "v_pk_mul_f32 v[56:57], v[54:55], v[54:55]\n"
        "v_pk_mul_f32 v[62:63], v[60:61], v[60:61]\n"
        "v_pk_mul_f32 v[68:69], v[66:67], v[66:67]\n"
        "v_pk_fma_f32 v[50:51], v[50:51], v[18:19], v[16:17]\n"
        "v_pk_fma_f32 v[56:57], v[56:57], v[18:19], v[16:17]\n"
        "v_pk_fma_f32 v[62:63], v[62:63], v[18:19], v[16:17]\n"
        "v_pk_fma_f32 v[68:69], v[68:69], v[18:19], v[16:17]\n"
        "v_pk_mul_f32 v[50:51], v[50:51], v[48:49]\n"
        "v_pk_mul_f32 v[56:57], v[56:57], v[54:55]\n"
        "v_pk_mul_f32 v[62:63], v[62:63], v[60:61]\n"
        "v_pk_mul_f32 v[68:69], v[68:69], v[66:67]\n"
        "v_exp_f32 v50, v50\n"
        "v_exp_f32 v51, v51\n"
        "v_exp_f32 v56, v56\n"
        "v_exp_f32 v57, v57\n"
        "v_exp_f32 v62, v62\n"
        "v_exp_f32 v63, v63\n"
        "v_exp_f32 v68, v68\n"
        "v_exp_f32 v69, v69\n"
        "v_pk_add_f32 v[50:51], v[50:51], 1.0 op_sel_hi:[1,0]\n"
        "v_pk_add_f32 v[56:57], v[56:57], 1.0 op_sel_hi:[1,0]\n"
        "v_pk_add_f32 v[62:63], v[62:63], 1.0 op_sel_hi:[1,0]\n"
        "v_pk_add_f32 v[68:69], v[68:69], 1.0 op_sel_hi:[1,0]\n"
        "v_rcp_f32 v50, v50\n"
        "v_rcp_f32 v51, v51\n"
        "v_rcp_f32 v56, v56\n"
        "v_rcp_f32 v57, v57\n"
        "v_rcp_f32 v62, v62\n"
        "v_rcp_f32 v63, v63\n"
        "v_rcp_f32 v68, v68\n"
        "v_rcp_f32 v69, v69\n"
        "s_mov_b32 s70, 48234496\n"
        "s_mov_b32 s71, 48365568\n"
        "s_mov_b32 s72, 48496640\n"
        "s_mov_b32 s73, 48627712\n"
        "v_pk_mul_f32 v[52:53], v[50:51], v[48:49]\n"
        "v_pk_mul_f32 v[58:59], v[56:57], v[54:55]\n"
        "v_pk_mul_f32 v[64:65], v[62:63], v[60:61]\n"
        "v_pk_mul_f32 v[70:71], v[68:69], v[66:67]\n"
        "buffer_store_dwordx2 v[52:53], v2, s[44:47], s70 offen nt\n"
        "buffer_store_dwordx2 v[58:59], v2, s[44:47], s71 offen nt\n"
        "buffer_store_dwordx2 v[64:65], v2, s[44:47], s72 offen nt\n"
        "buffer_store_dwordx2 v[70:71], v2, s[44:47], s73 offen nt\n"
        "v_accvgpr_read_b32 v24, a192\n"
        "v_accvgpr_read_b32 v25, a196\n"
        "v_accvgpr_read_b32 v30, a193\n"
        "v_accvgpr_read_b32 v31, a197\n"
        "v_accvgpr_read_b32 v36, a194\n"
        "v_accvgpr_read_b32 v37, a198\n"
        "v_accvgpr_read_b32 v42, a195\n"
        "v_accvgpr_read_b32 v43, a199\n"
        "v_pk_mul_f32 v[26:27], v[24:25], v[24:25]\n"
        "v_pk_mul_f32 v[32:33], v[30:31], v[30:31]\n"
        "v_pk_mul_f32 v[38:39], v[36:37], v[36:37]\n"
        "v_pk_mul_f32 v[44:45], v[42:43], v[42:43]\n"
        "v_pk_fma_f32 v[26:27], v[26:27], v[18:19], v[16:17]\n"
        "v_pk_fma_f32 v[32:33], v[32:33], v[18:19], v[16:17]\n"
        "v_pk_fma_f32 v[38:39], v[38:39], v[18:19], v[16:17]\n"
        "v_pk_fma_f32 v[44:45], v[44:45], v[18:19], v[16:17]\n"
        "v_pk_mul_f32 v[26:27], v[26:27], v[24:25]\n"
        "v_pk_mul_f32 v[32:33], v[32:33], v[30:31]\n"
        "v_pk_mul_f32 v[38:39], v[38:39], v[36:37]\n"
        "v_pk_mul_f32 v[44:45], v[44:45], v[42:43]\n"
        "v_exp_f32 v26, v26\n"
        "v_exp_f32 v27, v27\n"
        "v_exp_f32 v32, v32\n"
        "v_exp_f32 v33, v33\n"
        "v_exp_f32 v38, v38\n"
        "v_exp_f32 v39, v39\n"
        "v_exp_f32 v44, v44\n"
        "v_exp_f32 v45, v45\n"
        "v_pk_add_f32 v[26:27], v[26:27], 1.0 op_sel_hi:[1,0]\n"
        "v_pk_add_f32 v[32:33], v[32:33], 1.0 op_sel_hi:[1,0]\n"
        "v_pk_add_f32 v[38:39], v[38:39], 1.0 op_sel_hi:[1,0]\n"
        "v_pk_add_f32 v[44:45], v[44:45], 1.0 op_sel_hi:[1,0]\n"
        "v_rcp_f32 v26, v26\n"
        "v_rcp_f32 v27, v27\n"
        "v_rcp_f32 v32, v32\n"
        "v_rcp_f32 v33, v33\n"
        "v_rcp_f32 v38, v38\n"
        "v_rcp_f32 v39, v39\n"
        "v_rcp_f32 v44, v44\n"
        "v_rcp_f32 v45, v45\n"
        "s_mov_b32 s70, 50331648\n"
        "s_mov_b32 s71, 50462720\n"
        "s_mov_b32 s72, 50593792\n"
        "s_mov_b32 s73, 50724864\n"
        "v_pk_mul_f32 v[28:29], v[26:27], v[24:25]\n"
        "v_pk_mul_f32 v[34:35], v[32:33], v[30:31]\n"
        "v_pk_mul_f32 v[40:41], v[38:39], v[36:37]\n"
        "v_pk_mul_f32 v[46:47], v[44:45], v[42:43]\n"
        "buffer_store_dwordx2 v[28:29], v2, s[44:47], s70 offen nt\n"
        "buffer_store_dwordx2 v[34:35], v2, s[44:47], s71 offen nt\n"
        "buffer_store_dwordx2 v[40:41], v2, s[44:47], s72 offen nt\n"
        "buffer_store_dwordx2 v[46:47], v2, s[44:47], s73 offen nt\n"
        "v_accvgpr_read_b32 v48, a200\n"
        "v_accvgpr_read_b32 v49, a204\n"
        "v_accvgpr_read_b32 v54, a201\n"
        "v_accvgpr_read_b32 v55, a205\n"
        "v_accvgpr_read_b32 v60, a202\n"
        "v_accvgpr_read_b32 v61, a206\n"
        "v_accvgpr_read_b32 v66, a203\n"
        "v_accvgpr_read_b32 v67, a207\n"
        "v_pk_mul_f32 v[50:51], v[48:49], v[48:49]\n"
        "v_pk_mul_f32 v[56:57], v[54:55], v[54:55]\n"
        "v_pk_mul_f32 v[62:63], v[60:61], v[60:61]\n"
        "v_pk_mul_f32 v[68:69], v[66:67], v[66:67]\n"
        "v_pk_fma_f32 v[50:51], v[50:51], v[18:19], v[16:17]\n"
        "v_pk_fma_f32 v[56:57], v[56:57], v[18:19], v[16:17]\n"
        "v_pk_fma_f32 v[62:63], v[62:63], v[18:19], v[16:17]\n"
        "v_pk_fma_f32 v[68:69], v[68:69], v[18:19], v[16:17]\n"
        "v_pk_mul_f32 v[50:51], v[50:51], v[48:49]\n"
        "v_pk_mul_f32 v[56:57], v[56:57], v[54:55]\n"
        "v_pk_mul_f32 v[62:63], v[62:63], v[60:61]\n"
        "v_pk_mul_f32 v[68:69], v[68:69], v[66:67]\n"
        "v_exp_f32 v50, v50\n"
        "v_exp_f32 v51, v51\n"
        "v_exp_f32 v56, v56\n"
        "v_exp_f32 v57, v57\n"
        "v_exp_f32 v62, v62\n"
        "v_exp_f32 v63, v63\n"
        "v_exp_f32 v68, v68\n"
        "v_exp_f32 v69, v69\n"
        "v_pk_add_f32 v[50:51], v[50:51], 1.0 op_sel_hi:[1,0]\n"
        "v_pk_add_f32 v[56:57], v[56:57], 1.0 op_sel_hi:[1,0]\n"
        "v_pk_add_f32 v[62:63], v[62:63], 1.0 op_sel_hi:[1,0]\n"
        "v_pk_add_f32 v[68:69], v[68:69], 1.0 op_sel_hi:[1,0]\n"
        "v_rcp_f32 v50, v50\n"
        "v_rcp_f32 v51, v51\n"
        "v_rcp_f32 v56, v56\n"
        "v_rcp_f32 v57, v57\n"
        "v_rcp_f32 v62, v62\n"
        "v_rcp_f32 v63, v63\n"
        "v_rcp_f32 v68, v68\n"
        "v_rcp_f32 v69, v69\n"
        "s_mov_b32 s70, 52428800\n"
        "s_mov_b32 s71, 52559872\n"
        "s_mov_b32 s72, 52690944\n"
        "s_mov_b32 s73, 52822016\n"
        "v_pk_mul_f32 v[52:53], v[50:51], v[48:49]\n"
        "v_pk_mul_f32 v[58:59], v[56:57], v[54:55]\n"
        "v_pk_mul_f32 v[64:65], v[62:63], v[60:61]\n"
        "v_pk_mul_f32 v[70:71], v[68:69], v[66:67]\n"
        "buffer_store_dwordx2 v[52:53], v2, s[44:47], s70 offen nt\n"
        "buffer_store_dwordx2 v[58:59], v2, s[44:47], s71 offen nt\n"
        "buffer_store_dwordx2 v[64:65], v2, s[44:47], s72 offen nt\n"
        "buffer_store_dwordx2 v[70:71], v2, s[44:47], s73 offen nt\n"
        "v_accvgpr_read_b32 v24, a208\n"
        "v_accvgpr_read_b32 v25, a212\n"
        "v_accvgpr_read_b32 v30, a209\n"
        "v_accvgpr_read_b32 v31, a213\n"
        "v_accvgpr_read_b32 v36, a210\n"
        "v_accvgpr_read_b32 v37, a214\n"
        "v_accvgpr_read_b32 v42, a211\n"
        "v_accvgpr_read_b32 v43, a215\n"
        "v_pk_mul_f32 v[26:27], v[24:25], v[24:25]\n"
        "v_pk_mul_f32 v[32:33], v[30:31], v[30:31]\n"
        "v_pk_mul_f32 v[38:39], v[36:37], v[36:37]\n"
        "v_pk_mul_f32 v[44:45], v[42:43], v[42:43]\n"
        "v_pk_fma_f32 v[26:27], v[26:27], v[18:19], v[16:17]\n"
        "v_pk_fma_f32 v[32:33], v[32:33], v[18:19], v[16:17]\n"
        "v_pk_fma_f32 v[38:39], v[38:39], v[18:19], v[16:17]\n"
        "v_pk_fma_f32 v[44:45], v[44:45], v[18:19], v[16:17]\n"
        "v_pk_mul_f32 v[26:27], v[26:27], v[24:25]\n"
        "v_pk_mul_f32 v[32:33], v[32:33], v[30:31]\n"
        "v_pk_mul_f32 v[38:39], v[38:39], v[36:37]\n"
        "v_pk_mul_f32 v[44:45], v[44:45], v[42:43]\n"
        "v_exp_f32 v26, v26\n"
        "v_exp_f32 v27, v27\n"
        "v_exp_f32 v32, v32\n"
        "v_exp_f32 v33, v33\n"
        "v_exp_f32 v38, v38\n"
        "v_exp_f32 v39, v39\n"
        "v_exp_f32 v44, v44\n"
        "v_exp_f32 v45, v45\n"
        "v_pk_add_f32 v[26:27], v[26:27], 1.0 op_sel_hi:[1,0]\n"
        "v_pk_add_f32 v[32:33], v[32:33], 1.0 op_sel_hi:[1,0]\n"
        "v_pk_add_f32 v[38:39], v[38:39], 1.0 op_sel_hi:[1,0]\n"
        "v_pk_add_f32 v[44:45], v[44:45], 1.0 op_sel_hi:[1,0]\n"
        "v_rcp_f32 v26, v26\n"
        "v_rcp_f32 v27, v27\n"
        "v_rcp_f32 v32, v32\n"
        "v_rcp_f32 v33, v33\n"
        "v_rcp_f32 v38, v38\n"
        "v_rcp_f32 v39, v39\n"
        "v_rcp_f32 v44, v44\n"
        "v_rcp_f32 v45, v45\n"
        "s_mov_b32 s70, 54525952\n"
        "s_mov_b32 s71, 54657024\n"
        "s_mov_b32 s72, 54788096\n"
        "s_mov_b32 s73, 54919168\n"
        "v_pk_mul_f32 v[28:29], v[26:27], v[24:25]\n"
        "v_pk_mul_f32 v[34:35], v[32:33], v[30:31]\n"
        "v_pk_mul_f32 v[40:41], v[38:39], v[36:37]\n"
        "v_pk_mul_f32 v[46:47], v[44:45], v[42:43]\n"
        "buffer_store_dwordx2 v[28:29], v2, s[44:47], s70 offen nt\n"
        "buffer_store_dwordx2 v[34:35], v2, s[44:47], s71 offen nt\n"
        "buffer_store_dwordx2 v[40:41], v2, s[44:47], s72 offen nt\n"
        "buffer_store_dwordx2 v[46:47], v2, s[44:47], s73 offen nt\n"
        "v_accvgpr_read_b32 v48, a216\n"
        "v_accvgpr_read_b32 v49, a220\n"
        "v_accvgpr_read_b32 v54, a217\n"
        "v_accvgpr_read_b32 v55, a221\n"
        "v_accvgpr_read_b32 v60, a218\n"
        "v_accvgpr_read_b32 v61, a222\n"
        "v_accvgpr_read_b32 v66, a219\n"
        "v_accvgpr_read_b32 v67, a223\n"
        "v_pk_mul_f32 v[50:51], v[48:49], v[48:49]\n"
        "v_pk_mul_f32 v[56:57], v[54:55], v[54:55]\n"
        "v_pk_mul_f32 v[62:63], v[60:61], v[60:61]\n"
        "v_pk_mul_f32 v[68:69], v[66:67], v[66:67]\n"
        "v_pk_fma_f32 v[50:51], v[50:51], v[18:19], v[16:17]\n"
        "v_pk_fma_f32 v[56:57], v[56:57], v[18:19], v[16:17]\n"
        "v_pk_fma_f32 v[62:63], v[62:63], v[18:19], v[16:17]\n"
        "v_pk_fma_f32 v[68:69], v[68:69], v[18:19], v[16:17]\n"
        "v_pk_mul_f32 v[50:51], v[50:51], v[48:49]\n"
        "v_pk_mul_f32 v[56:57], v[56:57], v[54:55]\n"
        "v_pk_mul_f32 v[62:63], v[62:63], v[60:61]\n"
        "v_pk_mul_f32 v[68:69], v[68:69], v[66:67]\n"
        "v_exp_f32 v50, v50\n"
        "v_exp_f32 v51, v51\n"
        "v_exp_f32 v56, v56\n"
        "v_exp_f32 v57, v57\n"
        "v_exp_f32 v62, v62\n"
        "v_exp_f32 v63, v63\n"
        "v_exp_f32 v68, v68\n"
        "v_exp_f32 v69, v69\n"
        "v_pk_add_f32 v[50:51], v[50:51], 1.0 op_sel_hi:[1,0]\n"
        "v_pk_add_f32 v[56:57], v[56:57], 1.0 op_sel_hi:[1,0]\n"
        "v_pk_add_f32 v[62:63], v[62:63], 1.0 op_sel_hi:[1,0]\n"
        "v_pk_add_f32 v[68:69], v[68:69], 1.0 op_sel_hi:[1,0]\n"
        "v_rcp_f32 v50, v50\n"
        "v_rcp_f32 v51, v51\n"
        "v_rcp_f32 v56, v56\n"
        "v_rcp_f32 v57, v57\n"
        "v_rcp_f32 v62, v62\n"
        "v_rcp_f32 v63, v63\n"
        "v_rcp_f32 v68, v68\n"
        "v_rcp_f32 v69, v69\n"
        "s_mov_b32 s70, 56623104\n"
        "s_mov_b32 s71, 56754176\n"
        "s_mov_b32 s72, 56885248\n"
        "s_mov_b32 s73, 57016320\n"
        "v_pk_mul_f32 v[52:53], v[50:51], v[48:49]\n"
        "v_pk_mul_f32 v[58:59], v[56:57], v[54:55]\n"
        "v_pk_mul_f32 v[64:65], v[62:63], v[60:61]\n"
        "v_pk_mul_f32 v[70:71], v[68:69], v[66:67]\n"
        "buffer_store_dwordx2 v[52:53], v2, s[44:47], s70 offen nt\n"
        "buffer_store_dwordx2 v[58:59], v2, s[44:47], s71 offen nt\n"
        "buffer_store_dwordx2 v[64:65], v2, s[44:47], s72 offen nt\n"
        "buffer_store_dwordx2 v[70:71], v2, s[44:47], s73 offen nt\n"
        "v_accvgpr_read_b32 v24, a224\n"
        "v_accvgpr_read_b32 v25, a228\n"
        "v_accvgpr_read_b32 v30, a225\n"
        "v_accvgpr_read_b32 v31, a229\n"
        "v_accvgpr_read_b32 v36, a226\n"
        "v_accvgpr_read_b32 v37, a230\n"
        "v_accvgpr_read_b32 v42, a227\n"
        "v_accvgpr_read_b32 v43, a231\n"
        "v_pk_mul_f32 v[26:27], v[24:25], v[24:25]\n"
        "v_pk_mul_f32 v[32:33], v[30:31], v[30:31]\n"
        "v_pk_mul_f32 v[38:39], v[36:37], v[36:37]\n"
        "v_pk_mul_f32 v[44:45], v[42:43], v[42:43]\n"
        "v_pk_fma_f32 v[26:27], v[26:27], v[18:19], v[16:17]\n"
        "v_pk_fma_f32 v[32:33], v[32:33], v[18:19], v[16:17]\n"
        "v_pk_fma_f32 v[38:39], v[38:39], v[18:19], v[16:17]\n"
        "v_pk_fma_f32 v[44:45], v[44:45], v[18:19], v[16:17]\n"
        "v_pk_mul_f32 v[26:27], v[26:27], v[24:25]\n"
        "v_pk_mul_f32 v[32:33], v[32:33], v[30:31]\n"
        "v_pk_mul_f32 v[38:39], v[38:39], v[36:37]\n"
        "v_pk_mul_f32 v[44:45], v[44:45], v[42:43]\n"
        "v_exp_f32 v26, v26\n"
        "v_exp_f32 v27, v27\n"
        "v_exp_f32 v32, v32\n"
        "v_exp_f32 v33, v33\n"
        "v_exp_f32 v38, v38\n"
        "v_exp_f32 v39, v39\n"
        "v_exp_f32 v44, v44\n"
        "v_exp_f32 v45, v45\n"
        "v_pk_add_f32 v[26:27], v[26:27], 1.0 op_sel_hi:[1,0]\n"
        "v_pk_add_f32 v[32:33], v[32:33], 1.0 op_sel_hi:[1,0]\n"
        "v_pk_add_f32 v[38:39], v[38:39], 1.0 op_sel_hi:[1,0]\n"
        "v_pk_add_f32 v[44:45], v[44:45], 1.0 op_sel_hi:[1,0]\n"
        "v_rcp_f32 v26, v26\n"
        "v_rcp_f32 v27, v27\n"
        "v_rcp_f32 v32, v32\n"
        "v_rcp_f32 v33, v33\n"
        "v_rcp_f32 v38, v38\n"
        "v_rcp_f32 v39, v39\n"
        "v_rcp_f32 v44, v44\n"
        "v_rcp_f32 v45, v45\n"
        "s_mov_b32 s70, 58720256\n"
        "s_mov_b32 s71, 58851328\n"
        "s_mov_b32 s72, 58982400\n"
        "s_mov_b32 s73, 59113472\n"
        "v_pk_mul_f32 v[28:29], v[26:27], v[24:25]\n"
        "v_pk_mul_f32 v[34:35], v[32:33], v[30:31]\n"
        "v_pk_mul_f32 v[40:41], v[38:39], v[36:37]\n"
        "v_pk_mul_f32 v[46:47], v[44:45], v[42:43]\n"
        "buffer_store_dwordx2 v[28:29], v2, s[44:47], s70 offen nt\n"
        "buffer_store_dwordx2 v[34:35], v2, s[44:47], s71 offen nt\n"
        "buffer_store_dwordx2 v[40:41], v2, s[44:47], s72 offen nt\n"
        "buffer_store_dwordx2 v[46:47], v2, s[44:47], s73 offen nt\n"
        "v_accvgpr_read_b32 v48, a232\n"
        "v_accvgpr_read_b32 v49, a236\n"
        "v_accvgpr_read_b32 v54, a233\n"
        "v_accvgpr_read_b32 v55, a237\n"
        "v_accvgpr_read_b32 v60, a234\n"
        "v_accvgpr_read_b32 v61, a238\n"
        "v_accvgpr_read_b32 v66, a235\n"
        "v_accvgpr_read_b32 v67, a239\n"
        "v_pk_mul_f32 v[50:51], v[48:49], v[48:49]\n"
        "v_pk_mul_f32 v[56:57], v[54:55], v[54:55]\n"
        "v_pk_mul_f32 v[62:63], v[60:61], v[60:61]\n"
        "v_pk_mul_f32 v[68:69], v[66:67], v[66:67]\n"
        "v_pk_fma_f32 v[50:51], v[50:51], v[18:19], v[16:17]\n"
        "v_pk_fma_f32 v[56:57], v[56:57], v[18:19], v[16:17]\n"
        "v_pk_fma_f32 v[62:63], v[62:63], v[18:19], v[16:17]\n"
        "v_pk_fma_f32 v[68:69], v[68:69], v[18:19], v[16:17]\n"
        "v_pk_mul_f32 v[50:51], v[50:51], v[48:49]\n"
        "v_pk_mul_f32 v[56:57], v[56:57], v[54:55]\n"
        "v_pk_mul_f32 v[62:63], v[62:63], v[60:61]\n"
        "v_pk_mul_f32 v[68:69], v[68:69], v[66:67]\n"
        "v_exp_f32 v50, v50\n"
        "v_exp_f32 v51, v51\n"
        "v_exp_f32 v56, v56\n"
        "v_exp_f32 v57, v57\n"
        "v_exp_f32 v62, v62\n"
        "v_exp_f32 v63, v63\n"
        "v_exp_f32 v68, v68\n"
        "v_exp_f32 v69, v69\n"
        "v_pk_add_f32 v[50:51], v[50:51], 1.0 op_sel_hi:[1,0]\n"
        "v_pk_add_f32 v[56:57], v[56:57], 1.0 op_sel_hi:[1,0]\n"
        "v_pk_add_f32 v[62:63], v[62:63], 1.0 op_sel_hi:[1,0]\n"
        "v_pk_add_f32 v[68:69], v[68:69], 1.0 op_sel_hi:[1,0]\n"
        "v_rcp_f32 v50, v50\n"
        "v_rcp_f32 v51, v51\n"
        "v_rcp_f32 v56, v56\n"
        "v_rcp_f32 v57, v57\n"
        "v_rcp_f32 v62, v62\n"
        "v_rcp_f32 v63, v63\n"
        "v_rcp_f32 v68, v68\n"
        "v_rcp_f32 v69, v69\n"
        "s_mov_b32 s70, 60817408\n"
        "s_mov_b32 s71, 60948480\n"
        "s_mov_b32 s72, 61079552\n"
        "s_mov_b32 s73, 61210624\n"
        "v_pk_mul_f32 v[52:53], v[50:51], v[48:49]\n"
        "v_pk_mul_f32 v[58:59], v[56:57], v[54:55]\n"
        "v_pk_mul_f32 v[64:65], v[62:63], v[60:61]\n"
        "v_pk_mul_f32 v[70:71], v[68:69], v[66:67]\n"
        "buffer_store_dwordx2 v[52:53], v2, s[44:47], s70 offen nt\n"
        "buffer_store_dwordx2 v[58:59], v2, s[44:47], s71 offen nt\n"
        "buffer_store_dwordx2 v[64:65], v2, s[44:47], s72 offen nt\n"
        "buffer_store_dwordx2 v[70:71], v2, s[44:47], s73 offen nt\n"
        "v_accvgpr_read_b32 v24, a240\n"
        "v_accvgpr_read_b32 v25, a244\n"
        "v_accvgpr_read_b32 v30, a241\n"
        "v_accvgpr_read_b32 v31, a245\n"
        "v_accvgpr_read_b32 v36, a242\n"
        "v_accvgpr_read_b32 v37, a246\n"
        "v_accvgpr_read_b32 v42, a243\n"
        "v_accvgpr_read_b32 v43, a247\n"
        "v_pk_mul_f32 v[26:27], v[24:25], v[24:25]\n"
        "v_pk_mul_f32 v[32:33], v[30:31], v[30:31]\n"
        "v_pk_mul_f32 v[38:39], v[36:37], v[36:37]\n"
        "v_pk_mul_f32 v[44:45], v[42:43], v[42:43]\n"
        "v_pk_fma_f32 v[26:27], v[26:27], v[18:19], v[16:17]\n"
        "v_pk_fma_f32 v[32:33], v[32:33], v[18:19], v[16:17]\n"
        "v_pk_fma_f32 v[38:39], v[38:39], v[18:19], v[16:17]\n"
        "v_pk_fma_f32 v[44:45], v[44:45], v[18:19], v[16:17]\n"
        "v_pk_mul_f32 v[26:27], v[26:27], v[24:25]\n"
        "v_pk_mul_f32 v[32:33], v[32:33], v[30:31]\n"
        "v_pk_mul_f32 v[38:39], v[38:39], v[36:37]\n"
        "v_pk_mul_f32 v[44:45], v[44:45], v[42:43]\n"
        "v_exp_f32 v26, v26\n"
        "v_exp_f32 v27, v27\n"
        "v_exp_f32 v32, v32\n"
        "v_exp_f32 v33, v33\n"
        "v_exp_f32 v38, v38\n"
        "v_exp_f32 v39, v39\n"
        "v_exp_f32 v44, v44\n"
        "v_exp_f32 v45, v45\n"
        "v_pk_add_f32 v[26:27], v[26:27], 1.0 op_sel_hi:[1,0]\n"
        "v_pk_add_f32 v[32:33], v[32:33], 1.0 op_sel_hi:[1,0]\n"
        "v_pk_add_f32 v[38:39], v[38:39], 1.0 op_sel_hi:[1,0]\n"
        "v_pk_add_f32 v[44:45], v[44:45], 1.0 op_sel_hi:[1,0]\n"
        "v_rcp_f32 v26, v26\n"
        "v_rcp_f32 v27, v27\n"
        "v_rcp_f32 v32, v32\n"
        "v_rcp_f32 v33, v33\n"
        "v_rcp_f32 v38, v38\n"
        "v_rcp_f32 v39, v39\n"
        "v_rcp_f32 v44, v44\n"
        "v_rcp_f32 v45, v45\n"
        "s_mov_b32 s70, 62914560\n"
        "s_mov_b32 s71, 63045632\n"
        "s_mov_b32 s72, 63176704\n"
        "s_mov_b32 s73, 63307776\n"
        "v_pk_mul_f32 v[28:29], v[26:27], v[24:25]\n"
        "v_pk_mul_f32 v[34:35], v[32:33], v[30:31]\n"
        "v_pk_mul_f32 v[40:41], v[38:39], v[36:37]\n"
        "v_pk_mul_f32 v[46:47], v[44:45], v[42:43]\n"
        "buffer_store_dwordx2 v[28:29], v2, s[44:47], s70 offen nt\n"
        "buffer_store_dwordx2 v[34:35], v2, s[44:47], s71 offen nt\n"
        "buffer_store_dwordx2 v[40:41], v2, s[44:47], s72 offen nt\n"
        "buffer_store_dwordx2 v[46:47], v2, s[44:47], s73 offen nt\n"
        "v_accvgpr_read_b32 v48, a248\n"
        "v_accvgpr_read_b32 v49, a252\n"
        "v_accvgpr_read_b32 v54, a249\n"
        "v_accvgpr_read_b32 v55, a253\n"
        "v_accvgpr_read_b32 v60, a250\n"
        "v_accvgpr_read_b32 v61, a254\n"
        "v_accvgpr_read_b32 v66, a251\n"
        "v_accvgpr_read_b32 v67, a255\n"
        "v_pk_mul_f32 v[50:51], v[48:49], v[48:49]\n"
        "v_pk_mul_f32 v[56:57], v[54:55], v[54:55]\n"
        "v_pk_mul_f32 v[62:63], v[60:61], v[60:61]\n"
        "v_pk_mul_f32 v[68:69], v[66:67], v[66:67]\n"
        "v_pk_fma_f32 v[50:51], v[50:51], v[18:19], v[16:17]\n"
        "v_pk_fma_f32 v[56:57], v[56:57], v[18:19], v[16:17]\n"
        "v_pk_fma_f32 v[62:63], v[62:63], v[18:19], v[16:17]\n"
        "v_pk_fma_f32 v[68:69], v[68:69], v[18:19], v[16:17]\n"
        "v_pk_mul_f32 v[50:51], v[50:51], v[48:49]\n"
        "v_pk_mul_f32 v[56:57], v[56:57], v[54:55]\n"
        "v_pk_mul_f32 v[62:63], v[62:63], v[60:61]\n"
        "v_pk_mul_f32 v[68:69], v[68:69], v[66:67]\n"
        "v_exp_f32 v50, v50\n"
        "v_exp_f32 v51, v51\n"
        "v_exp_f32 v56, v56\n"
        "v_exp_f32 v57, v57\n"
        "v_exp_f32 v62, v62\n"
        "v_exp_f32 v63, v63\n"
        "v_exp_f32 v68, v68\n"
        "v_exp_f32 v69, v69\n"
        "v_pk_add_f32 v[50:51], v[50:51], 1.0 op_sel_hi:[1,0]\n"
        "v_pk_add_f32 v[56:57], v[56:57], 1.0 op_sel_hi:[1,0]\n"
        "v_pk_add_f32 v[62:63], v[62:63], 1.0 op_sel_hi:[1,0]\n"
        "v_pk_add_f32 v[68:69], v[68:69], 1.0 op_sel_hi:[1,0]\n"
        "v_rcp_f32 v50, v50\n"
        "v_rcp_f32 v51, v51\n"
        "v_rcp_f32 v56, v56\n"
        "v_rcp_f32 v57, v57\n"
        "v_rcp_f32 v62, v62\n"
        "v_rcp_f32 v63, v63\n"
        "v_rcp_f32 v68, v68\n"
        "v_rcp_f32 v69, v69\n"
        "s_mov_b32 s70, 65011712\n"
        "s_mov_b32 s71, 65142784\n"
        "s_mov_b32 s72, 65273856\n"
        "s_mov_b32 s73, 65404928\n"
        "v_pk_mul_f32 v[52:53], v[50:51], v[48:49]\n"
        "v_pk_mul_f32 v[58:59], v[56:57], v[54:55]\n"
        "v_pk_mul_f32 v[64:65], v[62:63], v[60:61]\n"
        "v_pk_mul_f32 v[70:71], v[68:69], v[66:67]\n"
        "buffer_store_dwordx2 v[52:53], v2, s[44:47], s70 offen nt\n"
        "buffer_store_dwordx2 v[58:59], v2, s[44:47], s71 offen nt\n"
        "buffer_store_dwordx2 v[64:65], v2, s[44:47], s72 offen nt\n"
        "buffer_store_dwordx2 v[70:71], v2, s[44:47], s73 offen nt\n"
        "s_waitcnt vmcnt(0)\n"
        :
        : [wave] "s"(wave), [bid] "s"(bid), [rank] "s"(p.rank), [x] "s"(p.x), [w] "s"(p.w), [out] "s"(p.out),
          [wsl] "s"(wsl), [wsd] "s"(wsd), [lds] "s"(lds)
        : "v0", "v1", "v2", "v3", "v4", "v5", "v6", "v7", "v8", "v9", "v10", "v11", "v12", "v13", "v14", "v15", "v16", "v17", "v18", "v19", "v20", "v21", "v22", "v23", "v24", "v25", "v26", "v27", "v28", "v29", "v30", "v31", "v32", "v33", "v34", "v35", "v36", "v37", "v38", "v39", "v40", "v41", "v42", "v43", "v44", "v45", "v46", "v47", "v48", "v49", "v50", "v51", "v52", "v53", "v54", "v55", "v56", "v57", "v58", "v59", "v60", "v61", "v62", "v63", "v64", "v65", "v66", "v67", "v68", "v69", "v70", "v71", "v72", "v73", "v74", "v75", "v76", "v77", "v78", "v79", "v80", "v81", "v82", "v83", "v84", "v85", "v86", "v87", "v88", "v89", "v90", "v91", "v92", "v93", "v94", "v95", "v96", "v97", "v98", "v99", "v100", "v101", "v102", "v103", "v104", "v105", "v106", "v107", "v108", "v109", "v110", "v111", "v112", "v113", "v114", "v115", "v116", "v117", "v118", "v119", "v120", "v121", "v122", "v123", "v124", "v125", "v126", "v127", "v128", "v129", "v130", "v131", "v132", "v133", "v134", "v135", "v136", "v137", "v138", "v139", "v140", "v141", "v142", "v143", "v144", "v145", "v146", "v147", "v148", "v149", "v150", "v151", "v152", "v153", "a0", "a1", "a2", "a3", "a4", "a5", "a6", "a7", "a8", "a9", "a10", "a11", "a12", "a13", "a14", "a15", "a16", "a17", "a18", "a19", "a20", "a21", "a22", "a23", "a24", "a25", "a26", "a27", "a28", "a29", "a30", "a31", "a32", "a33", "a34", "a35", "a36", "a37", "a38", "a39", "a40", "a41", "a42", "a43", "a44", "a45", "a46", "a47", "a48", "a49", "a50", "a51", "a52", "a53", "a54", "a55", "a56", "a57", "a58", "a59", "a60", "a61", "a62", "a63", "a64", "a65", "a66", "a67", "a68", "a69", "a70", "a71", "a72", "a73", "a74", "a75", "a76", "a77", "a78", "a79", "a80", "a81", "a82", "a83", "a84", "a85", "a86", "a87", "a88", "a89", "a90", "a91", "a92", "a93", "a94", "a95", "a96", "a97", "a98", "a99", "a100", "a101", "a102", "a103", "a104", "a105", "a106", "a107", "a108", "a109", "a110", "a111", "a112", "a113", "a114", "a115", "a116", "a117", "a118", "a119", "a120", "a121", "a122", "a123", "a124", "a125", "a126", "a127", "a128", "a129", "a130", "a131", "a132", "a133", "a134", "a135", "a136", "a137", "a138", "a139", "a140", "a141", "a142", "a143", "a144", "a145", "a146", "a147", "a148", "a149", "a150", "a151", "a152", "a153", "a154", "a155", "a156", "a157", "a158", "a159", "a160", "a161", "a162", "a163", "a164", "a165", "a166", "a167", "a168", "a169", "a170", "a171", "a172", "a173", "a174", "a175", "a176", "a177", "a178", "a179", "a180", "a181", "a182", "a183", "a184", "a185", "a186", "a187", "a188", "a189", "a190", "a191", "a192", "a193", "a194", "a195", "a196", "a197", "a198", "a199", "a200", "a201", "a202", "a203", "a204", "a205", "a206", "a207", "a208", "a209", "a210", "a211", "a212", "a213", "a214", "a215", "a216", "a217", "a218", "a219", "a220", "a221", "a222", "a223", "a224", "a225", "a226", "a227", "a228", "a229", "a230", "a231", "a232", "a233", "a234", "a235", "a236", "a237", "a238", "a239", "a240", "a241", "a242", "a243", "a244", "a245", "a246", "a247", "a248", "a249", "a250", "a251", "a252", "a253", "a254", "a255", "s20", "s21", "s22", "s23", "s24", "s25", "s26", "s27", "s28", "s29", "s30", "s31", "s32", "s33", "s34", "s35", "s36", "s37", "s38", "s39", "s40", "s41", "s42", "s43", "s44", "s45", "s46", "s47", "s48", "s49", "s50", "s51", "s52", "s53", "s54", "s55", "s56", "s57", "s58", "s59", "s60", "s61", "s62", "s63", "s64", "s65", "s66", "s67", "s68", "s69", "s70", "s71", "s72", "s73", "s74", "s75", "s76", "s77", "s78", "s79", "s80", "s81", "s82", "s83", "s84", "s85", "s86", "s87", "s88", "s89", "s90", "s91", "s92", "s93", "s94", "s95", "s96", "s97", "s98", "s99", "vcc", "scc", "m0", "memory");
}

extern "C" void kernel_launch(int rank, int nranks, ncclComm_t comm, hipStream_t stream,
                              void* const* d_in, const int* in_sizes, int n_in, void* d_out, int out_size,
                              const int* in_dtypes, int out_dtype, void* const* d_out_all,
                              void* const* d_ws_all, size_t ws_bytes) {
    if (nranks != NR || ws_bytes < WS_NEED || n_in != 2 || (unsigned)rank >= (unsigned)NR ||
        in_sizes[0] != 4096 * 512 || in_sizes[1] != 4096 * 32768 || out_size != 512 * 32768) { fprintf(stderr, "bad config\n"); return; }
    Params p;
    p.x = (const float*)d_in[0]; p.w = (const float*)d_in[1]; p.out = (float*)d_out; p.rank = rank;
    for (int i = 0; i < NR; ++i) p.ws[i] = (unsigned char*)d_ws_all[i];
    fused_a2a_gemm<<<256, 256, 0, stream>>>(p);
}
```
